```python
import numpy as np

kernel_name = "gpu_ag_gemm_m4096_k4096_n4096_e4m3e5m2_dequant"


ABSMAX_RTOL = 0.05

COMPUTE_DTYPE = "float32"
EPILOGUE = "dequant"
IN_QUANT = None
OUT_QUANT = None
IN_FP8 = "e4m3e5m2"
IN_DTYPES = {"x": "e4m3", "w_mat": "e5m2",
             "scale_x": "float32", "scale_w": "float32"}
WORLD_SIZE = 8

def _snap_fp8(x, mant_bits, max_abs, min_normal):
    a = np.abs(x.astype(np.float64))
    _m, _e = np.frexp(a)
    q = np.where(a >= min_normal, np.ldexp(1.0, _e - 1 - mant_bits),
                 min_normal * 2.0 ** -mant_bits)
    snapped = np.minimum(np.round(a / q) * q, max_abs)
    return (np.sign(x) * snapped).astype(np.float32)

def setup_inputs():

    x = _snap_fp8(np.random.randn(4096, 4096).astype(np.float32), 3, 448.0, 2.0 ** -6)
    w_mat = _snap_fp8(np.random.randn(4096, 4096).astype(np.float32), 2, 57344.0, 2.0 ** -14)
    scale_x = (0.02 * (1.0 + np.random.rand(1))).astype(np.float32)
    scale_w = (0.02 * (1.0 + np.random.rand(1))).astype(np.float32)
    return {"x": x, "w_mat": w_mat, "scale_x": scale_x, "scale_w": scale_w}

def reference(x, w_mat, scale_x, scale_w):
    acc = (x @ w_mat).astype(np.float32)
    y = (acc * (scale_x * scale_w)).astype(np.float32)
    return y
```

```cpp
#include <hip/hip_runtime.h>
#include <rccl/rccl.h>
#include <cstdio>

#define HIP_CHECK(expr) do { hipError_t _e = (expr); if (_e != hipSuccess) { \
    fprintf(stderr, "HIP error %s at %s:%d\n", hipGetErrorString(_e), __FILE__, __LINE__); \
    return; } } while (0)

typedef unsigned u32x4 __attribute__((ext_vector_type(4)));
typedef int i32x8 __attribute__((ext_vector_type(8)));
typedef int i32x2 __attribute__((ext_vector_type(2)));
typedef float f32x4 __attribute__((ext_vector_type(4)));
#define LDS_AS __attribute__((address_space(3)))
#define GLB_AS __attribute__((address_space(1)))

constexpr int NR = 8;
constexpr int MSH = 512, KD = 4096, NSH = 512;
constexpr size_t SHARD = (size_t)MSH * KD;
constexpr int NBLK = MSH / 16;
constexpr int NUNIT = NBLK * 8;
constexpr int UNIT = 8192;
constexpr int GRID = 256, BLOCK = 512;
constexpr size_t X_OFF = 0;
constexpr size_t FLAG_OFF = X_OFF + 2 * NR * SHARD;
constexpr size_t CNT_OFF = FLAG_OFF + (size_t)NR * NUNIT * 128;
constexpr size_t TICK_OFF = CNT_OFF + (size_t)GRID * 128;
constexpr size_t TMO_OFF = TICK_OFF + NR * 128;
constexpr size_t DUMMY_OFF = TMO_OFF + 128;
constexpr size_t WS_NEED = DUMMY_OFF + (size_t)GRID * 128;
constexpr unsigned long long DEADLINE_TICKS = 50000000ull;
constexpr int PUSH_WAVES = 2;
constexpr int NPUSH = 8 * PUSH_WAVES;
constexpr int LDS_BYTES = 8 * 16384;
constexpr int PART_STRIDE = 272, PART_BYTES = 16 * PART_STRIDE;

struct Params {
    const unsigned char* x;
    const unsigned char* w;
    const float* sx;
    const float* sw;
    float* out;
    unsigned char* ws[NR];
    int rank;
    int pad;
};

template <int N> __device__ __forceinline__ void wait_vm() { asm volatile("s_waitcnt vmcnt(%0)" ::"i"(N) : "memory"); }

template <int OFF> __device__ __forceinline__ void ld16(u32x4& v, const unsigned char* p) {
    asm volatile("global_load_dwordx4 %0, %1, off offset:%2" : "=v"(v) : "v"(p), "i"(OFF) : "memory");
}
template <int OFF> __device__ __forceinline__ void st16_sys(unsigned char* p, u32x4 v) {
    asm volatile("global_store_dwordx4 %0, %1, off offset:%2 sc0 sc1\n\ts_nop 1" ::"v"(p), "v"(v), "i"(OFF) : "memory");
}
__device__ __forceinline__ void flag_store(unsigned char* p, unsigned v) {
    asm volatile("s_mov_b64 exec, 1\n\tglobal_store_dword %0, %1, off sc0 sc1\n\ts_mov_b64 exec, -1" ::"v"(p), "v"(v) : "memory");
}
__device__ __forceinline__ void ticket(unsigned& v, unsigned char* p) {
    asm volatile("s_mov_b64 exec, 1\n\tglobal_atomic_add %0, %1, %2, off sc0\n\ts_mov_b64 exec, -1" : "=&v"(v) : "v"(p), "v"(1u) : "memory");
}

__device__ __forceinline__ void load_unit_rows(u32x4 (&R)[8], const unsigned char* p) {
    ld16<0>(R[0], p); ld16<64>(R[1], p); ld16<128>(R[2], p); ld16<192>(R[3], p);
    ld16<256>(R[4], p); ld16<320>(R[5], p); ld16<384>(R[6], p); ld16<448>(R[7], p);
}
__device__ __forceinline__ void store_unit(unsigned char* p4096, u32x4 (&R)[8]) {
    st16_sys<-4096>(p4096, R[0]); st16_sys<-3072>(p4096, R[1]); st16_sys<-2048>(p4096, R[2]); st16_sys<-1024>(p4096, R[3]);
    st16_sys<0>(p4096, R[4]); st16_sys<1024>(p4096, R[5]); st16_sys<2048>(p4096, R[6]); st16_sys<3072>(p4096, R[7]);
}
__device__ __forceinline__ void pin8(u32x4 (&R)[8]) {
    asm volatile("" : "+v"(R[0]), "+v"(R[1]), "+v"(R[2]), "+v"(R[3]), "+v"(R[4]), "+v"(R[5]), "+v"(R[6]), "+v"(R[7]));
}

__global__ void __launch_bounds__(BLOCK, 2) ag_gemm(Params p) {
    extern __shared__ __attribute__((aligned(16))) unsigned char smem[];
    const int lane = threadIdx.x & 63;
    const int wave = __builtin_amdgcn_readfirstlane(threadIdx.x >> 6);
    const int bid = blockIdx.x;
    const int xg = bid & 7, idx = bid >> 3;
    unsigned char* const myws = p.ws[0];
    unsigned* const cntp = (unsigned*)(myws + CNT_OFF + (size_t)bid * 128);

    if (idx < 8) {
        if (idx == 7 || wave >= PUSH_WAVES) return;
        unsigned char* const peer = p.ws[idx + 1];
        const int pw = xg * PUSH_WAVES + wave;
        unsigned* const mycnt = cntp + wave;
        const unsigned char* const src = p.x + (size_t)(lane & 15) * KD + (lane >> 4) * 16;
        u32x4 A[8], B[8];
        unsigned ucur = pw, unext = pw + NPUSH;
        load_unit_rows(A, src + (size_t)(ucur >> 3) * (16 * KD) + (ucur & 7) * 512);
        unsigned cv;
        asm volatile("global_load_dword %0, %1, off sc1\n\ts_waitcnt vmcnt(0)" : "=&v"(cv) : "v"(mycnt) : "memory");
        const unsigned call = __builtin_amdgcn_readfirstlane(cv) + 1u;
        const unsigned tbase = (call - 1u) * NUNIT - 2u * NPUSH;
        unsigned char* const dst = peer + X_OFF + (size_t)((call & 1u) * NR + p.rank) * SHARD + 4096 + lane * 16;
        unsigned char* const fl = peer + FLAG_OFF + (size_t)p.rank * NUNIT * 128;
        unsigned char* const tick = myws + TICK_OFF + idx * 128;
        unsigned char* pflag = (unsigned char*)mycnt;
        unsigned tv;
        while (true) {
            pin8(A);
            bool more = unext < NUNIT;
            unsigned un = more ? unext : ucur;
            ticket(tv, tick);
            load_unit_rows(B, src + (size_t)(un >> 3) * (16 * KD) + (un & 7) * 512);
            store_unit(dst + (size_t)ucur * UNIT, A);
            wait_vm<18>();
            flag_store(pflag, call);
            pflag = fl + (size_t)ucur * 128;
            if (!more) break;
            wait_vm<9>();
            asm volatile("" : "+v"(tv));
            ucur = un; unext = __builtin_amdgcn_readfirstlane(tv) - tbase;
            pin8(B);
            more = unext < NUNIT;
            un = more ? unext : ucur;
            ticket(tv, tick);
            load_unit_rows(A, src + (size_t)(un >> 3) * (16 * KD) + (un & 7) * 512);
            store_unit(dst + (size_t)ucur * UNIT, B);
            wait_vm<18>();
            flag_store(pflag, call);
            pflag = fl + (size_t)ucur * 128;
            if (!more) break;
            wait_vm<9>();
            asm volatile("" : "+v"(tv));
            ucur = un; unext = __builtin_amdgcn_readfirstlane(tv) - tbase;
        }
        wait_vm<0>();
        flag_store(pflag, call);
        wait_vm<0>();
        return;
    }

    const int j = idx - 8;
    const int grp = xg * 3 + (j >> 3);
    const int cg = j & 7;
    const bool own = grp >= 21;
    const int srel = own ? 0 : 1 + grp % 7;
    const int slot = own ? grp - 21 : grp / 7;
    const int src_rank = (p.rank + srel) & 7;
    const unsigned long long t0 = __builtin_amdgcn_s_memrealtime();

    i32x8 Wf[4][4];
    LDS_AS unsigned char* const wl = (LDS_AS unsigned char*)smem + wave * 16384;
    unsigned cv = 0;
    {
        const unsigned char* wsrc = p.w + (size_t)(wave * 512 + (lane >> 2)) * NSH + cg * 64 + (lane & 3) * 16;
        const int g = lane >> 4, q = (lane & 15) >> 1, h = lane & 1;
        LDS_AS unsigned char* const rd = wl + (16 * g + q) * 64 + h * 8;
#pragma unroll
        for (int half = 0; half < 2; ++half) {
#pragma unroll
            for (int i = 0; i < 16; ++i)
                __builtin_amdgcn_global_load_lds((const GLB_AS unsigned*)(wsrc + (size_t)(half * 256 + i * 16) * NSH),
                                                 (LDS_AS unsigned*)(wl + i * 1024), 16, 0, 0);
            if (half == 0) cv = __hip_atomic_load(cntp, __ATOMIC_RELAXED, __HIP_MEMORY_SCOPE_AGENT);
            wait_vm<0>();
#pragma unroll
            for (int s2 = 0; s2 < 2; ++s2)
#pragma unroll
                for (int t = 0; t < 4; ++t)
#pragma unroll
                    for (int pc = 0; pc < 4; ++pc) {
                        i32x2 v = __builtin_amdgcn_ds_read_tr8_b64_v2i32(
                            (LDS_AS i32x2*)(rd + (s2 * 128 + (pc >> 1) * 64 + (pc & 1) * 8) * 64 + t * 16));
                        Wf[half * 2 + s2][t][pc * 2] = v.x;
                        Wf[half * 2 + s2][t][pc * 2 + 1] = v.y;
                    }
            asm volatile("s_waitcnt lgkmcnt(0)" ::: "memory");
        }
    }
    const unsigned call = __builtin_amdgcn_readfirstlane(cv) + 1u;
    const float scale = p.sx[0] * p.sw[0];
    unsigned* const tmo = (unsigned*)(myws + TMO_OFF);
    const unsigned char* const xl = own ? p.x + (size_t)(lane & 15) * KD + (lane >> 4) * 16 + wave * 512
                                        : myws + X_OFF + (size_t)((call & 1u) * NR + src_rank) * SHARD + wave * UNIT + 4096 + lane * 16;
    const unsigned* const flags = (const unsigned*)(myws + FLAG_OFF + ((size_t)src_rank * NUNIT + wave) * 128);
    LDS_AS unsigned char* const pw = wl + (lane & 15) * PART_STRIDE + (lane >> 4) * 16;
    LDS_AS unsigned char* const pr = (LDS_AS unsigned char*)smem + (wave * 4 + (lane >> 4)) * PART_STRIDE + (lane & 15) * 16;
    float* const outp = p.out + (size_t)(src_rank * MSH + wave * 4 + (lane >> 4)) * NSH + cg * 64 + (lane & 15) * 4;
    int buf = 0;
    bool bad = false;
    for (int b = slot; b < NBLK; b += 3) {
        u32x4 R[8];
        if (own) {
            const unsigned char* s = xl + (size_t)b * (16 * KD);
            asm volatile(
                "global_load_dwordx4 %0, %8, off\n\tglobal_load_dwordx4 %1, %8, off offset:64\n\t"
                "global_load_dwordx4 %2, %8, off offset:128\n\tglobal_load_dwordx4 %3, %8, off offset:192\n\t"
                "global_load_dwordx4 %4, %8, off offset:256\n\tglobal_load_dwordx4 %5, %8, off offset:320\n\t"
                "global_load_dwordx4 %6, %8, off offset:384\n\tglobal_load_dwordx4 %7, %8, off offset:448\n\t"
                "s_waitcnt vmcnt(0)"
                : "=&v"(R[0]), "=&v"(R[1]), "=&v"(R[2]), "=&v"(R[3]), "=&v"(R[4]), "=&v"(R[5]), "=&v"(R[6]), "=&v"(R[7])
                : "v"(s) : "memory");
        } else {
            const unsigned* f = flags + (size_t)b * (8 * 32);
            unsigned spins = 0;
            while (!bad && __hip_atomic_load(f, __ATOMIC_RELAXED, __HIP_MEMORY_SCOPE_SYSTEM) < call) {
                if ((++spins & 1023u) == 0u &&
                    (__hip_atomic_load(tmo, __ATOMIC_RELAXED, __HIP_MEMORY_SCOPE_AGENT) == call ||
                     __builtin_amdgcn_s_memrealtime() - t0 > DEADLINE_TICKS)) {
                    __hip_atomic_store(tmo, call, __ATOMIC_RELAXED, __HIP_MEMORY_SCOPE_AGENT);
                    bad = true;
                }
                __builtin_amdgcn_s_sleep(1);
            }
            const unsigned char* s = xl + (size_t)b * (8 * UNIT);
            asm volatile(
                "global_load_dwordx4 %0, %8, off offset:-4096 sc0 sc1\n\tglobal_load_dwordx4 %1, %8, off offset:-3072 sc0 sc1\n\t"
                "global_load_dwordx4 %2, %8, off offset:-2048 sc0 sc1\n\tglobal_load_dwordx4 %3, %8, off offset:-1024 sc0 sc1\n\t"
                "global_load_dwordx4 %4, %8, off sc0 sc1\n\tglobal_load_dwordx4 %5, %8, off offset:1024 sc0 sc1\n\t"
                "global_load_dwordx4 %6, %8, off offset:2048 sc0 sc1\n\tglobal_load_dwordx4 %7, %8, off offset:3072 sc0 sc1\n\t"
                "s_waitcnt vmcnt(0)"
                : "=&v"(R[0]), "=&v"(R[1]), "=&v"(R[2]), "=&v"(R[3]), "=&v"(R[4]), "=&v"(R[5]), "=&v"(R[6]), "=&v"(R[7])
                : "v"(s) : "memory");
        }
        f32x4 acc[4] = {};
#pragma unroll
        for (int s = 0; s < 4; ++s) {
            i32x8 xf;
            xf[0] = R[2 * s].x; xf[1] = R[2 * s].y; xf[2] = R[2 * s].z; xf[3] = R[2 * s].w;
            xf[4] = R[2 * s + 1].x; xf[5] = R[2 * s + 1].y; xf[6] = R[2 * s + 1].z; xf[7] = R[2 * s + 1].w;
#pragma unroll
            for (int t = 0; t < 4; ++t)
                acc[t] = __builtin_amdgcn_mfma_scale_f32_16x16x128_f8f6f4(Wf[s][t], xf, acc[t], 1, 0, 0, 0, 0, 0);
        }
        if (bad) {
            const float nan = __builtin_nanf("");
#pragma unroll
            for (int t = 0; t < 4; ++t) acc[t] = f32x4{nan, nan, nan, nan};
        }
#pragma unroll
        for (int t = 0; t < 4; ++t) *(LDS_AS f32x4*)(pw + buf * PART_BYTES + t * 64) = acc[t];
        asm volatile("s_waitcnt lgkmcnt(0)" ::: "memory");
        __builtin_amdgcn_s_barrier();
        asm volatile("" ::: "memory");
        if (wave < 4) {
            f32x4 sum = *(LDS_AS f32x4*)(pr + buf * PART_BYTES);
#pragma unroll
            for (int w2 = 1; w2 < 8; ++w2) sum += *(LDS_AS f32x4*)(pr + buf * PART_BYTES + w2 * 16384);
            sum *= scale;
            *(f32x4*)(outp + (size_t)b * (16 * NSH)) = sum;
        }
        buf ^= 1;
    }
    if (threadIdx.x == 0) __hip_atomic_store(cntp, call, __ATOMIC_RELAXED, __HIP_MEMORY_SCOPE_AGENT);
}

extern "C" void kernel_launch(int rank, int nranks, ncclComm_t comm, hipStream_t stream,
                              void* const* d_in, const int* in_sizes, int n_in,
                              void* d_out, int out_size,
                              const int* in_dtypes, int out_dtype,
                              void* const* d_out_all,
                              void* const* d_ws_all, size_t ws_bytes) {
    (void)comm; (void)in_dtypes; (void)out_dtype; (void)d_out_all;
    if (nranks != NR || (unsigned)rank >= (unsigned)NR || n_in != 4 || ws_bytes < WS_NEED ||
        in_sizes[0] != (int)SHARD || in_sizes[1] != KD * NSH || out_size != NR * MSH * NSH) {
        fprintf(stderr, "unexpected problem shape\n");
        return;
    }
    static bool init = false;
    if (!init) {
        HIP_CHECK(hipFuncSetAttribute((const void*)ag_gemm, hipFuncAttributeMaxDynamicSharedMemorySize, LDS_BYTES));
        init = true;
    }
    Params p;
    p.x = (const unsigned char*)d_in[0];
    p.w = (const unsigned char*)d_in[1];
    p.sx = (const float*)d_in[2];
    p.sw = (const float*)d_in[3];
    p.out = (float*)d_out;
    for (int d = 0; d < NR; ++d) p.ws[d] = (unsigned char*)d_ws_all[(rank + d) % NR];
    p.rank = rank;
    p.pad = 0;
    hipLaunchKernelGGL(ag_gemm, dim3(GRID), dim3(BLOCK), LDS_BYTES, stream, p);
    HIP_CHECK(hipGetLastError());
}
```

```cpp
#include <hip/hip_runtime.h>
#include <rccl/rccl.h>
#include <cstdio>

#define HIP_CHECK(expr) do { hipError_t _e = (expr); if (_e != hipSuccess) { \
    fprintf(stderr, "HIP error %s at %s:%d\n", hipGetErrorString(_e), __FILE__, __LINE__); \
    return; } } while (0)

typedef unsigned u32x4 __attribute__((ext_vector_type(4)));
typedef int i32x8 __attribute__((ext_vector_type(8)));
typedef int i32x2 __attribute__((ext_vector_type(2)));
typedef float f32x4 __attribute__((ext_vector_type(4)));
#define LDS_AS __attribute__((address_space(3)))
#define GLB_AS __attribute__((address_space(1)))

#define PUSH_BLOCKS 3
#define RAW_UNITS 8
#define WIN 8
#define PWIN 12
#define NDEC 4

constexpr int NR = 8;
constexpr int MSH = 512, KD = 4096, NSH = 512;
constexpr size_t SHARD = (size_t)MSH * KD;
constexpr int NBLK = MSH / 16;
constexpr int NUNIT = NBLK * 8;
constexpr int UNIT = 8192;
constexpr int GRID = 256, BLOCK = 512;
constexpr int NCOMM = 64, NCONS = GRID - NCOMM;
constexpr size_t X_OFF = 0;
constexpr size_t C_OFF = X_OFF + 2 * NR * SHARD;
constexpr size_t FLAG_OFF = C_OFF + 2 * SHARD;
constexpr size_t RDY_OFF = FLAG_OFF + (size_t)NR * NUNIT * 128;
constexpr size_t CNT_OFF = RDY_OFF + (size_t)2 * NR * NUNIT * 8;
constexpr size_t TICK_OFF = CNT_OFF + (size_t)GRID * 128;
constexpr size_t TMO_OFF = TICK_OFF + NR * 128;
constexpr size_t WS_NEED = TMO_OFF + 128;
constexpr unsigned long long DEADLINE_TICKS = 50000000ull;
constexpr int NPUSHU = PUSH_BLOCKS * 8;
constexpr int NPULLU = NUNIT - NPUSHU;
constexpr int LDS_BYTES = 8 * 16384;
constexpr int PART_STRIDE = 272, PART_BYTES = 16 * PART_STRIDE;

struct Params {
    const unsigned char* x;
    const unsigned char* w;
    const float* sx;
    const float* sw;
    float* out;
    unsigned char* ws[NR];
    int rank;
    int pad;
};

#define STAMP(i)
#define STAMPV(i, v)
#define NOW() 0ull

template <int N> __device__ __forceinline__ void wait_vm() { asm volatile("s_waitcnt vmcnt(%0)" ::"i"(N) : "memory"); }
__device__ __forceinline__ void wait_lgkm() { asm volatile("s_waitcnt lgkmcnt(0)" ::: "memory"); }

template <int OFF> __device__ __forceinline__ void ld16(u32x4& v, const unsigned char* p) {
    asm volatile("global_load_dwordx4 %0, %1, off offset:%2" : "=v"(v) : "v"(p), "i"(OFF) : "memory");
}
template <int OFF> __device__ __forceinline__ void st16_sys(unsigned char* p, u32x4 v) {
    asm volatile("global_store_dwordx4 %0, %1, off offset:%2 sc0 sc1\n\ts_nop 1" ::"v"(p), "v"(v), "i"(OFF) : "memory");
}
__device__ __forceinline__ void flag_store(unsigned char* p, unsigned v) {
    asm volatile("s_mov_b64 exec, 1\n\tglobal_store_dword %0, %1, off sc0 sc1\n\ts_mov_b64 exec, -1" ::"v"(p), "v"(v) : "memory");
}
__device__ __forceinline__ void ticket(unsigned& v, unsigned char* p) {
    asm volatile("s_mov_b64 exec, 1\n\tglobal_atomic_add %0, %1, %2, off sc0\n\ts_mov_b64 exec, -1" : "=&v"(v) : "v"(p), "v"(1u) : "memory");
}
template <int OFF> __device__ __forceinline__ void glds_sys(const unsigned char* g, unsigned lds) {
    asm volatile("s_mov_b32 m0, %1\n\ts_nop 0\n\tglobal_load_lds_dwordx4 %0, off offset:%2 sc0 sc1" ::"v"(g), "s"(lds), "i"(OFF) : "memory");
}
template <int OFF> __device__ __forceinline__ void glds_sys_mask(const unsigned char* g, unsigned lds, unsigned long long mask) {
    asm volatile("s_mov_b32 m0, %1\n\ts_mov_b64 exec, %3\n\tglobal_load_lds_dwordx4 %0, off offset:%2 sc0 sc1\n\ts_mov_b64 exec, -1"
                 ::"v"(g), "s"(lds), "i"(OFF), "s"(mask) : "memory");
}
__device__ __forceinline__ void load_unit_rows(u32x4 (&R)[8], const unsigned char* p) {
    ld16<0>(R[0], p); ld16<64>(R[1], p); ld16<128>(R[2], p); ld16<192>(R[3], p);
    ld16<256>(R[4], p); ld16<320>(R[5], p); ld16<384>(R[6], p); ld16<448>(R[7], p);
}
__device__ __forceinline__ void store_unit(unsigned char* p4096, u32x4 (&R)[8]) {
    st16_sys<-4096>(p4096, R[0]); st16_sys<-3072>(p4096, R[1]); st16_sys<-2048>(p4096, R[2]); st16_sys<-1024>(p4096, R[3]);
    st16_sys<0>(p4096, R[4]); st16_sys<1024>(p4096, R[5]); st16_sys<2048>(p4096, R[6]); st16_sys<3072>(p4096, R[7]);
}
template <int W> __device__ __forceinline__ void store_unit_win(unsigned char* p4096, u32x4 (&R)[8]) {
    wait_vm<W>(); st16_sys<-4096>(p4096, R[0]); wait_vm<W>(); st16_sys<-3072>(p4096, R[1]);
    wait_vm<W>(); st16_sys<-2048>(p4096, R[2]); wait_vm<W>(); st16_sys<-1024>(p4096, R[3]);
    wait_vm<W>(); st16_sys<0>(p4096, R[4]); wait_vm<W>(); st16_sys<1024>(p4096, R[5]);
    wait_vm<W>(); st16_sys<2048>(p4096, R[6]); wait_vm<W>(); st16_sys<3072>(p4096, R[7]);
}
__device__ __forceinline__ void pin8(u32x4 (&R)[8]) {
    asm volatile("" : "+v"(R[0]), "+v"(R[1]), "+v"(R[2]), "+v"(R[3]), "+v"(R[4]), "+v"(R[5]), "+v"(R[6]), "+v"(R[7]));
}

constexpr int CAP1 = 3072, CAP2 = 1024;
constexpr int CL_A2 = 0;
constexpr int CL_A3 = CAP1 + 16;
constexpr int CL_TB = CL_A3 + CAP1 + 16;
constexpr int CL_DUMMY = CL_TB + 1552;
constexpr int CL_BYTES = CL_DUMMY + 16;
constexpr unsigned INFO_RAW = 0x80000000u;

typedef unsigned u32x2 __attribute__((ext_vector_type(2)));
#define LDS_FENCE() asm volatile("" ::: "memory")

__device__ __forceinline__ unsigned vperm(unsigned hi, unsigned lo, unsigned sel) { return __builtin_amdgcn_perm(hi, lo, sel); }
__device__ __forceinline__ unsigned bfi(unsigned m, unsigned a, unsigned b) { return (a & m) | (b & ~m); }
__device__ __forceinline__ unsigned wave_incl_scan(unsigned v) {
    v += __builtin_amdgcn_update_dpp(0u, v, 0x111, 0xf, 0xf, false);
    v += __builtin_amdgcn_update_dpp(0u, v, 0x112, 0xf, 0xf, false);
    v += __builtin_amdgcn_update_dpp(0u, v, 0x114, 0xf, 0xf, false);
    v += __builtin_amdgcn_update_dpp(0u, v, 0x118, 0xf, 0xf, false);
    v += __builtin_amdgcn_update_dpp(0u, v, 0x142, 0xa, 0xf, false);
    v += __builtin_amdgcn_update_dpp(0u, v, 0x143, 0xc, 0xf, false);
    return v;
}
__device__ __forceinline__ unsigned esc_count(unsigned l, unsigned acc) { return acc + __builtin_popcount(l & (l >> 1) & 0x55555555u); }

__device__ __forceinline__ u32x2 lut_entry(unsigned b, bool enc) {
    unsigned sel = 0, cnt = 0;
#pragma unroll
    for (int k = 0; k < 4; ++k) {
        const unsigned c = (b >> (2 * k)) & 3u;
        if (enc) { if (c == 3u) { sel |= (unsigned)k << (8 * cnt); ++cnt; } }
        else { sel |= (c == 3u ? 4u + cnt : c) << (8 * k); cnt += (c == 3u); }
    }
    if (enc && cnt < 4u) sel |= 0x07060504u << (8 * cnt);
    return u32x2{sel, 0x03020100u + (4u - cnt) * 0x01010101u};
}

__device__ __forceinline__ unsigned lds_addr(const LDS_AS void* p) { return (unsigned)(size_t)p; }
template <int NW>
__device__ __forceinline__ void compact(const unsigned (&P)[NW], const unsigned (&B8)[NW], unsigned pos_end, unsigned c,
                                        LDS_AS unsigned char* dst, LDS_AS unsigned char* dummy, const LDS_AS unsigned char* lut) {
    unsigned w = 0, w2 = 0, u3 = lds_addr(dst) + pos_end + 3u - 4u * NW;
    u32x2 E[NW];
#pragma unroll
    for (int j = NW - 1; j >= 0; --j) E[j] = *(const LDS_AS u32x2*)(lut + B8[j]);
    LDS_FENCE();
#pragma unroll
    for (int j = NW - 1; j >= 0; --j) {
        w2 = vperm(w2, w, E[j].y);
        w = vperm(w, P[j], E[j].x);
        u3 += E[j].y & 0xffu;
        ((LDS_AS unsigned*)(size_t)(u3 & ~3u))[j] = __builtin_amdgcn_alignbyte(w2, w, ~u3);
    }
    LDS_FENCE();
#pragma unroll
    for (int k = 0; k < 7; ++k)
        ((LDS_AS unsigned char*)(size_t)((unsigned)k < c ? u3 - 3u : lds_addr(dummy)))[k] = (unsigned char)((k < 4 ? w : w2) >> (8 * (k & 3)));
    LDS_FENCE();
}

__device__ __forceinline__ unsigned encode_unit(const u32x4 (&R)[8], u32x4 (&C)[7], LDS_AS unsigned char* sc,
                                                const LDS_AS unsigned char* lut, int lane) {
    unsigned X[32];
#pragma unroll
    for (int i = 0; i < 8; ++i) { X[4 * i] = R[i].x; X[4 * i + 1] = R[i].y; X[4 * i + 2] = R[i].z; X[4 * i + 3] = R[i].w; }
#pragma unroll
    for (int i = 0; i < 3; ++i) *(LDS_AS u32x4*)(sc + CL_A2 + lane * 48 + i * 16) = u32x4{0, 0, 0, 0};
    unsigned s = 0;
#pragma unroll
    for (int j = 0; j < 32; j += 4) s = __builtin_amdgcn_udot4(X[j] & 0x78787878u, 0x01010101u, s, false);
    s = __builtin_amdgcn_readlane(wave_incl_scan(s), 63);
    const unsigned mu = s - 819u;
    const unsigned b = (s < 819u ? 0u : mu) >> 14;
    const unsigned t5 = ((mu & 16383u) >= 7373u) ? b + 3u : b - 3u;
    const unsigned t[6] = {b & 15u, (b + 1u) & 15u, (b + 2u) & 15u, (b - 1u) & 15u, (b - 2u) & 15u, t5 & 15u};
    unsigned long long m0 = 0x1b1b1b1b1b1b1b1bull, m1 = m0;
    unsigned info = 0;
#pragma unroll
    for (int i = 0; i < 6; ++i) {
        const unsigned long long x = (unsigned long long)(0x1bu ^ (i < 3 ? (unsigned)i << 3 : 0x18u | (unsigned)(i - 3))) << (8 * (t[i] & 7u));
        if (t[i] & 8u) m1 ^= x; else m0 ^= x;
        info |= t[i] << (4 * i);
    }
    const unsigned M0 = (unsigned)m0, M1 = (unsigned)(m0 >> 32), M2 = (unsigned)m1, M3 = (unsigned)(m1 >> 32);
    unsigned P[32], B8[32], L1[8], N[16];
#pragma unroll
    for (int j = 0; j < 32; ++j) {
        const unsigned x = X[j];
        const unsigned s7 = (x >> 3) & 0x07070707u;
        const unsigned hi = vperm(x << 1, x << 9, 0x0b090a08u);
        const unsigned m = bfi(hi, vperm(M3, M2, s7), vperm(M1, M0, s7));
        B8[j] = __builtin_amdgcn_udot4(m & 0x18181818u, 0x40100401u, 0u, false);
        P[j] = bfi(0x78787878u, x, m);
    }
    unsigned c1 = 0;
#pragma unroll
    for (int i = 0; i < 8; ++i) {
        L1[i] = ((B8[4 * i] | (B8[4 * i + 1] << 8) | (B8[4 * i + 2] << 16)) >> 3) | (B8[4 * i + 3] << 21);
        c1 = esc_count(L1[i], c1);
    }
#pragma unroll
    for (int i = 0; i < 16; ++i) N[i] = bfi(0x87878787u, X[2 * i], __builtin_amdgcn_alignbit(X[2 * i + 1], X[2 * i + 1], 28));
#pragma unroll
    for (int i = 0; i < 4; ++i) C[i] = u32x4{N[4 * i], N[4 * i + 1], N[4 * i + 2], N[4 * i + 3]};
    C[4] = u32x4{L1[0], L1[1], L1[2], L1[3]};
    C[5] = u32x4{L1[4], L1[5], L1[6], L1[7]};
    C[6] = u32x4{0, 0, 0, 0};
    const unsigned e1 = wave_incl_scan(c1);
    const unsigned n1 = __builtin_amdgcn_readlane(e1, 63);
    if (n1 > (unsigned)CAP1) return INFO_RAW;
    LDS_FENCE();
    compact<32>(P, B8, e1, c1, sc + CL_A2, sc + CL_DUMMY, lut);
    unsigned Q[12], B2[12], L2[3];
#pragma unroll
    for (int i = 0; i < 3; ++i) {
        const u32x4 v = *(const LDS_AS u32x4*)(sc + CL_A2 + lane * 48 + i * 16);
        Q[4 * i] = v.x; Q[4 * i + 1] = v.y; Q[4 * i + 2] = v.z; Q[4 * i + 3] = v.w;
    }
    unsigned c2 = 0;
#pragma unroll
    for (int j = 0; j < 12; ++j) B2[j] = __builtin_amdgcn_udot4(Q[j] & 0x03030303u, 0x40100401u, 0u, false);
#pragma unroll
    for (int i = 0; i < 3; ++i) {
        L2[i] = B2[4 * i] | (B2[4 * i + 1] << 8) | (B2[4 * i + 2] << 16) | (B2[4 * i + 3] << 24);
        c2 = esc_count(L2[i], c2);
    }
#pragma unroll
    for (int j = 0; j < 12; ++j) B2[j] <<= 3;
    const unsigned e2 = wave_incl_scan(c2);
    const unsigned n2 = __builtin_amdgcn_readlane(e2, 63);
    const unsigned l3off = ((n1 + 31u) >> 5) << 3;
    const unsigned tail = l3off + (((n2 + 7u) >> 3) << 2);
    if (n2 > (unsigned)CAP2 || tail > 1024u) return INFO_RAW;
    compact<12>(Q, B2, e2, c2, sc + CL_A3, sc + CL_DUMMY, lut);
    const u32x4 a3 = *(const LDS_AS u32x4*)(sc + CL_A3 + lane * 16);
#pragma unroll
    for (int i = 0; i < 3; ++i) *(LDS_AS unsigned*)(sc + CL_TB + lane * 12 + i * 4) = L2[i];
    LDS_FENCE();
    *(LDS_AS u32x2*)(sc + CL_TB + l3off + lane * 8) =
        u32x2{bfi(0x0f0f0f0fu, a3.x >> 3, a3.y << 1), bfi(0x0f0f0f0fu, a3.z >> 3, a3.w << 1)};
    LDS_FENCE();
    C[6] = *(const LDS_AS u32x4*)(sc + CL_TB + lane * 16);
    return info | (((tail + 15u) >> 4) << 24);
}

__device__ __forceinline__ unsigned esc_bits(unsigned l) { return l & (l >> 1) & 0x55555555u; }
__device__ __forceinline__ unsigned lds_window(unsigned q) {
    const LDS_AS unsigned* p = (const LDS_AS unsigned*)(size_t)(q & ~3u);
    return __builtin_amdgcn_alignbyte(p[1], p[0], q);
}
__device__ __forceinline__ void decode_unit(const u32x4 (&C)[8], u32x4 (&R)[8], unsigned info, LDS_AS unsigned char* sc,
                                            const LDS_AS unsigned char* tb, const LDS_AS unsigned char* lut, int lane) {
    const unsigned L1[8] = {C[4].x, C[4].y, C[4].z, C[4].w, C[5].x, C[5].y, C[5].z, C[5].w};
    const unsigned T1 = ((info & 0xfu) << 3) | ((info & 0xf0u) << 7) | ((info & 0xf00u) << 11);
    const unsigned T2 = ((info >> 9) & 0x78u) | ((info >> 5) & 0x7800u) | ((info >> 1) & 0x780000u);
    LDS_FENCE();
    unsigned L2[3];
#pragma unroll
    for (int i = 0; i < 3; ++i) L2[i] = *(const LDS_AS unsigned*)(tb + lane * 12 + i * 4);
    LDS_FENCE();
    unsigned E1[8], c1 = 0;
#pragma unroll
    for (int i = 0; i < 8; ++i) { E1[i] = esc_bits(L1[i]); c1 += __builtin_popcount(E1[i]); }
    const unsigned e1 = wave_incl_scan(c1);
    unsigned n1 = __builtin_amdgcn_readlane(e1, 63);
    n1 = n1 > (unsigned)CAP1 ? (unsigned)CAP1 : n1;
    const unsigned l3off = ((n1 + 31u) >> 5) << 3;
    const u32x2 l3 = *(const LDS_AS u32x2*)(tb + l3off + lane * 8);
    unsigned S2[12];
#pragma unroll
    for (int j = 0; j < 12; ++j) S2[j] = *(const LDS_AS unsigned*)(lut + (((L2[j >> 2] >> (8 * (j & 3))) & 0xffu) << 2));
    LDS_FENCE();
    unsigned E2[3], c2 = 0;
#pragma unroll
    for (int i = 0; i < 3; ++i) { E2[i] = esc_bits(L2[i]); c2 += __builtin_popcount(E2[i]); }
    unsigned pos = wave_incl_scan(c2) - c2 + lds_addr(sc + CL_A3);
    *(LDS_AS u32x4*)(sc + CL_A3 + lane * 16) =
        u32x4{(l3.x << 3) & 0x78787878u, (l3.x >> 1) & 0x78787878u, (l3.y << 3) & 0x78787878u, (l3.y >> 1) & 0x78787878u};
    LDS_FENCE();
    unsigned Q[12];
#pragma unroll
    for (int j = 0; j < 12; ++j) {
        const unsigned q = (j & 3) ? pos + __builtin_popcount(E2[j >> 2] & ((1u << (8 * (j & 3))) - 1u)) : pos;
        Q[j] = lds_window(q);
        if ((j & 3) == 3) pos += __builtin_popcount(E2[j >> 2]);
    }
    LDS_FENCE();
    unsigned S1[32];
#pragma unroll
    for (int j = 0; j < 32; ++j) S1[j] = *(const LDS_AS unsigned*)(lut + (((L1[j >> 2] >> (8 * (j & 3))) & 0xffu) << 2));
    LDS_FENCE();
#pragma unroll
    for (int j = 0; j < 12; ++j) Q[j] = vperm(Q[j], T2, S2[j]);
#pragma unroll
    for (int i = 0; i < 3; ++i) *(LDS_AS u32x4*)(sc + CL_A2 + lane * 48 + i * 16) = u32x4{Q[4 * i], Q[4 * i + 1], Q[4 * i + 2], Q[4 * i + 3]};
    LDS_FENCE();
    pos = e1 - c1;
    pos = (pos > (unsigned)CAP1 ? (unsigned)CAP1 : pos) + lds_addr(sc + CL_A2);
    unsigned X[32];
#pragma unroll
    for (int j = 0; j < 32; ++j) {
        const unsigned q = (j & 3) ? pos + __builtin_popcount(E1[j >> 2] & ((1u << (8 * (j & 3))) - 1u)) : pos;
        X[j] = lds_window(q);
        if ((j & 3) == 3) pos += __builtin_popcount(E1[j >> 2]);
    }
    LDS_FENCE();
    const unsigned N[16] = {C[0].x, C[0].y, C[0].z, C[0].w, C[1].x, C[1].y, C[1].z, C[1].w,
                            C[2].x, C[2].y, C[2].z, C[2].w, C[3].x, C[3].y, C[3].z, C[3].w};
#pragma unroll
    for (int j = 0; j < 32; ++j) {
        const unsigned n = (j & 1) ? __builtin_amdgcn_alignbit(N[j >> 1], N[j >> 1], 4) : N[j >> 1];
        X[j] = (n & 0x87878787u) | vperm(X[j], T1, S1[j]);
    }
#pragma unroll
    for (int i = 0; i < 8; ++i) R[i] = u32x4{X[4 * i], X[4 * i + 1], X[4 * i + 2], X[4 * i + 3]};
}

constexpr int NSLOT = 8;
constexpr int CM_LUT = 0;
constexpr int CM_META = 1024;
constexpr int CM_DONE = CM_META + NSLOT * 16;
constexpr int CM_END = CM_DONE + NSLOT * 4;
constexpr int CM_SLOT = 2048;
constexpr int CM_SC = CM_SLOT + NSLOT * UNIT;
static_assert(CM_SC + NDEC * CL_BYTES <= LDS_BYTES && NDEC >= 3 && NDEC <= 6 && WIN >= 3 && WIN <= 8, "config");

__global__ void __launch_bounds__(BLOCK, 2) ag_gemm(Params p) {
    extern __shared__ __attribute__((aligned(16))) unsigned char smem[];
    const unsigned long long ts0 = NOW();
    const int lane = threadIdx.x & 63;
    const int wave = __builtin_amdgcn_readfirstlane(threadIdx.x >> 6);
    const int bid = blockIdx.x;
    unsigned char* const myws = p.ws[0];
    unsigned* const cntp = (unsigned*)(myws + CNT_OFF + (size_t)bid * 128);
    unsigned* const tmo = (unsigned*)(myws + TMO_OFF);
    const unsigned char* const xrow = p.x + (size_t)(lane & 15) * KD + (lane >> 4) * 16;
    LDS_AS unsigned char* const lds = (LDS_AS unsigned char*)smem;
    const unsigned long long t0 = __builtin_amdgcn_s_memrealtime();

    if (bid < NCOMM) {
        const int c_src = bid >> 3, c_k = bid & 7;
        unsigned char* const peer = p.ws[(c_src + 1) & 7];
        constexpr int PUSHER = NDEC + 1;
        if (wave == PUSHER) {
            u32x4 A[PUSH_BLOCKS][8];
            unsigned cv;
            asm volatile("global_load_dword %0, %1, off sc1" : "=&v"(cv) : "v"(cntp) : "memory");
#pragma unroll
            for (int i = 0; i < PUSH_BLOCKS; ++i) load_unit_rows(A[i], xrow + (size_t)i * (16 * KD) + c_k * 512);
            wait_vm<8 * (PUSH_BLOCKS - 1)>();
            asm volatile("" : "+v"(cv));
            const unsigned call = __builtin_amdgcn_readfirstlane(cv) + 1u;
            const unsigned long long ts1 = NOW();
            unsigned char* const dst = peer + X_OFF + ((size_t)(call & 1u) * NR + p.rank) * SHARD + (size_t)c_k * UNIT + 4096 + lane * 16;
#pragma unroll
            for (int i = 0; i < PUSH_BLOCKS; ++i) {
                if (i == 1) wait_vm<8>();
                pin8(A[i]);
                if (i == 0) store_unit(dst, A[0]); else store_unit_win<PWIN - 1>(dst + (size_t)i * (8 * UNIT), A[i]);
                if (i == 0) __builtin_amdgcn_s_barrier();
            }
            wait_vm<0>();
            const unsigned long long ts2 = NOW();
            unsigned char* fp = peer + FLAG_OFF + ((size_t)p.rank * NUNIT + c_k + 8 * lane) * 128;
            if (lane == PUSH_BLOCKS) fp = (unsigned char*)cntp;
            asm volatile("s_mov_b64 exec, %2\n\tglobal_store_dword %0, %1, off sc0 sc1\n\ts_mov_b64 exec, -1"
                         ::"v"(fp), "v"(call), "i"((2 << PUSH_BLOCKS) - 1) : "memory");
            STAMPV(0, ts0); STAMPV(1, ts1); STAMPV(2, ts2);
            return;
        }
        unsigned cv;
        asm volatile("global_load_dword %0, %1, off sc1" : "=&v"(cv) : "v"(cntp) : "memory");
        if (threadIdx.x < 256) *(LDS_AS unsigned*)(lds + CM_LUT + threadIdx.x * 4) = lut_entry(threadIdx.x, false).x;
        if (wave == 0) *(LDS_AS unsigned*)(lds + CM_META + lane * 4) = 0u;
        asm volatile("s_waitcnt vmcnt(0) lgkmcnt(0)" : "+v"(cv) :: "memory");
        __builtin_amdgcn_s_barrier();
        if (c_src == 7 || wave > NDEC) return;
        const unsigned call = __builtin_amdgcn_readfirstlane(cv) + 1u;
        const int src_rank = (p.rank + c_src + 1) & 7;
        const size_t xo = X_OFF + ((size_t)(call & 1u) * NR + src_rank) * SHARD;
        if (wave == 0) {
            unsigned inf[4];
            unsigned q = 0;
            const unsigned char* rp = myws + RDY_OFF + ((size_t)(call & 1u) * NR + src_rank) * NUNIT * 8 + lane * 32;
            bool ok = true;
            for (unsigned spins = 0;;) {
                u32x4 r0, r1;
                asm volatile("global_load_dwordx4 %0, %2, off sc0 sc1\n\tglobal_load_dwordx4 %1, %2, off offset:16 sc0 sc1\n\ts_waitcnt vmcnt(0)"
                             : "=&v"(r0), "=&v"(r1) : "v"(rp) : "memory");
                inf[0] = r0.x; inf[1] = r0.z; inf[2] = r1.x; inf[3] = r1.z;
                if (__all(lane * 4 < NPUSHU || (r0.y >= call && r0.w >= call && r1.y >= call && r1.w >= call))) break;
                if ((++spins & 1023u) == 0u &&
                    (__hip_atomic_load(tmo, __ATOMIC_RELAXED, __HIP_MEMORY_SCOPE_AGENT) == call ||
                     __builtin_amdgcn_s_memrealtime() - t0 > DEADLINE_TICKS)) {
                    __hip_atomic_store(tmo, call, __ATOMIC_RELAXED, __HIP_MEMORY_SCOPE_AGENT);
                    ok = false;
                    break;
                }
            }
            const unsigned long long ts3 = NOW();
            const unsigned char* const rawp = peer + xo + lane * 16;
            const unsigned char* const cmpp = peer + C_OFF + (size_t)(call & 1u) * SHARD + lane * 16;
            unsigned char* const tick = myws + TICK_OFF + c_src * 128;
            const unsigned tbase = (call - 1u) * NPULLU - 8u - NPUSHU;
            unsigned u = ok ? NPUSHU + c_k : NUNIT;
            unsigned pu = 0, pinfo = 0;
            unsigned tv;
            while (u < NUNIT) {
                const unsigned l4 = u >> 2;
                const unsigned i0 = __builtin_amdgcn_readlane(inf[0], l4), i1 = __builtin_amdgcn_readlane(inf[1], l4);
                const unsigned i2 = __builtin_amdgcn_readlane(inf[2], l4), i3 = __builtin_amdgcn_readlane(inf[3], l4);
                const unsigned info = (u & 2u) ? ((u & 1u) ? i3 : i2) : ((u & 1u) ? i1 : i0);
                const bool raw = (info & INFO_RAW) != 0u;
                const unsigned slot = q & (NSLOT - 1);
                if (q >= NSLOT)
                    while (*(volatile LDS_AS unsigned*)(lds + CM_DONE + slot * 4) != q - NSLOT + 1u) __builtin_amdgcn_s_sleep(1);
                const unsigned char* g = (raw ? rawp : cmpp) + (size_t)u * UNIT;
                const unsigned la = lds_addr(lds + CM_SLOT) + slot * UNIT;
                unsigned nt = raw ? 64u : ((info >> 24) & 127u);
                nt = nt < 1u ? 1u : nt;
                const unsigned long long mask = nt >= 64u ? ~0ull : (1ull << nt) - 1ull;
#define PUBLISH_AT(i) if (WIN - 2 == (i) && q > 0) { if (lane == 0) *(volatile LDS_AS u32x4*)(lds + CM_META + ((q - 1u) & (NSLOT - 1)) * 16) = u32x4{q, pu, pinfo, q}; }
                ticket(tv, tick);
                glds_sys<0>(g, la);
                wait_vm<WIN - 1>(); PUBLISH_AT(1) glds_sys<1024>(g, la);
                wait_vm<WIN - 1>(); PUBLISH_AT(2) glds_sys<2048>(g, la);
                wait_vm<WIN - 1>(); PUBLISH_AT(3) glds_sys<3072>(g, la);
                wait_vm<WIN - 1>(); PUBLISH_AT(4) glds_sys<0>(g + 4096, la + 4096);
                wait_vm<WIN - 1>(); PUBLISH_AT(5) glds_sys<1024>(g + 4096, la + 4096);
                wait_vm<WIN - 1>(); PUBLISH_AT(6) glds_sys_mask<2048>(g + 4096, la + 4096, mask);
                if (raw) { wait_vm<WIN - 1>(); glds_sys<3072>(g + 4096, la + 4096); }
                wait_vm<WIN - 1>();
                asm volatile("" : "+v"(tv));
                pu = u; pinfo = info; ++q;
                u = __builtin_amdgcn_readfirstlane(tv) - tbase;
            }
            wait_vm<0>();
            const unsigned long long ts4 = NOW();
            if (lane == 0) {
                if (q > 0) *(volatile LDS_AS u32x4*)(lds + CM_META + ((q - 1u) & (NSLOT - 1)) * 16) = u32x4{q, pu, pinfo, q};
                *(volatile LDS_AS unsigned*)(lds + CM_END) = q + 1u;
            }
            STAMPV(0, ts0); STAMPV(3, ts3); STAMPV(4, ts4);
            return;
        }
        LDS_AS unsigned char* const sc = lds + CM_SC + (wave - 1) * CL_BYTES;
        unsigned char* const dst = myws + xo + 4096 + lane * 16;
        unsigned char* const fl = myws + FLAG_OFF + (size_t)src_rank * NUNIT * 128;
        unsigned long long ts5 = 0, ts6 = 0;
        for (unsigned q = wave - 1;; q += NDEC) {
            const unsigned slot = q & (NSLOT - 1);
            u32x4 m;
            bool end = false;
            while (true) {
                m = *(volatile LDS_AS u32x4*)(lds + CM_META + slot * 16);
                const unsigned e = *(volatile LDS_AS unsigned*)(lds + CM_END);
                if (__builtin_amdgcn_readfirstlane(m.x) == q + 1u && __builtin_amdgcn_readfirstlane(m.w) == q + 1u) break;
                if (__builtin_amdgcn_readfirstlane(e) - 1u <= q) { end = true; break; }
                __builtin_amdgcn_s_sleep(1);
            }
            if (end) break;
            const unsigned u = __builtin_amdgcn_readfirstlane(m.y), info = __builtin_amdgcn_readfirstlane(m.z);
            const LDS_AS unsigned char* const sl = lds + CM_SLOT + slot * UNIT;
            u32x4 C[8], R[8];
            LDS_FENCE();
            if (info & INFO_RAW) {
#pragma unroll
                for (int i = 0; i < 8; ++i) R[i] = *(const LDS_AS u32x4*)(sl + i * 1024 + lane * 16);
            } else {
#pragma unroll
                for (int i = 5; i >= 0; --i) C[i] = *(const LDS_AS u32x4*)(sl + i * 1024 + lane * 16);
                decode_unit(C, R, info, sc, sl + 6144, lds + CM_LUT, lane);
            }
            pin8(R);
            ts5 = NOW();
            store_unit(dst + (size_t)(u & (NUNIT - 1)) * UNIT, R);
            if (lane == 0) *(volatile LDS_AS unsigned*)(lds + CM_DONE + slot * 4) = q + 1u;
            wait_vm<0>();
            ts6 = NOW();
            flag_store(fl + (size_t)(u & (NUNIT - 1)) * 128, call);
        }
        STAMPV(0, ts0); STAMPV(5, ts5); STAMPV(6, ts6);
        return;
    }

    const int cn = bid - NCOMM;
    unsigned cv;
    unsigned long long ts7 = 0, ts8 = 0, ts9 = 0;
    {
        const unsigned su = NPUSHU + wave * NCONS + cn;
        const bool stage = wave < 2 && su < NUNIT;
        u32x4 A[8];
        if (stage) load_unit_rows(A, xrow + (size_t)(su >> 3) * (16 * KD) + (su & 7) * 512);
        asm volatile("global_load_dword %0, %1, off sc1" : "=&v"(cv) : "v"(cntp) : "memory");
        if (threadIdx.x >= 256) *(LDS_AS u32x2*)(lds + (threadIdx.x - 256) * 8) = lut_entry(threadIdx.x - 256, true);
        asm volatile("s_waitcnt vmcnt(0) lgkmcnt(0)" : "+v"(cv) :: "memory");
        __builtin_amdgcn_s_barrier();
        if (stage) {
            const unsigned call = __builtin_amdgcn_readfirstlane(cv) + 1u;
            pin8(A);
            ts7 = NOW();
            store_unit(myws + X_OFF + ((size_t)(call & 1u) * NR + p.rank) * SHARD + (size_t)su * UNIT + 4096 + lane * 16, A);
            unsigned info = INFO_RAW;
            if (su < NUNIT - RAW_UNITS) {
                u32x4 C[7];
                info = encode_unit(A, C, lds + 2048 + wave * CL_BYTES, lds, lane);
                unsigned char* cp = myws + C_OFF + (size_t)(call & 1u) * SHARD + (size_t)su * UNIT + 4096 + lane * 16;
                ts8 = NOW();
                st16_sys<-4096>(cp, C[0]); st16_sys<-3072>(cp, C[1]); st16_sys<-2048>(cp, C[2]); st16_sys<-1024>(cp, C[3]);
                st16_sys<0>(cp, C[4]); st16_sys<1024>(cp, C[5]); st16_sys<2048>(cp, C[6]);
            }
            unsigned char* fp = p.ws[0] + FLAG_OFF + ((size_t)p.rank * NUNIT + su) * 128;
#pragma unroll
            for (int d = 1; d < NR; ++d) if (lane == d) fp = p.ws[d] + RDY_OFF + (((size_t)(call & 1u) * NR + p.rank) * NUNIT + su) * 8;
            const u32x2 fv = u32x2{lane == 0 ? call : info, call};
            wait_vm<0>();
            ts9 = NOW();
            asm volatile("s_mov_b64 exec, 0xff\n\tglobal_store_dwordx2 %0, %1, off sc0 sc1\n\ts_mov_b64 exec, -1" ::"v"(fp), "v"(fv) : "memory");
            wait_lgkm();
        }
        __builtin_amdgcn_s_barrier();
    }
    const unsigned call = __builtin_amdgcn_readfirstlane(cv) + 1u;
    if (wave == 7) flag_store((unsigned char*)cntp, call);

    const int grp = (cn & 7) * 3 + (cn >> 6);
    const int cg = (cn >> 3) & 7;
    const int srel = grp & 7;
    const int slot = grp >> 3;
    const int src_rank = (p.rank + srel) & 7;

    i32x8 Wf[4][4];
    LDS_AS unsigned char* const wl = lds + wave * 16384;
    {
        const unsigned char* wsrc = p.w + (size_t)(wave * 512 + (lane >> 2)) * NSH + cg * 64 + (lane & 3) * 16;
        const int g = lane >> 4, q = (lane & 15) >> 1, h = lane & 1;
        LDS_AS unsigned char* const rd = wl + (16 * g + q) * 64 + h * 8;
#pragma unroll
        for (int half = 0; half < 2; ++half) {
#pragma unroll
            for (int i = 0; i < 16; ++i)
                __builtin_amdgcn_global_load_lds((const GLB_AS unsigned*)(wsrc + (size_t)(half * 256 + i * 16) * NSH),
                                                 (LDS_AS unsigned*)(wl + i * 1024), 16, 0, 0);
            wait_vm<0>();
#pragma unroll
            for (int s2 = 0; s2 < 2; ++s2)
#pragma unroll
                for (int t = 0; t < 4; ++t)
#pragma unroll
                    for (int pc = 0; pc < 4; ++pc) {
                        i32x2 v = __builtin_amdgcn_ds_read_tr8_b64_v2i32(
                            (LDS_AS i32x2*)(rd + (s2 * 128 + (pc >> 1) * 64 + (pc & 1) * 8) * 64 + t * 16));
                        Wf[half * 2 + s2][t][pc * 2] = v.x;
                        Wf[half * 2 + s2][t][pc * 2 + 1] = v.y;
                    }
            wait_lgkm();
        }
    }
    const float scale = p.sx[0] * p.sw[0];
    const unsigned char* const xl = myws + X_OFF + ((size_t)(call & 1u) * NR + src_rank) * SHARD + wave * UNIT + 4096 + lane * 16;
    const unsigned* const flags = (const unsigned*)(myws + FLAG_OFF + ((size_t)src_rank * NUNIT + wave) * 128);
    LDS_AS unsigned char* const pw = wl + (lane & 15) * PART_STRIDE + (lane >> 4) * 16;
    LDS_AS unsigned char* const pr = lds + (wave * 4 + (lane >> 4)) * PART_STRIDE + (lane & 15) * 16;
    float* const outp = p.out + (size_t)(src_rank * MSH + wave * 4 + (lane >> 4)) * NSH + cg * 64 + (lane & 15) * 4;
    int buf = 0;
    const unsigned long long ts1 = NOW();
    unsigned long long ts3 = 0, ts4 = 0;
    bool bad = false;
    for (int b = slot; b < NBLK; b += 3) {
        u32x4 R[8];
        {
            const unsigned* f = flags + (size_t)b * (8 * 32);
            unsigned spins = 0;
            while (!bad && __hip_atomic_load(f, __ATOMIC_RELAXED, __HIP_MEMORY_SCOPE_SYSTEM) < call) {
                if ((++spins & 1023u) == 0u &&
                    (__hip_atomic_load(tmo, __ATOMIC_RELAXED, __HIP_MEMORY_SCOPE_AGENT) == call ||
                     __builtin_amdgcn_s_memrealtime() - t0 > DEADLINE_TICKS)) {
                    __hip_atomic_store(tmo, call, __ATOMIC_RELAXED, __HIP_MEMORY_SCOPE_AGENT);
                    bad = true;
                }
                __builtin_amdgcn_s_sleep(1);
            }
            ts3 = NOW();
            const unsigned char* s = xl + (size_t)b * (8 * UNIT);
            asm volatile(
                "global_load_dwordx4 %0, %8, off offset:-4096 sc0 sc1\n\tglobal_load_dwordx4 %1, %8, off offset:-3072 sc0 sc1\n\t"
                "global_load_dwordx4 %2, %8, off offset:-2048 sc0 sc1\n\tglobal_load_dwordx4 %3, %8, off offset:-1024 sc0 sc1\n\t"
                "global_load_dwordx4 %4, %8, off sc0 sc1\n\tglobal_load_dwordx4 %5, %8, off offset:1024 sc0 sc1\n\t"
                "global_load_dwordx4 %6, %8, off offset:2048 sc0 sc1\n\tglobal_load_dwordx4 %7, %8, off offset:3072 sc0 sc1\n\t"
                "s_waitcnt vmcnt(0)"
                : "=&v"(R[0]), "=&v"(R[1]), "=&v"(R[2]), "=&v"(R[3]), "=&v"(R[4]), "=&v"(R[5]), "=&v"(R[6]), "=&v"(R[7])
                : "v"(s) : "memory");
        }
        ts4 = NOW();
        f32x4 acc[4] = {};
#pragma unroll
        for (int s = 0; s < 4; ++s) {
            i32x8 xf;
            xf[0] = R[2 * s].x; xf[1] = R[2 * s].y; xf[2] = R[2 * s].z; xf[3] = R[2 * s].w;
            xf[4] = R[2 * s + 1].x; xf[5] = R[2 * s + 1].y; xf[6] = R[2 * s + 1].z; xf[7] = R[2 * s + 1].w;
#pragma unroll
            for (int t = 0; t < 4; ++t)
                acc[t] = __builtin_amdgcn_mfma_scale_f32_16x16x128_f8f6f4(Wf[s][t], xf, acc[t], 1, 0, 0, 0, 0, 0);
        }
        if (bad) {
            const float nan = __builtin_nanf("");
#pragma unroll
            for (int t = 0; t < 4; ++t) acc[t] = f32x4{nan, nan, nan, nan};
        }
#pragma unroll
        for (int t = 0; t < 4; ++t) *(LDS_AS f32x4*)(pw + buf * PART_BYTES + t * 64) = acc[t];
        wait_lgkm();
        __builtin_amdgcn_s_barrier();
        asm volatile("" ::: "memory");
        if (wave < 4) {
            f32x4 sum = *(LDS_AS f32x4*)(pr + buf * PART_BYTES);
#pragma unroll
            for (int w2 = 1; w2 < 8; ++w2) sum += *(LDS_AS f32x4*)(pr + buf * PART_BYTES + w2 * 16384);
            sum *= scale;
            st16_sys<0>((unsigned char*)(outp + (size_t)b * (16 * NSH)), __builtin_bit_cast(u32x4, sum));
        }
        buf ^= 1;
    }
}

extern "C" void kernel_launch(int rank, int nranks, ncclComm_t comm, hipStream_t stream,
                              void* const* d_in, const int* in_sizes, int n_in,
                              void* d_out, int out_size,
                              const int* in_dtypes, int out_dtype,
                              void* const* d_out_all,
                              void* const* d_ws_all, size_t ws_bytes) {
    (void)comm; (void)in_dtypes; (void)out_dtype; (void)d_out_all;
    if (nranks != NR || (unsigned)rank >= (unsigned)NR || n_in != 4 || ws_bytes < WS_NEED ||
        in_sizes[0] != (int)SHARD || in_sizes[1] != KD * NSH || out_size != NR * MSH * NSH) {
        fprintf(stderr, "unexpected problem shape\n");
        return;
    }
    static bool init = false;
    if (!init) {
        HIP_CHECK(hipFuncSetAttribute((const void*)ag_gemm, hipFuncAttributeMaxDynamicSharedMemorySize, LDS_BYTES));
        init = true;
    }
    Params p;
    p.x = (const unsigned char*)d_in[0];
    p.w = (const unsigned char*)d_in[1];
    p.sx = (const float*)d_in[2];
    p.sw = (const float*)d_in[3];
    p.out = (float*)d_out;
    for (int d = 0; d < NR; ++d) p.ws[d] = (unsigned char*)d_ws_all[(rank + d) % NR];
    p.rank = rank;
    p.pad = 0;
    hipLaunchKernelGGL(ag_gemm, dim3(GRID), dim3(BLOCK), LDS_BYTES, stream, p);
    HIP_CHECK(hipGetLastError());
}
```

```cpp
#include <hip/hip_runtime.h>
#include <rccl/rccl.h>
#include <cstdio>

#define HIP_CHECK(expr) do { hipError_t _e = (expr); if (_e != hipSuccess) { \
    fprintf(stderr, "HIP error %s at %s:%d\n", hipGetErrorString(_e), __FILE__, __LINE__); \
    return; } } while (0)

typedef unsigned u32x4 __attribute__((ext_vector_type(4)));
typedef int i32x8 __attribute__((ext_vector_type(8)));
typedef int i32x2 __attribute__((ext_vector_type(2)));
typedef float f32x4 __attribute__((ext_vector_type(4)));
#define LDS_AS __attribute__((address_space(3)))
#define GLB_AS __attribute__((address_space(1)))

#define PUSH_BLOCKS 3
#define RAW_UNITS 8
#define PW2 4
#define CSLEEP 1
#define DSLEEP 1
#define ACKWAIT 1
#define OPENEND 0
#define PBAR 1

constexpr int NR = 8;
constexpr int MSH = 512, KD = 4096, NSH = 512;
constexpr size_t SHARD = (size_t)MSH * KD;
constexpr int NBLK = MSH / 16;
constexpr int NUNIT = NBLK * 8;
constexpr int UNIT = 8192;
constexpr int GRID = 256, BLOCK = 512;
constexpr int NCOMM = 64, NCONS = GRID - NCOMM, NPUSHWG = 64, NSTAGE = NCONS - NPUSHWG;
constexpr size_t X_OFF = 0;
constexpr size_t C_OFF = X_OFF + 2 * NR * SHARD;
constexpr size_t FLAG_OFF = C_OFF + 2 * SHARD;
constexpr size_t RDY_OFF = FLAG_OFF + (size_t)NR * NUNIT * 128;
constexpr size_t CNT_OFF = RDY_OFF + (size_t)2 * NR * NUNIT * 128;
constexpr size_t TICK_OFF = CNT_OFF + (size_t)GRID * 128;
constexpr size_t TMO_OFF = TICK_OFF + NR * 128;
constexpr size_t WS_NEED = TMO_OFF + 128;
constexpr unsigned long long DEADLINE_TICKS = 50000000ull;
constexpr int NPUSHU = PUSH_BLOCKS * 8;
constexpr int NPULLU = NUNIT - NPUSHU;
constexpr int WIN = 8;
constexpr int LDS_BYTES = 8 * 16384;
constexpr int PART_STRIDE = 272, PART_BYTES = 16 * PART_STRIDE;

struct Params {
    const unsigned char* x;
    const unsigned char* w;
    const float* sx;
    const float* sw;
    float* out;
    unsigned char* ws[NR];
    int rank;
    int pad;
};

#define STAMP(i)
#define STAMPV(i, v)
#define NOW() 0ull

template <int N> __device__ __forceinline__ void wait_vm() { asm volatile("s_waitcnt vmcnt(%0)" ::"i"(N) : "memory"); }
__device__ __forceinline__ void wait_lgkm() { asm volatile("s_waitcnt lgkmcnt(0)" ::: "memory"); }

template <int OFF> __device__ __forceinline__ void ld16(u32x4& v, const unsigned char* p) {
    asm volatile("global_load_dwordx4 %0, %1, off offset:%2" : "=v"(v) : "v"(p), "i"(OFF) : "memory");
}
template <int OFF> __device__ __forceinline__ void st16_sys(unsigned char* p, u32x4 v) {
    asm volatile("global_store_dwordx4 %0, %1, off offset:%2 sc0 sc1\n\ts_nop 1" ::"v"(p), "v"(v), "i"(OFF) : "memory");
}
__device__ __forceinline__ void flag_store(unsigned char* p, unsigned v) {
    asm volatile("s_mov_b64 exec, 1\n\tglobal_store_dword %0, %1, off sc0 sc1\n\ts_mov_b64 exec, -1" ::"v"(p), "v"(v) : "memory");
}
__device__ __forceinline__ void ticket(unsigned& v, unsigned char* p) {
    asm volatile("s_mov_b64 exec, 1\n\tglobal_atomic_add %0, %1, %2, off sc0\n\ts_mov_b64 exec, -1" : "=&v"(v) : "v"(p), "v"(1u) : "memory");
}
template <int OFF> __device__ __forceinline__ void glds_sys(const unsigned char* g, unsigned lds) {
    asm volatile("s_mov_b32 m0, %1\n\ts_nop 0\n\tglobal_load_lds_dwordx4 %0, off offset:%2 sc0 sc1" ::"v"(g), "s"(lds), "i"(OFF) : "memory");
}
template <int OFF> __device__ __forceinline__ void glds_sys_mask(const unsigned char* g, unsigned lds, unsigned long long mask) {
    asm volatile("s_mov_b32 m0, %1\n\ts_mov_b64 exec, %3\n\tglobal_load_lds_dwordx4 %0, off offset:%2 sc0 sc1\n\ts_mov_b64 exec, -1"
                 ::"v"(g), "s"(lds), "i"(OFF), "s"(mask) : "memory");
}

constexpr int CAP1 = 1024, CAP2 = 512, CAP3 = 256, TAIL_MAX = 1024;
constexpr int CL_A2 = 0;
constexpr int CL_A3 = CAP1 + 16;
constexpr int CL_A4 = CL_A3 + CAP1 + 16;
constexpr int CL_DUMMY = CL_A4 + CAP2 + 16;
constexpr int CL_BYTES = CL_DUMMY + 16;
constexpr unsigned INFO_RAW = 0x80000000u, INFO_FAIL = 0xffffffffu;
constexpr unsigned HDR_NOFIT = 0x3ffu;
constexpr unsigned rel8(int a, int b, int c) { return (unsigned)(((a & 15) << 3) | ((b & 15) << 11) | ((c & 15) << 19)); }
constexpr unsigned map_byte(int r, bool var) {
    return r <= 2 ? (unsigned)r << 3 : r == 15 ? 0x18u : r == 14 ? 0x19u : r == (var ? 3 : 13) ? 0x1au
         : r == 12 ? 0x1bu : r == 11 ? 0x1fu : r == (var ? 13 : 3) ? 0x9bu : 0x9fu;
}
constexpr unsigned map_word(int r, bool var) {
    return map_byte(r, var) | (map_byte(r + 1, var) << 8) | (map_byte(r + 2, var) << 16) | (map_byte(r + 3, var) << 24);
}

typedef unsigned u32x2 __attribute__((ext_vector_type(2)));
#define LDS_FENCE() asm volatile("" ::: "memory")

__device__ __forceinline__ unsigned vperm(unsigned hi, unsigned lo, unsigned sel) { return __builtin_amdgcn_perm(hi, lo, sel); }
__device__ __forceinline__ unsigned bfi(unsigned m, unsigned a, unsigned b) { return (a & m) | (b & ~m); }
__device__ __forceinline__ unsigned wave_incl_scan(unsigned v) {
    v += __builtin_amdgcn_update_dpp(0u, v, 0x111, 0xf, 0xf, false);
    v += __builtin_amdgcn_update_dpp(0u, v, 0x112, 0xf, 0xf, false);
    v += __builtin_amdgcn_update_dpp(0u, v, 0x114, 0xf, 0xf, false);
    v += __builtin_amdgcn_update_dpp(0u, v, 0x118, 0xf, 0xf, false);
    v += __builtin_amdgcn_update_dpp(0u, v, 0x142, 0xa, 0xf, false);
    v += __builtin_amdgcn_update_dpp(0u, v, 0x143, 0xc, 0xf, false);
    return v;
}
__device__ __forceinline__ unsigned esc_bits(unsigned l) { return l & (l >> 1) & 0x55555555u; }

__device__ __forceinline__ u32x2 lut_entry(unsigned b, bool enc) {
    unsigned sel = 0, cnt = 0;
#pragma unroll
    for (int k = 0; k < 4; ++k) {
        const unsigned c = (b >> (2 * k)) & 3u;
        if (enc) { if (c == 3u) { sel |= (unsigned)k << (8 * cnt); ++cnt; } }
        else { sel |= (c == 3u ? 4u + cnt : c) << (8 * k); cnt += (c == 3u); }
    }
    if (enc && cnt < 4u) sel |= 0x07060504u << (8 * cnt);
    return u32x2{sel, 0x03020100u + (4u - cnt) * 0x01010101u};
}

__device__ __forceinline__ unsigned lds_addr(const LDS_AS void* p) { return (unsigned)(size_t)p; }
template <int NW>
__device__ __forceinline__ void compact(const unsigned (&P)[NW], const unsigned (&B8)[NW], unsigned pos_end, unsigned c,
                                        LDS_AS unsigned char* dst, LDS_AS unsigned char* dummy, const LDS_AS unsigned char* lut) {
    unsigned w = 0, w2 = 0, u3 = lds_addr(dst) + pos_end + 3u - 4u * NW;
    u32x2 E[NW];
#pragma unroll
    for (int j = NW - 1; j >= 0; --j) E[j] = *(const LDS_AS u32x2*)(lut + B8[j]);
    LDS_FENCE();
#pragma unroll
    for (int j = NW - 1; j >= 0; --j) {
        w2 = vperm(w2, w, E[j].y);
        w = vperm(w, P[j], E[j].x);
        u3 += E[j].y & 0xffu;
        ((LDS_AS unsigned*)(size_t)(u3 & ~3u))[j] = __builtin_amdgcn_alignbyte(w2, w, ~u3);
    }
    LDS_FENCE();
#pragma unroll
    for (int k = 0; k < 7; ++k)
        ((LDS_AS unsigned char*)(size_t)((unsigned)k < c ? u3 - 3u : lds_addr(dummy)))[k] = (unsigned char)((k < 4 ? w : w2) >> (8 * (k & 3)));
    LDS_FENCE();
}

struct Packed { u32x4 N; u32x2 L1; unsigned L2, L3, L4, hdr; };

__device__ __forceinline__ void encode_part(const u32x4 R0, const u32x4 R1, Packed& o, LDS_AS unsigned char* sc,
                                            const LDS_AS unsigned char* lut, int lane) {
    const unsigned X[8] = {R0.x, R0.y, R0.z, R0.w, R1.x, R1.y, R1.z, R1.w};
    *(LDS_AS u32x4*)(sc + CL_A2 + lane * 16) = u32x4{0, 0, 0, 0};
    *(LDS_AS u32x2*)(sc + CL_A3 + lane * 8) = u32x2{0, 0};
    unsigned s = 0;
#pragma unroll
    for (int j = 0; j < 8; ++j) s = __builtin_amdgcn_udot4(X[j] & 0x78787878u, 0x01010101u, s, false);
    s = __builtin_amdgcn_readlane(wave_incl_scan(s), 63);
    const unsigned mu = s + (16384u - 819u);
    const unsigned b = ((mu >> 14) - 1u) & 15u;
    const bool var = (mu & 16383u) >= 7373u;
    const unsigned K = ((16u - b) & 15u) * 0x08080808u;
    const unsigned M0 = var ? map_word(0, true) : map_word(0, false), M1 = map_word(4, true), M2 = map_word(8, true),
                   M3 = var ? map_word(12, true) : map_word(12, false);
    unsigned P[8], B8[8];
#pragma unroll
    for (int j = 0; j < 8; ++j) {
        const unsigned y = (X[j] & 0x78787878u) + K;
        const unsigned s7 = (y >> 3) & 0x07070707u;
        const unsigned hi = vperm(y << 1, y << 9, 0x0b090a08u);
        const unsigned m = bfi(hi, vperm(M3, M2, s7), vperm(M1, M0, s7));
        B8[j] = __builtin_amdgcn_udot4(m & 0x18181818u, 0x40100401u, 0u, false);
        P[j] = bfi(0x78787878u, X[j], m);
    }
    o.L1.x = ((B8[0] | (B8[1] << 8) | (B8[2] << 16)) >> 3) | (B8[3] << 21);
    o.L1.y = ((B8[4] | (B8[5] << 8) | (B8[6] << 16)) >> 3) | (B8[7] << 21);
    const unsigned c1 = __builtin_popcount(esc_bits(o.L1.x)) + __builtin_popcount(esc_bits(o.L1.y));
    o.N.x = bfi(0x87878787u, X[0], __builtin_amdgcn_alignbit(X[1], X[1], 28));
    o.N.y = bfi(0x87878787u, X[2], __builtin_amdgcn_alignbit(X[3], X[3], 28));
    o.N.z = bfi(0x87878787u, X[4], __builtin_amdgcn_alignbit(X[5], X[5], 28));
    o.N.w = bfi(0x87878787u, X[6], __builtin_amdgcn_alignbit(X[7], X[7], 28));
    const unsigned e1 = wave_incl_scan(c1);
    const unsigned n1 = __builtin_amdgcn_readlane(e1, 63);
    o.L2 = 0; o.L3 = 0; o.L4 = 0; o.hdr = HDR_NOFIT;
    if (n1 > (unsigned)CAP1) return;
    LDS_FENCE();
    compact<8>(P, B8, e1, c1, sc + CL_A2, sc + CL_DUMMY, lut);
    const u32x4 v = *(const LDS_AS u32x4*)(sc + CL_A2 + lane * 16);
    const unsigned Q[4] = {v.x, v.y, v.z, v.w};
    unsigned B2[4];
#pragma unroll
    for (int j = 0; j < 4; ++j) B2[j] = __builtin_amdgcn_udot4(Q[j] & 0x03030303u, 0x40100401u, 0u, false);
    o.L2 = B2[0] | (B2[1] << 8) | (B2[2] << 16) | (B2[3] << 24);
    const unsigned c2 = __builtin_popcount(esc_bits(o.L2));
#pragma unroll
    for (int j = 0; j < 4; ++j) B2[j] <<= 3;
    const unsigned e2 = wave_incl_scan(c2);
    const unsigned n2 = __builtin_amdgcn_readlane(e2, 63);
    if (n2 > (unsigned)CAP2) return;
    compact<4>(Q, B2, e2, c2, sc + CL_A3, sc + CL_DUMMY, lut);
    const u32x2 a3 = *(const LDS_AS u32x2*)(sc + CL_A3 + lane * 8);
    const unsigned Q3[2] = {a3.x, a3.y};
    unsigned B3[2];
#pragma unroll
    for (int j = 0; j < 2; ++j) B3[j] = __builtin_amdgcn_udot4(bfi(0x01010101u, Q3[j] >> 2, Q3[j] >> 6) & 0x03030303u, 0x40100401u, 0u, false);
    o.L3 = B3[0] | (B3[1] << 8);
    const unsigned c3 = __builtin_popcount(esc_bits(o.L3));
    unsigned n3 = 0;
    if (__any(c3 != 0u)) {
        B3[0] <<= 3; B3[1] <<= 3;
        const unsigned e3 = wave_incl_scan(c3);
        n3 = __builtin_amdgcn_readlane(e3, 63);
        if (n3 > (unsigned)CAP3) return;
        compact<2>(Q3, B3, e3, c3, sc + CL_A4, sc + CL_DUMMY, lut);
        const unsigned a4 = *(const LDS_AS unsigned*)(sc + CL_A4 + lane * 4);
        o.L4 = bfi(0x0f0fu, a4 >> 3, a4 >> 15);
    }
    const unsigned h3 = (n2 + 7u) >> 3, h4 = (n3 + 3u) >> 2;
    o.hdr = ((n1 + 15u) >> 4) | ((h3 + h4) << 10) | (h3 << 20) | (b << 27) | (var ? 0x80000000u : 0u);
}
struct TailMap { unsigned o2, o3, o4, total, hdr; };
__device__ __forceinline__ TailMap tail_map(const u32x4 h, unsigned d) {
    const unsigned a0 = h.x & 0x3ffu, a1 = h.y & 0x3ffu, a2 = h.z & 0x3ffu, a3 = h.w & 0x3ffu;
    const unsigned b0 = (h.x >> 10) & 0x3ffu, b1 = (h.y >> 10) & 0x3ffu, b2 = (h.z >> 10) & 0x3ffu, b3 = (h.w >> 10) & 0x3ffu;
    const unsigned hd = d == 0 ? h.x : d == 1 ? h.y : d == 2 ? h.z : h.w;
    TailMap t;
    const unsigned base3 = 16u + 4u * (a0 + a1 + a2 + a3);
    t.o2 = 16u + 4u * ((d > 0 ? a0 : 0u) + (d > 1 ? a1 : 0u) + (d > 2 ? a2 : 0u));
    t.o3 = base3 + 2u * ((d > 0 ? b0 : 0u) + (d > 1 ? b1 : 0u) + (d > 2 ? b2 : 0u));
    t.o4 = t.o3 + 2u * ((hd >> 20) & 0x7fu);
    t.total = base3 + 2u * (b0 + b1 + b2 + b3);
    t.hdr = ((t.o2 >> 2) & 0xffu) | (((t.o3 >> 1) & 0x1ffu) << 8) | (((hd >> 10) & 0x3ffu) != ((hd >> 20) & 0x7fu) ? 0x20000u : 0u) | (hd & 0xfff00000u);
    return t;
}
__device__ __forceinline__ u32x4 uniform4(const u32x4 v) {
    return u32x4{(unsigned)__builtin_amdgcn_readfirstlane(v.x), (unsigned)__builtin_amdgcn_readfirstlane(v.y),
                 (unsigned)__builtin_amdgcn_readfirstlane(v.z), (unsigned)__builtin_amdgcn_readfirstlane(v.w)};
}

__device__ __forceinline__ unsigned lds_window(unsigned q) {
    const LDS_AS unsigned* p = (const LDS_AS unsigned*)(size_t)(q & ~3u);
    return __builtin_amdgcn_alignbyte(p[1], p[0], q);
}
__device__ __forceinline__ unsigned lut_sel(const LDS_AS unsigned char* lut, unsigned w, int byte) {
    return *(const LDS_AS unsigned*)(lut + (((w >> (8 * byte)) & 0xffu) << 2));
}
template <int NP>
__device__ __forceinline__ void decode_exps(const LDS_AS unsigned char* sl, unsigned d0, unsigned (&X)[NP][8], LDS_AS unsigned char* sc0,
                                            const LDS_AS unsigned char* lut, int lane) {
    const LDS_AS unsigned char* const tb = sl + 6144;
    unsigned hd[NP], L2[NP], L3[NP], S1[NP][8], S2[NP][4], S3[NP][2], E0[NP], E1[NP], p1[NP], T1[NP], T2[NP], T3[NP], w3[NP][2];
    u32x2 L1[NP];
    const LDS_AS unsigned char* s3[NP];
#pragma unroll
    for (int k = 0; k < NP; ++k) {
        hd[k] = *(const LDS_AS unsigned*)(tb + (d0 + k) * 4);
        L1[k] = *(const LDS_AS u32x2*)(sl + 4096 + (d0 + k) * 512 + lane * 8);
    }
    LDS_FENCE();
#pragma unroll
    for (int k = 0; k < NP; ++k) {
        hd[k] = __builtin_amdgcn_readfirstlane(hd[k]);
        s3[k] = tb + ((hd[k] >> 7) & 0x3feu) + lane * 2;
        L3[k] = *(const LDS_AS unsigned short*)s3[k];
        L2[k] = *(const LDS_AS unsigned*)(tb + (hd[k] & 0xffu) * 4 + lane * 4);
    }
    LDS_FENCE();
#pragma unroll
    for (int k = 0; k < NP; ++k) {
        const unsigned kb = ((hd[k] >> 27) & 15u) * 0x080808u;
        const bool var = (hd[k] >> 31) != 0u;
        T1[k] = (rel8(0, 1, 2) + kb) & 0x787878u;
        T2[k] = ((var ? rel8(-1, -2, 3) : rel8(-1, -2, -3)) + kb) & 0x787878u;
        T3[k] = ((var ? rel8(-4, -5, -3) : rel8(-4, -5, 3)) + kb) & 0x787878u;
        E0[k] = esc_bits(L1[k].x); E1[k] = esc_bits(L1[k].y);
        const unsigned c1 = __builtin_popcount(E0[k]) + __builtin_popcount(E1[k]);
        p1[k] = wave_incl_scan(c1) - c1;
        p1[k] = (p1[k] > (unsigned)CAP1 ? (unsigned)CAP1 : p1[k]) + lds_addr(sc0 + k * CL_BYTES + CL_A2);
    }
#pragma unroll
    for (int k = 0; k < NP; ++k) {
        S3[k][0] = lut_sel(lut, L3[k], 0); S3[k][1] = lut_sel(lut, L3[k], 1);
#pragma unroll
        for (int j = 0; j < 4; ++j) S2[k][j] = lut_sel(lut, L2[k], j);
    }
    LDS_FENCE();
#pragma unroll
    for (int k = 0; k < NP; ++k) {
        w3[k][0] = 0; w3[k][1] = 0;
        if (hd[k] & 0x20000u) {
            LDS_AS unsigned char* const sc = sc0 + k * CL_BYTES;
            const unsigned l4 = *(const LDS_AS unsigned short*)(s3[k] + ((hd[k] >> 19) & 0xfeu));
            *(LDS_AS unsigned*)(sc + CL_A4 + lane * 4) = ((l4 << 3) & 0x7878u) | ((l4 << 15) & 0x78780000u);
            LDS_FENCE();
            const unsigned E3 = esc_bits(L3[k]);
            const unsigned c3 = __builtin_popcount(E3);
            const unsigned p3 = wave_incl_scan(c3) - c3 + lds_addr(sc + CL_A4);
            w3[k][0] = lds_window(p3);
            w3[k][1] = lds_window(p3 + __builtin_popcount(E3 & 0xffu));
        }
    }
    unsigned E2[NP], p2[NP];
#pragma unroll
    for (int k = 0; k < NP; ++k) {
        E2[k] = esc_bits(L2[k]);
        const unsigned c2 = __builtin_popcount(E2[k]);
        p2[k] = wave_incl_scan(c2) - c2 + lds_addr(sc0 + k * CL_BYTES + CL_A3);
        *(LDS_AS u32x2*)(sc0 + k * CL_BYTES + CL_A3 + lane * 8) = u32x2{vperm(w3[k][0], T3[k], S3[k][0]), vperm(w3[k][1], T3[k], S3[k][1])};
    }
    LDS_FENCE();
    unsigned Q[NP][4];
#pragma unroll
    for (int k = 0; k < NP; ++k)
#pragma unroll
        for (int j = 0; j < 4; ++j) Q[k][j] = lds_window(j ? p2[k] + __builtin_popcount(E2[k] & ((1u << (8 * j)) - 1u)) : p2[k]);
#pragma unroll
    for (int k = 0; k < NP; ++k)
#pragma unroll
        for (int j = 0; j < 8; ++j) S1[k][j] = lut_sel(lut, (j & 4) ? L1[k].y : L1[k].x, j & 3);
    LDS_FENCE();
#pragma unroll
    for (int k = 0; k < NP; ++k)
        *(LDS_AS u32x4*)(sc0 + k * CL_BYTES + CL_A2 + lane * 16) =
            u32x4{vperm(Q[k][0], T2[k], S2[k][0]), vperm(Q[k][1], T2[k], S2[k][1]), vperm(Q[k][2], T2[k], S2[k][2]), vperm(Q[k][3], T2[k], S2[k][3])};
    LDS_FENCE();
#pragma unroll
    for (int k = 0; k < NP; ++k) {
        const unsigned p1b = p1[k] + __builtin_popcount(E0[k]);
#pragma unroll
        for (int j = 0; j < 8; ++j) {
            const unsigned base = (j & 4) ? p1b : p1[k], E = (j & 4) ? E1[k] : E0[k];
            X[k][j] = vperm(lds_window((j & 3) ? base + __builtin_popcount(E & ((1u << (8 * (j & 3))) - 1u)) : base), T1[k], S1[k][j]);
        }
    }
    LDS_FENCE();
}
__device__ __forceinline__ void merge_part(const u32x4 N, const unsigned (&X)[8], u32x4& R0, u32x4& R1) {
    constexpr unsigned K = 0x87878787u;
    R0 = u32x4{(N.x & K) | X[0], (__builtin_amdgcn_alignbit(N.x, N.x, 4) & K) | X[1], (N.y & K) | X[2], (__builtin_amdgcn_alignbit(N.y, N.y, 4) & K) | X[3]};
    R1 = u32x4{(N.z & K) | X[4], (__builtin_amdgcn_alignbit(N.z, N.z, 4) & K) | X[5], (N.w & K) | X[6], (__builtin_amdgcn_alignbit(N.w, N.w, 4) & K) | X[7]};
}

constexpr int NSLOT = 8;
constexpr int CM_LUT = 0;
constexpr int CM_META = 1024;
constexpr int CM_DONE = CM_META + NSLOT * 16;
constexpr int CM_LAND = CM_DONE + NSLOT * 4;
constexpr int CM_NEXT = CM_LAND + 8;
constexpr int CM_TAB = 2048;
constexpr int CM_SLOT = 3072;
constexpr int CM_SC = CM_SLOT + NSLOT * UNIT;
static_assert(CM_SC + 14 * CL_BYTES <= LDS_BYTES && NPULLU >= NSTAGE && NPULLU <= 2 * NSTAGE, "config");

__device__ __forceinline__ bool timed_out(unsigned& spins, unsigned* tmo, unsigned call, unsigned long long t0) {
    if ((++spins & 1023u) != 0u) return false;
    if (__hip_atomic_load(tmo, __ATOMIC_RELAXED, __HIP_MEMORY_SCOPE_AGENT) != call &&
        __builtin_amdgcn_s_memrealtime() - t0 <= DEADLINE_TICKS) return false;
    __hip_atomic_store(tmo, call, __ATOMIC_RELAXED, __HIP_MEMORY_SCOPE_AGENT);
    return true;
}

__global__ void __launch_bounds__(BLOCK, 2) ag_gemm(Params p) {
    extern __shared__ __attribute__((aligned(16))) unsigned char smem[];
    const unsigned long long ts0 = NOW();
    const int lane = threadIdx.x & 63;
    const int wave = __builtin_amdgcn_readfirstlane(threadIdx.x >> 6);
    const int bid = blockIdx.x;
    unsigned char* const peer = p.ws[((bid >> 3) + 1) & 7];
    asm volatile("" ::"s"(p.x), "s"(p.w), "s"(p.sx), "s"(p.sw), "s"(p.out), "s"(p.rank), "s"(peer), "s"(p.ws[0]), "s"(p.ws[1]),
                 "s"(p.ws[2]), "s"(p.ws[3]), "s"(p.ws[4]), "s"(p.ws[5]), "s"(p.ws[6]), "s"(p.ws[7]));
    unsigned char* const myws = p.ws[0];
    unsigned* const cntp = (unsigned*)(myws + CNT_OFF + (size_t)bid * 128);
    unsigned* const tmo = (unsigned*)(myws + TMO_OFF);
    const unsigned char* const xrow = p.x + (size_t)(lane & 15) * KD + (lane >> 4) * 16;
    LDS_AS unsigned char* const lds = (LDS_AS unsigned char*)smem;
    const unsigned long long t0 = __builtin_amdgcn_s_memrealtime();
    unsigned call;
#define LOAD_COUNT() unsigned cv; asm volatile("global_load_dword %0, %1, off sc1" : "=&v"(cv) : "v"(cntp) : "memory")
    unsigned long long ts2 = 0, ts7 = 0, ts8 = 0, ts9 = 0;

    if (__builtin_expect(bid >= NCOMM && bid < NCOMM + NPUSHWG, 1)) {
        LOAD_COUNT();
        const int c_k = bid & 7;
        u32x4 A[PUSH_BLOCKS];
        const unsigned char* s = xrow + c_k * 512 + wave * 64;
        ld16<0>(A[0], s);
        if (PBAR) __builtin_amdgcn_s_barrier();
#pragma unroll
        for (int i = 1; i < PUSH_BLOCKS; ++i) ld16<0>(A[i], s + (size_t)i * (16 * KD));
        unsigned char* dst = peer + X_OFF + (size_t)p.rank * SHARD + (size_t)c_k * UNIT + wave * 1024 + lane * 16;
        asm volatile("s_waitcnt vmcnt(%3)" : "+v"(cv), "+v"(A[0]), "+v"(dst) : "i"(PUSH_BLOCKS - 1) : "memory");
        call = __builtin_amdgcn_readfirstlane(cv) + 1u;
        dst += (size_t)(call & 1u) * NR * SHARD;
        ts7 = NOW();
        st16_sys<0>(dst, A[0]);
#pragma unroll
        for (int i = 1; i < PUSH_BLOCKS; ++i) {
            if (wave < PW2) wait_vm<1>(); else wait_vm<0>();
            asm volatile("" : "+v"(A[i]));
            st16_sys<0>(dst + (size_t)i * (8 * UNIT), A[i]);
        }
        wait_vm<0>();
        ts2 = NOW();
        __builtin_amdgcn_s_barrier();
        if (wave == 0) {
            unsigned char* fp = peer + FLAG_OFF + ((size_t)p.rank * NUNIT + c_k + 8 * (lane >> 2)) * 128 + (lane & 3) * 8;
            asm volatile("s_mov_b64 exec, %2\n\tglobal_store_dword %0, %1, off sc0 sc1\n\ts_mov_b64 exec, -1"
                         ::"v"(fp), "v"(call), "i"((1 << (4 * PUSH_BLOCKS)) - 1) : "memory");
        }
    } else if (__builtin_expect(bid >= NCOMM, 1)) {
        LOAD_COUNT();
        const unsigned d = wave & 3, sn = bid - NCOMM - NPUSHWG;
        const unsigned su0 = NPUSHU + (wave < 4 ? (sn + 2 * NSTAGE - NPULLU) & (NSTAGE - 1) : NSTAGE + sn);
        const bool stage = su0 < NUNIT;
        const unsigned su = stage ? su0 : NUNIT - 1;
        u32x4 R0, R1;
        {
            const unsigned char* s = xrow + (size_t)(su >> 3) * (16 * KD) + (su & 7) * 512 + d * 128;
            ld16<0>(R0, s); ld16<64>(R1, s);
        }
        if (threadIdx.x >= 256) *(LDS_AS u32x2*)(lds + (threadIdx.x - 256) * 8) = lut_entry(threadIdx.x - 256, true);
        unsigned char* fp = myws + FLAG_OFF + (lane & 3) * 8;
#pragma unroll
        for (int k = 1; k < NR; ++k) if (lane == k) fp = p.ws[k] + RDY_OFF;
        fp += ((size_t)p.rank * NUNIT + su) * 128;
        asm volatile("s_waitcnt vmcnt(0) lgkmcnt(0)" : "+v"(cv), "+v"(R0), "+v"(R1), "+v"(fp) :: "memory");
        __builtin_amdgcn_s_barrier();
        call = __builtin_amdgcn_readfirstlane(cv) + 1u;
        ts7 = NOW();
        Packed o;
        o.hdr = HDR_NOFIT;
        unsigned char* const cp = myws + C_OFF + (size_t)(call & 1u) * SHARD + (size_t)su * UNIT;
        if (stage) {
            unsigned char* x = myws + X_OFF + ((size_t)(call & 1u) * NR + p.rank) * SHARD + (size_t)su * UNIT + d * 2048 + lane * 16;
            st16_sys<0>(x, R0); st16_sys<1024>(x, R1);
        }
        if (stage && su < NUNIT - RAW_UNITS) {
            encode_part(R0, R1, o, lds + 2048 + 64 + wave * CL_BYTES, lds, lane);
            st16_sys<0>(cp + d * 1024 + lane * 16, o.N);
            asm volatile("global_store_dwordx2 %0, %1, off sc0 sc1" ::"v"(cp + 4096 + d * 512 + lane * 8), "v"(o.L1) : "memory");
        }
        ts8 = NOW();
        if (lane == 0) *(LDS_AS unsigned*)(lds + 2048 + wave * 4) = o.hdr;
        wait_lgkm();
        __builtin_amdgcn_s_barrier();
        LDS_FENCE();
        const TailMap tm = tail_map(uniform4(*(const LDS_AS u32x4*)(lds + 2048 + (wave >> 2) * 16)), d);
        const bool fits = tm.total <= (unsigned)TAIL_MAX;
        LDS_AS unsigned char* const tb = lds + 2048 + 64 + 8 * CL_BYTES + (wave >> 2) * 1024;
        if (fits) {
            const unsigned h3 = (o.hdr >> 20) & 0x7fu, h4 = ((o.hdr >> 10) & 0x3ffu) - h3;
            if ((unsigned)lane < (o.hdr & 0x3ffu)) *(LDS_AS unsigned*)(tb + tm.o2 + lane * 4) = o.L2;
            if ((unsigned)lane < h3) *(LDS_AS unsigned short*)(tb + tm.o3 + lane * 2) = (unsigned short)o.L3;
            if ((unsigned)lane < h4) *(LDS_AS unsigned short*)(tb + tm.o4 + lane * 2) = (unsigned short)o.L4;
            if (lane == 0) *(LDS_AS unsigned*)(tb + d * 4) = tm.hdr;
        }
        if (lane < 8) fp += (size_t)(call & 1u) * NR * NUNIT * 128;
        u32x2 fv = u32x2{lane >= 8 ? call : (fits ? (tm.total + 15u) >> 4 : INFO_RAW), call};
        asm volatile("s_waitcnt vmcnt(0) lgkmcnt(0)" : "+v"(fp), "+v"(fv) :: "memory");
        __builtin_amdgcn_s_barrier();
        LDS_FENCE();
        if (d == 0 && fits) {
            st16_sys<0>(cp + 6144 + lane * 16, *(const LDS_AS u32x4*)(tb + lane * 16));
            wait_vm<0>();
        }
        ts9 = NOW();
        if (d == 0 && stage)
            asm volatile("s_mov_b64 exec, 0xffe\n\tglobal_store_dwordx2 %0, %1, off sc0 sc1\n\ts_mov_b64 exec, -1" ::"v"(fp), "v"(fv) : "memory");
        __builtin_amdgcn_s_barrier();
    } else {
        const int c_src = bid >> 3, c_k = bid & 7;
        if (c_src == 7) return;
        LOAD_COUNT();
        if (threadIdx.x < 256) *(LDS_AS unsigned*)(lds + CM_LUT + threadIdx.x * 4) = lut_entry(threadIdx.x, false).x;
        else *(LDS_AS unsigned*)(lds + CM_TAB + (threadIdx.x - 256) * 4) = 0u;
        if (wave == 0) *(LDS_AS unsigned*)(lds + CM_META + lane * 4) = 0u;
        asm volatile("s_waitcnt vmcnt(0) lgkmcnt(0)" : "+v"(cv) :: "memory");
        __builtin_amdgcn_s_barrier();
        call = __builtin_amdgcn_readfirstlane(cv) + 1u;
        const int src_rank = (p.rank + c_src + 1) & 7;
        const size_t xo = X_OFF + ((size_t)(call & 1u) * NR + src_rank) * SHARD;
        if (wave == 0) {
            unsigned long long ts3 = 0;
            const unsigned char* const rawp = peer + xo + lane * 16;
            const unsigned char* const cmpp = peer + C_OFF + (size_t)(call & 1u) * SHARD + lane * 16;
            unsigned char* const tick = myws + TICK_OFF + c_src * 128;
            const unsigned tbase = (call - 1u) * (NPULLU - 8) - 16u - NPUSHU;
            unsigned q = 0;
            unsigned n = WIN;
#define STEP() do { ++n; wait_vm<WIN - 1>(); if (lane == 0) *(volatile LDS_AS unsigned*)(lds + CM_LAND) = n - (WIN - 1); } while (0)
            auto staged = [&](unsigned u) -> unsigned {
                if (u - NPUSHU >= (unsigned)NPULLU) return INFO_FAIL;
                unsigned info;
                while ((info = __builtin_amdgcn_readfirstlane(*(volatile LDS_AS unsigned*)(lds + CM_TAB + u * 4))) == 0u) {}
                return info;
            };
            auto first_half = [&](unsigned u, unsigned info) -> const unsigned char* {
                const unsigned slot = q & (NSLOT - 1);
                if (lane == 0) *(volatile LDS_AS u32x4*)(lds + CM_META + slot * 16) = u32x4{q + 1u, u | (info & INFO_RAW), n, 0u};
                const unsigned la = lds_addr(lds + CM_SLOT) + slot * UNIT + 4096;
                const unsigned char* g = ((info & INFO_RAW) ? rawp : cmpp) + (size_t)u * UNIT;
                if (info & INFO_RAW) {
                    glds_sys<0>(g + 4096, la); STEP(); glds_sys<1024>(g + 4096, la); STEP();
                    glds_sys<2048>(g + 4096, la); STEP(); glds_sys<3072>(g + 4096, la); STEP();
                } else {
                    glds_sys_mask<2048>(g + 4096, la, info >= 64u ? ~0ull : (1ull << info) - 1ull); STEP();
                    glds_sys<0>(g + 4096, la); STEP(); glds_sys<1024>(g + 4096, la); STEP();
                }
                ++q;
                return g;
            };
            auto second_half = [&](unsigned qq, const unsigned char* g) {
                const unsigned slot = qq & (NSLOT - 1);
                if (lane == 0) *(volatile LDS_AS unsigned*)(lds + CM_META + slot * 16 + 12) = n;
                const unsigned la = lds_addr(lds + CM_SLOT) + slot * UNIT;
                glds_sys<0>(g, la); STEP(); glds_sys<1024>(g, la); STEP(); glds_sys<2048>(g, la); STEP(); glds_sys<3072>(g, la); STEP();
            };
            unsigned u = NPUSHU + c_k;
            unsigned info = staged(u);
            ts3 = NOW();
            if (info != INFO_FAIL) {
                const unsigned char* gcur = first_half(u, info);
                u += 8u;
                while ((info = staged(u)) != INFO_FAIL) {
                    if (q >= NSLOT)
                        while (*(volatile LDS_AS unsigned*)(lds + CM_DONE + (q & (NSLOT - 1)) * 4) != 2u * (q / NSLOT)) __builtin_amdgcn_s_sleep(1);
                    unsigned tv;
                    ticket(tv, tick); ++n;
                    const unsigned char* gnext = first_half(u, info);
                    second_half(q - 2u, gcur);
                    asm volatile("" : "+v"(tv));
                    u = __builtin_amdgcn_readfirstlane(tv) - tbase;
                    gcur = gnext;
                }
                second_half(q - 1u, gcur);
            }
            if (lane == 0) *(volatile LDS_AS unsigned*)(lds + CM_LAND + 4) = q + 1u;
#define DRAIN(k) do { wait_vm<k>(); if (lane == 0) *(volatile LDS_AS unsigned*)(lds + CM_LAND) = n - (k); } while (0)
            DRAIN(6); DRAIN(5); DRAIN(4); DRAIN(3); DRAIN(2); DRAIN(1); DRAIN(0);
            STAMP(4); STAMPV(0, ts0); STAMPV(3, ts3); STAMPV(13, q);
            return;
        }
        if (wave == 7) {
            flag_store((unsigned char*)cntp, call);
            const unsigned char* rp = myws + RDY_OFF + ((size_t)(call & 1u) * NR + src_rank) * NUNIT * 128;
            unsigned spins = 0;
            for (int u0 = NPUSHU; u0 < NUNIT; u0 += 64) {
                const int u = u0 + lane < NUNIT ? u0 + lane : NUNIT - 1;
                while (true) {
                    u32x2 r;
                    asm volatile("global_load_dwordx2 %0, %1, off sc0 sc1\n\ts_waitcnt vmcnt(0)" : "=&v"(r) : "v"(rp + (size_t)u * 128) : "memory");
                    const unsigned t = r.y >= call ? r.x : 0u;
                    *(volatile LDS_AS unsigned*)(lds + CM_TAB + u * 4) = t;
                    if (__all(t != 0u)) break;
                    if (timed_out(spins, tmo, call, t0)) {
                        *(volatile LDS_AS u32x4*)(lds + CM_TAB + lane * 16) = u32x4{INFO_FAIL, INFO_FAIL, INFO_FAIL, INFO_FAIL};
                        u0 = NUNIT;
                        break;
                    }
                }
            }
            ts2 = NOW();
        }
        LDS_AS unsigned char* const sc = lds + CM_SC + (wave - 1) * (2 * CL_BYTES);
        unsigned char* const dst = myws + xo + lane * 16;
        unsigned char* const fl = myws + FLAG_OFF + (size_t)src_rank * NUNIT * 128;
        unsigned long long ts5 = 0, ts6 = 0;
        unsigned char* pend = nullptr;
        auto flush = [&]() {
            if (pend) {
                unsigned vm;
                asm volatile("s_getreg_b32 %0, hwreg(HW_REG_IB_STS, 0, 4)" : "=s"(vm));
                if (vm == 0u) { wait_vm<0>(); ts6 = NOW(); flag_store(pend, call); pend = nullptr; }
            } else if (DSLEEP) __builtin_amdgcn_s_sleep(DSLEEP);
        };
        while (true) {
            if (ACKWAIT && pend) { wait_vm<0>(); ts6 = NOW(); flag_store(pend, call); pend = nullptr; }
            unsigned rv = 0;
            if (lane == 0) rv = __hip_atomic_fetch_add((LDS_AS unsigned*)(lds + CM_NEXT), 1u, __ATOMIC_RELAXED, __HIP_MEMORY_SCOPE_WORKGROUP);
            const unsigned r = __builtin_amdgcn_readfirstlane(rv);
            const unsigned q = r >> 1, h = r & 1u;
            const unsigned slot = q & (NSLOT - 1);
            const LDS_AS unsigned char* const sl = lds + CM_SLOT + slot * UNIT;
            unsigned uw;
            bool end = false;
            while (true) {
                const u32x4 m = uniform4(*(volatile LDS_AS u32x4*)(lds + CM_META + slot * 16));
                const u32x2 lv = *(volatile LDS_AS u32x2*)(lds + CM_LAND);
                const unsigned land = __builtin_amdgcn_readfirstlane(lv.x), e = __builtin_amdgcn_readfirstlane(lv.y);
                uw = m.y;
                if (m.x == q + 1u) {
                    if (land >= m.z + ((uw & INFO_RAW) ? 4u * h : 2u + h)) break;
                } else if (e - 1u <= q) { end = true; break; }
                flush();
            }
            if (end) break;
            LDS_FENCE();
            const bool raw = (uw & INFO_RAW) != 0u;
            unsigned X[2][8];
            if (!raw) {
                decode_exps<2>(sl, 2u * h, X, sc, lds + CM_LUT, lane);
#pragma unroll
                for (int k = 0; k < 2; ++k)
                    asm volatile("" : "+v"(X[k][0]), "+v"(X[k][1]), "+v"(X[k][2]), "+v"(X[k][3]), "+v"(X[k][4]), "+v"(X[k][5]), "+v"(X[k][6]), "+v"(X[k][7]));
            }
            ts7 = NOW();
            if (!raw || h == 0u) {
                while (true) {
                    const unsigned ib = __builtin_amdgcn_readfirstlane(*(volatile LDS_AS unsigned*)(lds + CM_META + slot * 16 + 12));
                    const unsigned land = __builtin_amdgcn_readfirstlane(*(volatile LDS_AS unsigned*)(lds + CM_LAND));
                    if (ib != 0u && land >= ib + (raw ? 4u : 2u * h + 2u)) break;
                    if (pend) flush();
                }
                LDS_FENCE();
            }
            u32x4 R[4];
            if (raw) {
#pragma unroll
                for (int i = 0; i < 4; ++i) R[i] = *(const LDS_AS u32x4*)(sl + h * 4096 + i * 1024 + lane * 16);
            } else {
                const u32x4 N0 = *(const LDS_AS u32x4*)(sl + h * 2048 + lane * 16), N1 = *(const LDS_AS u32x4*)(sl + h * 2048 + 1024 + lane * 16);
                merge_part(N0, X[0], R[0], R[1]);
                merge_part(N1, X[1], R[2], R[3]);
            }
            ts5 = NOW();
            const unsigned u = uw & (NUNIT - 1);
            unsigned char* const o = dst + (size_t)u * UNIT + h * 4096;
            st16_sys<0>(o, R[0]); st16_sys<1024>(o, R[1]); st16_sys<2048>(o, R[2]); st16_sys<3072>(o, R[3]);
            if (lane == 0) __hip_atomic_fetch_add((LDS_AS unsigned*)(lds + CM_DONE + slot * 4), 1u, __ATOMIC_RELAXED, __HIP_MEMORY_SCOPE_WORKGROUP);
            if (pend) { wait_vm<4>(); flag_store(pend, call); }
            pend = fl + (size_t)u * 128 + h * 8;
        }
        if (pend) { wait_vm<0>(); ts6 = NOW(); flag_store(pend, call); }
        STAMPV(0, ts0); STAMPV(5, ts5); STAMPV(6, ts6); STAMPV(2, ts2); STAMPV(7, ts7);
        return;
    }

    const int cn = bid - NCOMM;
    const int cg = (cn >> 3) & 7;
    const int src_rank = (p.rank + cn + 3 * (cn >> 6)) & 7;
    if (wave == 7) flag_store((unsigned char*)cntp, call);

    i32x8 Wf[4][4];
    LDS_AS unsigned char* const wl = lds + wave * 16384;
    {
        const unsigned char* wsrc = p.w + (size_t)(wave * 512 + (lane >> 2)) * NSH + cg * 64 + (lane & 3) * 16;
        const int g = lane >> 4, q = (lane & 15) >> 1, h = lane & 1;
        LDS_AS unsigned char* const rd = wl + (16 * g + q) * 64 + h * 8;
#pragma unroll
        for (int half = 0; half < 2; ++half) {
#pragma unroll
            for (int i = 0; i < 16; ++i)
                __builtin_amdgcn_global_load_lds((const GLB_AS unsigned*)(wsrc + (size_t)(half * 256 + i * 16) * NSH),
                                                 (LDS_AS unsigned*)(wl + i * 1024), 16, 0, 0);
            wait_vm<0>();
#pragma unroll
            for (int s2 = 0; s2 < 2; ++s2)
#pragma unroll
                for (int t = 0; t < 4; ++t)
#pragma unroll
                    for (int pc = 0; pc < 4; ++pc) {
                        i32x2 v = __builtin_amdgcn_ds_read_tr8_b64_v2i32(
                            (LDS_AS i32x2*)(rd + (s2 * 128 + (pc >> 1) * 64 + (pc & 1) * 8) * 64 + t * 16));
                        Wf[half * 2 + s2][t][pc * 2] = v.x;
                        Wf[half * 2 + s2][t][pc * 2 + 1] = v.y;
                    }
            wait_lgkm();
        }
    }
    const float scale = p.sx[0] * p.sw[0];
    const unsigned char* const xl = myws + X_OFF + ((size_t)(call & 1u) * NR + src_rank) * SHARD + wave * UNIT + 4096 + lane * 16;
    const unsigned* const flags = (const unsigned*)(myws + FLAG_OFF + ((size_t)src_rank * NUNIT + wave) * 128 + (lane & 1) * 8);
    LDS_AS unsigned char* const pw = wl + (lane & 15) * PART_STRIDE + (lane >> 4) * 16;
    LDS_AS unsigned char* const pr = lds + (wave * 4 + (lane >> 4)) * PART_STRIDE + (lane & 15) * 16;
    float* const outp = p.out + (size_t)(src_rank * MSH + wave * 4 + (lane >> 4)) * NSH + cg * 64 + (lane & 15) * 4;
    int buf = 0;
    bool bad = false;
    const unsigned long long ts1 = NOW();
    unsigned long long ts3 = 0, ts4 = 0;
    for (int b = 2 - (cn >> 6); b < NBLK; b += 3) {
        u32x4 R[8];
        {
            const unsigned* f = flags + (size_t)b * (8 * 32);
            unsigned spins = 0;
            while (!bad && !__all(__hip_atomic_load(f, __ATOMIC_RELAXED, __HIP_MEMORY_SCOPE_SYSTEM) >= call)) {
                bad = timed_out(spins, tmo, call, t0);
                if (CSLEEP) __builtin_amdgcn_s_sleep(CSLEEP);
            }
            ts3 = NOW();
            const unsigned char* s = xl + (size_t)b * (8 * UNIT);
            asm volatile(
                "global_load_dwordx4 %0, %8, off offset:-4096 sc0 sc1\n\tglobal_load_dwordx4 %1, %8, off offset:-3072 sc0 sc1\n\t"
                "global_load_dwordx4 %2, %8, off offset:-2048 sc0 sc1\n\tglobal_load_dwordx4 %3, %8, off offset:-1024 sc0 sc1\n\t"
                "global_load_dwordx4 %4, %8, off sc0 sc1\n\tglobal_load_dwordx4 %5, %8, off offset:1024 sc0 sc1\n\t"
                "global_load_dwordx4 %6, %8, off offset:2048 sc0 sc1\n\tglobal_load_dwordx4 %7, %8, off offset:3072 sc0 sc1\n\t"
                "s_waitcnt vmcnt(0)"
                : "=&v"(R[0]), "=&v"(R[1]), "=&v"(R[2]), "=&v"(R[3]), "=&v"(R[4]), "=&v"(R[5]), "=&v"(R[6]), "=&v"(R[7])
                : "v"(s) : "memory");
        }
        ts4 = NOW();
        f32x4 acc[4] = {};
#pragma unroll
        for (int s = 0; s < 4; ++s) {
            i32x8 xf;
            xf[0] = R[2 * s].x; xf[1] = R[2 * s].y; xf[2] = R[2 * s].z; xf[3] = R[2 * s].w;
            xf[4] = R[2 * s + 1].x; xf[5] = R[2 * s + 1].y; xf[6] = R[2 * s + 1].z; xf[7] = R[2 * s + 1].w;
#pragma unroll
            for (int t = 0; t < 4; ++t)
                acc[t] = __builtin_amdgcn_mfma_scale_f32_16x16x128_f8f6f4(Wf[s][t], xf, acc[t], 1, 0, 0, 0, 0, 0);
        }
        if (bad) {
            const float nan = __builtin_nanf("");
#pragma unroll
            for (int t = 0; t < 4; ++t) acc[t] = f32x4{nan, nan, nan, nan};
        }
#pragma unroll
        for (int t = 0; t < 4; ++t) *(LDS_AS f32x4*)(pw + buf * PART_BYTES + t * 64) = acc[t];
        wait_lgkm();
        __builtin_amdgcn_s_barrier();
        asm volatile("" ::: "memory");
        if (wave < 4) {
            f32x4 sum = *(LDS_AS f32x4*)(pr + buf * PART_BYTES);
#pragma unroll
            for (int w2 = 1; w2 < 8; ++w2) sum += *(LDS_AS f32x4*)(pr + buf * PART_BYTES + w2 * 16384);
            sum *= scale;
            st16_sys<0>((unsigned char*)(outp + (size_t)b * (16 * NSH)), __builtin_bit_cast(u32x4, sum));
        }
        buf ^= 1;
    }
}

extern "C" void kernel_launch(int rank, int nranks, ncclComm_t comm, hipStream_t stream,
                              void* const* d_in, const int* in_sizes, int n_in,
                              void* d_out, int out_size,
                              const int* in_dtypes, int out_dtype,
                              void* const* d_out_all,
                              void* const* d_ws_all, size_t ws_bytes) {
    (void)comm; (void)in_dtypes; (void)out_dtype; (void)d_out_all;
    if (nranks != NR || (unsigned)rank >= (unsigned)NR || n_in != 4 || ws_bytes < WS_NEED ||
        in_sizes[0] != (int)SHARD || in_sizes[1] != KD * NSH || out_size != NR * MSH * NSH) {
        fprintf(stderr, "unexpected problem shape\n");
        return;
    }
    static bool init = false;
    if (!init) {
        HIP_CHECK(hipFuncSetAttribute((const void*)ag_gemm, hipFuncAttributeMaxDynamicSharedMemorySize, LDS_BYTES));
        init = true;
    }
    Params p;
    p.x = (const unsigned char*)d_in[0];
    p.w = (const unsigned char*)d_in[1];
    p.sx = (const float*)d_in[2];
    p.sw = (const float*)d_in[3];
    p.out = (float*)d_out;
    for (int d = 0; d < NR; ++d) p.ws[d] = (unsigned char*)d_ws_all[(rank + d) % NR];
    p.rank = rank;
    p.pad = 0;
    hipLaunchKernelGGL(ag_gemm, dim3(GRID), dim3(BLOCK), LDS_BYTES, stream, p);
    HIP_CHECK(hipGetLastError());
}
```

```cpp
#include <hip/hip_runtime.h>
#include <rccl/rccl.h>
#include <cstdio>

#define HIP_CHECK(expr) do { hipError_t _e = (expr); if (_e != hipSuccess) { \
    fprintf(stderr, "HIP error %s at %s:%d\n", hipGetErrorString(_e), __FILE__, __LINE__); \
    return; } } while (0)

typedef unsigned u32x4 __attribute__((ext_vector_type(4)));
typedef int i32x8 __attribute__((ext_vector_type(8)));
typedef int i32x2 __attribute__((ext_vector_type(2)));
typedef float f32x4 __attribute__((ext_vector_type(4)));
#define LDS_AS __attribute__((address_space(3)))
#define GLB_AS __attribute__((address_space(1)))

#define PUSH_BLOCKS 3
#define RAW_UNITS 8
#define PW2 4
#define CSLEEP 1
#define DSLEEP 1
#define ACKWAIT 1
#define OPENEND 0
#define TOUCH 1
#define PBAR 1

constexpr int NR = 8;
constexpr int MSH = 512, KD = 4096, NSH = 512;
constexpr size_t SHARD = (size_t)MSH * KD;
constexpr int NBLK = MSH / 16;
constexpr int NUNIT = NBLK * 8;
constexpr int UNIT = 8192;
constexpr int GRID = 256, BLOCK = 512;
constexpr int NCOMM = 64, NCONS = GRID - NCOMM, NPUSHWG = 64, NSTAGE = NCONS - NPUSHWG;
constexpr size_t X_OFF = 0;
constexpr size_t C_OFF = X_OFF + 2 * NR * SHARD;
constexpr size_t FLAG_OFF = C_OFF + 2 * SHARD;
constexpr size_t RDY_OFF = FLAG_OFF + (size_t)NR * NUNIT * 128;
constexpr size_t CNT_OFF = RDY_OFF + (size_t)2 * NR * NUNIT * 128;
constexpr size_t TICK_OFF = CNT_OFF + (size_t)GRID * 128;
constexpr size_t TMO_OFF = TICK_OFF + NR * 128;
constexpr size_t WS_NEED = TMO_OFF + 128;
constexpr unsigned long long DEADLINE_TICKS = 50000000ull;
constexpr int NPUSHU = PUSH_BLOCKS * 8;
constexpr int NPULLU = NUNIT - NPUSHU;
#define WINDOW 8
constexpr int WIN = WINDOW;
constexpr int LDS_BYTES = 8 * 16384;
constexpr int PART_STRIDE = 272, PART_BYTES = 16 * PART_STRIDE;

struct Params {
    const unsigned char* x;
    const unsigned char* w;
    const float* sx;
    const float* sw;
    float* out;
    unsigned char* ws[NR];
    int rank;
    int pad;
};

#define STAMP(i)
#define STAMPV(i, v)
#define NOW() 0ull

template <int N> __device__ __forceinline__ void wait_vm() { asm volatile("s_waitcnt vmcnt(%0)" ::"i"(N) : "memory"); }
__device__ __forceinline__ void wait_lgkm() { asm volatile("s_waitcnt lgkmcnt(0)" ::: "memory"); }

template <int OFF> __device__ __forceinline__ void ld16(u32x4& v, const unsigned char* p) {
    asm volatile("global_load_dwordx4 %0, %1, off offset:%2" : "=v"(v) : "v"(p), "i"(OFF) : "memory");
}
template <int OFF> __device__ __forceinline__ void st16_sys(unsigned char* p, u32x4 v) {
    asm volatile("global_store_dwordx4 %0, %1, off offset:%2 sc0 sc1\n\ts_nop 1" ::"v"(p), "v"(v), "i"(OFF) : "memory");
}
__device__ __forceinline__ void flag_store(unsigned char* p, unsigned v) {
    asm volatile("s_mov_b64 exec, 1\n\tglobal_store_dword %0, %1, off sc0 sc1\n\ts_mov_b64 exec, -1" ::"v"(p), "v"(v) : "memory");
}
__device__ __forceinline__ void ticket(unsigned& v, unsigned char* p) {
    asm volatile("s_mov_b64 exec, 1\n\tglobal_atomic_add %0, %1, %2, off sc0\n\ts_mov_b64 exec, -1" : "=&v"(v) : "v"(p), "v"(1u) : "memory");
}
template <int OFF> __device__ __forceinline__ void glds_sys(const unsigned char* g, unsigned lds) {
    asm volatile("s_mov_b32 m0, %1\n\ts_nop 0\n\tglobal_load_lds_dwordx4 %0, off offset:%2 sc0 sc1" ::"v"(g), "s"(lds), "i"(OFF) : "memory");
}
template <int OFF> __device__ __forceinline__ void glds_sys_mask(const unsigned char* g, unsigned lds, unsigned long long mask) {
    asm volatile("s_mov_b32 m0, %1\n\ts_mov_b64 exec, %3\n\tglobal_load_lds_dwordx4 %0, off offset:%2 sc0 sc1\n\ts_mov_b64 exec, -1"
                 ::"v"(g), "s"(lds), "i"(OFF), "s"(mask) : "memory");
}

constexpr int CAP1 = 1024, CAP2 = 512, CAP3 = 256, TAIL_MAX = 1024;
constexpr int CL_A2 = 0;
constexpr int CL_A3 = CAP1 + 16;
constexpr int CL_A4 = CL_A3 + CAP1 + 16;
constexpr int CL_DUMMY = CL_A4 + CAP2 + 16;
constexpr int CL_BYTES = CL_DUMMY + 16;
constexpr unsigned INFO_RAW = 0x80000000u, INFO_FAIL = 0xffffffffu;
constexpr unsigned HDR_NOFIT = 0x3ffu;
constexpr unsigned rel8(int a, int b, int c) { return (unsigned)(((a & 15) << 3) | ((b & 15) << 11) | ((c & 15) << 19)); }
constexpr unsigned map_byte(int r, bool var) {
    return r <= 2 ? (unsigned)r << 3 : r == 15 ? 0x18u : r == 14 ? 0x19u : r == (var ? 3 : 13) ? 0x1au
         : r == 12 ? 0x1bu : r == 11 ? 0x1fu : r == (var ? 13 : 3) ? 0x9bu : 0x9fu;
}
constexpr unsigned map_word(int r, bool var) {
    return map_byte(r, var) | (map_byte(r + 1, var) << 8) | (map_byte(r + 2, var) << 16) | (map_byte(r + 3, var) << 24);
}

typedef unsigned u32x2 __attribute__((ext_vector_type(2)));
#define LDS_FENCE() asm volatile("" ::: "memory")

__device__ __forceinline__ unsigned vperm(unsigned hi, unsigned lo, unsigned sel) { return __builtin_amdgcn_perm(hi, lo, sel); }
__device__ __forceinline__ unsigned bfi(unsigned m, unsigned a, unsigned b) { return (a & m) | (b & ~m); }
__device__ __forceinline__ unsigned wave_incl_scan(unsigned v) {
    v += __builtin_amdgcn_update_dpp(0u, v, 0x111, 0xf, 0xf, false);
    v += __builtin_amdgcn_update_dpp(0u, v, 0x112, 0xf, 0xf, false);
    v += __builtin_amdgcn_update_dpp(0u, v, 0x114, 0xf, 0xf, false);
    v += __builtin_amdgcn_update_dpp(0u, v, 0x118, 0xf, 0xf, false);
    v += __builtin_amdgcn_update_dpp(0u, v, 0x142, 0xa, 0xf, false);
    v += __builtin_amdgcn_update_dpp(0u, v, 0x143, 0xc, 0xf, false);
    return v;
}
__device__ __forceinline__ unsigned esc_bits(unsigned l) { return l & (l >> 1) & 0x55555555u; }

__device__ __forceinline__ u32x2 lut_entry(unsigned b, bool enc) {
    unsigned sel = 0, cnt = 0;
#pragma unroll
    for (int k = 0; k < 4; ++k) {
        const unsigned c = (b >> (2 * k)) & 3u;
        if (enc) { if (c == 3u) { sel |= (unsigned)k << (8 * cnt); ++cnt; } }
        else { sel |= (c == 3u ? 4u + cnt : c) << (8 * k); cnt += (c == 3u); }
    }
    if (enc && cnt < 4u) sel |= 0x07060504u << (8 * cnt);
    return u32x2{sel, 0x03020100u + (4u - cnt) * 0x01010101u};
}

__device__ __forceinline__ unsigned lds_addr(const LDS_AS void* p) { return (unsigned)(size_t)p; }
template <int NW>
__device__ __forceinline__ void compact(const unsigned (&P)[NW], const unsigned (&B8)[NW], unsigned pos_end, unsigned c,
                                        LDS_AS unsigned char* dst, LDS_AS unsigned char* dummy, const LDS_AS unsigned char* lut) {
    unsigned w = 0, w2 = 0, u3 = lds_addr(dst) + pos_end + 3u - 4u * NW;
    u32x2 E[NW];
#pragma unroll
    for (int j = NW - 1; j >= 0; --j) E[j] = *(const LDS_AS u32x2*)(lut + B8[j]);
    LDS_FENCE();
#pragma unroll
    for (int j = NW - 1; j >= 0; --j) {
        w2 = vperm(w2, w, E[j].y);
        w = vperm(w, P[j], E[j].x);
        u3 += E[j].y & 0xffu;
        ((LDS_AS unsigned*)(size_t)(u3 & ~3u))[j] = __builtin_amdgcn_alignbyte(w2, w, ~u3);
    }
    LDS_FENCE();
#pragma unroll
    for (int k = 0; k < 7; ++k)
        ((LDS_AS unsigned char*)(size_t)((unsigned)k < c ? u3 - 3u : lds_addr(dummy)))[k] = (unsigned char)((k < 4 ? w : w2) >> (8 * (k & 3)));
    LDS_FENCE();
}

struct Packed { u32x4 N; u32x2 L1; unsigned L2, L3, L4, hdr; };

__device__ __forceinline__ void encode_part(const u32x4 R0, const u32x4 R1, Packed& o, LDS_AS unsigned char* sc,
                                            const LDS_AS unsigned char* lut, int lane) {
    const unsigned X[8] = {R0.x, R0.y, R0.z, R0.w, R1.x, R1.y, R1.z, R1.w};
    *(LDS_AS u32x4*)(sc + CL_A2 + lane * 16) = u32x4{0, 0, 0, 0};
    *(LDS_AS u32x2*)(sc + CL_A3 + lane * 8) = u32x2{0, 0};
    unsigned s = 0;
#pragma unroll
    for (int j = 0; j < 8; ++j) s = __builtin_amdgcn_udot4(X[j] & 0x78787878u, 0x01010101u, s, false);
    s = __builtin_amdgcn_readlane(wave_incl_scan(s), 63);
    const unsigned mu = s + (16384u - 819u);
    const unsigned b = ((mu >> 14) - 1u) & 15u;
    const bool var = (mu & 16383u) >= 7373u;
    const unsigned K = ((16u - b) & 15u) * 0x08080808u;
    const unsigned M0 = var ? map_word(0, true) : map_word(0, false), M1 = map_word(4, true), M2 = map_word(8, true),
                   M3 = var ? map_word(12, true) : map_word(12, false);
    unsigned P[8], B8[8];
#pragma unroll
    for (int j = 0; j < 8; ++j) {
        const unsigned y = (X[j] & 0x78787878u) + K;
        const unsigned s7 = (y >> 3) & 0x07070707u;
        const unsigned hi = vperm(y << 1, y << 9, 0x0b090a08u);
        const unsigned m = bfi(hi, vperm(M3, M2, s7), vperm(M1, M0, s7));
        B8[j] = __builtin_amdgcn_udot4(m & 0x18181818u, 0x40100401u, 0u, false);
        P[j] = bfi(0x78787878u, X[j], m);
    }
    o.L1.x = ((B8[0] | (B8[1] << 8) | (B8[2] << 16)) >> 3) | (B8[3] << 21);
    o.L1.y = ((B8[4] | (B8[5] << 8) | (B8[6] << 16)) >> 3) | (B8[7] << 21);
    const unsigned c1 = __builtin_popcount(esc_bits(o.L1.x)) + __builtin_popcount(esc_bits(o.L1.y));
    o.N.x = bfi(0x87878787u, X[0], __builtin_amdgcn_alignbit(X[1], X[1], 28));
    o.N.y = bfi(0x87878787u, X[2], __builtin_amdgcn_alignbit(X[3], X[3], 28));
    o.N.z = bfi(0x87878787u, X[4], __builtin_amdgcn_alignbit(X[5], X[5], 28));
    o.N.w = bfi(0x87878787u, X[6], __builtin_amdgcn_alignbit(X[7], X[7], 28));
    const unsigned e1 = wave_incl_scan(c1);
    const unsigned n1 = __builtin_amdgcn_readlane(e1, 63);
    o.L2 = 0; o.L3 = 0; o.L4 = 0; o.hdr = HDR_NOFIT;
    if (n1 > (unsigned)CAP1) return;
    LDS_FENCE();
    compact<8>(P, B8, e1, c1, sc + CL_A2, sc + CL_DUMMY, lut);
    const u32x4 v = *(const LDS_AS u32x4*)(sc + CL_A2 + lane * 16);
    const unsigned Q[4] = {v.x, v.y, v.z, v.w};
    unsigned B2[4];
#pragma unroll
    for (int j = 0; j < 4; ++j) B2[j] = __builtin_amdgcn_udot4(Q[j] & 0x03030303u, 0x40100401u, 0u, false);
    o.L2 = B2[0] | (B2[1] << 8) | (B2[2] << 16) | (B2[3] << 24);
    const unsigned c2 = __builtin_popcount(esc_bits(o.L2));
#pragma unroll
    for (int j = 0; j < 4; ++j) B2[j] <<= 3;
    const unsigned e2 = wave_incl_scan(c2);
    const unsigned n2 = __builtin_amdgcn_readlane(e2, 63);
    if (n2 > (unsigned)CAP2) return;
    compact<4>(Q, B2, e2, c2, sc + CL_A3, sc + CL_DUMMY, lut);
    const u32x2 a3 = *(const LDS_AS u32x2*)(sc + CL_A3 + lane * 8);
    const unsigned Q3[2] = {a3.x, a3.y};
    unsigned B3[2];
#pragma unroll
    for (int j = 0; j < 2; ++j) B3[j] = __builtin_amdgcn_udot4(bfi(0x01010101u, Q3[j] >> 2, Q3[j] >> 6) & 0x03030303u, 0x40100401u, 0u, false);
    o.L3 = B3[0] | (B3[1] << 8);
    const unsigned c3 = __builtin_popcount(esc_bits(o.L3));
    unsigned n3 = 0;
    if (__any(c3 != 0u)) {
        B3[0] <<= 3; B3[1] <<= 3;
        const unsigned e3 = wave_incl_scan(c3);
        n3 = __builtin_amdgcn_readlane(e3, 63);
        if (n3 > (unsigned)CAP3) return;
        compact<2>(Q3, B3, e3, c3, sc + CL_A4, sc + CL_DUMMY, lut);
        const unsigned a4 = *(const LDS_AS unsigned*)(sc + CL_A4 + lane * 4);
        o.L4 = bfi(0x0f0fu, a4 >> 3, a4 >> 15);
    }
    const unsigned h3 = (n2 + 7u) >> 3, h4 = (n3 + 3u) >> 2;
    o.hdr = ((n1 + 15u) >> 4) | ((h3 + h4) << 10) | (h3 << 20) | (b << 27) | (var ? 0x80000000u : 0u);
}
struct TailMap { unsigned o2, o3, o4, total, hdr; };
__device__ __forceinline__ TailMap tail_map(const u32x4 h, unsigned d) {
    const unsigned a0 = h.x & 0x3ffu, a1 = h.y & 0x3ffu, a2 = h.z & 0x3ffu, a3 = h.w & 0x3ffu;
    const unsigned b0 = (h.x >> 10) & 0x3ffu, b1 = (h.y >> 10) & 0x3ffu, b2 = (h.z >> 10) & 0x3ffu, b3 = (h.w >> 10) & 0x3ffu;
    const unsigned hd = d == 0 ? h.x : d == 1 ? h.y : d == 2 ? h.z : h.w;
    TailMap t;
    const unsigned base3 = 16u + 4u * (a0 + a1 + a2 + a3);
    t.o2 = 16u + 4u * ((d > 0 ? a0 : 0u) + (d > 1 ? a1 : 0u) + (d > 2 ? a2 : 0u));
    t.o3 = base3 + 2u * ((d > 0 ? b0 : 0u) + (d > 1 ? b1 : 0u) + (d > 2 ? b2 : 0u));
    t.o4 = t.o3 + 2u * ((hd >> 20) & 0x7fu);
    t.total = base3 + 2u * (b0 + b1 + b2 + b3);
    t.hdr = ((t.o2 >> 2) & 0xffu) | (((t.o3 >> 1) & 0x1ffu) << 8) | (((hd >> 10) & 0x3ffu) != ((hd >> 20) & 0x7fu) ? 0x20000u : 0u) | (hd & 0xfff00000u);
    return t;
}
__device__ __forceinline__ u32x4 uniform4(const u32x4 v) {
    return u32x4{(unsigned)__builtin_amdgcn_readfirstlane(v.x), (unsigned)__builtin_amdgcn_readfirstlane(v.y),
                 (unsigned)__builtin_amdgcn_readfirstlane(v.z), (unsigned)__builtin_amdgcn_readfirstlane(v.w)};
}

__device__ __forceinline__ unsigned lds_window(unsigned q) {
    const LDS_AS unsigned* p = (const LDS_AS unsigned*)(size_t)(q & ~3u);
    return __builtin_amdgcn_alignbyte(p[1], p[0], q);
}
__device__ __forceinline__ unsigned lut_sel(const LDS_AS unsigned char* lut, unsigned w, int byte) {
    return *(const LDS_AS unsigned*)(lut + (((w >> (8 * byte)) & 0xffu) << 2));
}
template <int NP>
__device__ __forceinline__ void decode_exps(const LDS_AS unsigned char* sl, unsigned d0, unsigned (&X)[NP][8], LDS_AS unsigned char* sc0,
                                            const LDS_AS unsigned char* lut, int lane) {
    const LDS_AS unsigned char* const tb = sl + 6144;
    unsigned hd[NP], L2[NP], L3[NP], S1[NP][8], S2[NP][4], S3[NP][2], E0[NP], E1[NP], p1[NP], T1[NP], T2[NP], T3[NP], w3[NP][2];
    u32x2 L1[NP];
    const LDS_AS unsigned char* s3[NP];
#pragma unroll
    for (int k = 0; k < NP; ++k) {
        hd[k] = *(const LDS_AS unsigned*)(tb + (d0 + k) * 4);
        L1[k] = *(const LDS_AS u32x2*)(sl + 4096 + (d0 + k) * 512 + lane * 8);
    }
    LDS_FENCE();
#pragma unroll
    for (int k = 0; k < NP; ++k) {
        hd[k] = __builtin_amdgcn_readfirstlane(hd[k]);
        s3[k] = tb + ((hd[k] >> 7) & 0x3feu) + lane * 2;
        L3[k] = *(const LDS_AS unsigned short*)s3[k];
        L2[k] = *(const LDS_AS unsigned*)(tb + (hd[k] & 0xffu) * 4 + lane * 4);
    }
    LDS_FENCE();
#pragma unroll
    for (int k = 0; k < NP; ++k) {
        const unsigned kb = ((hd[k] >> 27) & 15u) * 0x080808u;
        const bool var = (hd[k] >> 31) != 0u;
        T1[k] = (rel8(0, 1, 2) + kb) & 0x787878u;
        T2[k] = ((var ? rel8(-1, -2, 3) : rel8(-1, -2, -3)) + kb) & 0x787878u;
        T3[k] = ((var ? rel8(-4, -5, -3) : rel8(-4, -5, 3)) + kb) & 0x787878u;
        E0[k] = esc_bits(L1[k].x); E1[k] = esc_bits(L1[k].y);
        const unsigned c1 = __builtin_popcount(E0[k]) + __builtin_popcount(E1[k]);
        p1[k] = wave_incl_scan(c1) - c1;
        p1[k] = (p1[k] > (unsigned)CAP1 ? (unsigned)CAP1 : p1[k]) + lds_addr(sc0 + k * CL_BYTES + CL_A2);
    }
#pragma unroll
    for (int k = 0; k < NP; ++k) {
        S3[k][0] = lut_sel(lut, L3[k], 0); S3[k][1] = lut_sel(lut, L3[k], 1);
#pragma unroll
        for (int j = 0; j < 4; ++j) S2[k][j] = lut_sel(lut, L2[k], j);
    }
    LDS_FENCE();
#pragma unroll
    for (int k = 0; k < NP; ++k) {
        w3[k][0] = 0; w3[k][1] = 0;
        if (hd[k] & 0x20000u) {
            LDS_AS unsigned char* const sc = sc0 + k * CL_BYTES;
            const unsigned l4 = *(const LDS_AS unsigned short*)(s3[k] + ((hd[k] >> 19) & 0xfeu));
            *(LDS_AS unsigned*)(sc + CL_A4 + lane * 4) = ((l4 << 3) & 0x7878u) | ((l4 << 15) & 0x78780000u);
            LDS_FENCE();
            const unsigned E3 = esc_bits(L3[k]);
            const unsigned c3 = __builtin_popcount(E3);
            const unsigned p3 = wave_incl_scan(c3) - c3 + lds_addr(sc + CL_A4);
            w3[k][0] = lds_window(p3);
            w3[k][1] = lds_window(p3 + __builtin_popcount(E3 & 0xffu));
        }
    }
    unsigned E2[NP], p2[NP];
#pragma unroll
    for (int k = 0; k < NP; ++k) {
        E2[k] = esc_bits(L2[k]);
        const unsigned c2 = __builtin_popcount(E2[k]);
        p2[k] = wave_incl_scan(c2) - c2 + lds_addr(sc0 + k * CL_BYTES + CL_A3);
        *(LDS_AS u32x2*)(sc0 + k * CL_BYTES + CL_A3 + lane * 8) = u32x2{vperm(w3[k][0], T3[k], S3[k][0]), vperm(w3[k][1], T3[k], S3[k][1])};
    }
    LDS_FENCE();
    unsigned Q[NP][4];
#pragma unroll
    for (int k = 0; k < NP; ++k)
#pragma unroll
        for (int j = 0; j < 4; ++j) Q[k][j] = lds_window(j ? p2[k] + __builtin_popcount(E2[k] & ((1u << (8 * j)) - 1u)) : p2[k]);
#pragma unroll
    for (int k = 0; k < NP; ++k)
#pragma unroll
        for (int j = 0; j < 8; ++j) S1[k][j] = lut_sel(lut, (j & 4) ? L1[k].y : L1[k].x, j & 3);
    LDS_FENCE();
#pragma unroll
    for (int k = 0; k < NP; ++k)
        *(LDS_AS u32x4*)(sc0 + k * CL_BYTES + CL_A2 + lane * 16) =
            u32x4{vperm(Q[k][0], T2[k], S2[k][0]), vperm(Q[k][1], T2[k], S2[k][1]), vperm(Q[k][2], T2[k], S2[k][2]), vperm(Q[k][3], T2[k], S2[k][3])};
    LDS_FENCE();
#pragma unroll
    for (int k = 0; k < NP; ++k) {
        const unsigned p1b = p1[k] + __builtin_popcount(E0[k]);
#pragma unroll
        for (int j = 0; j < 8; ++j) {
            const unsigned base = (j & 4) ? p1b : p1[k], E = (j & 4) ? E1[k] : E0[k];
            X[k][j] = vperm(lds_window((j & 3) ? base + __builtin_popcount(E & ((1u << (8 * (j & 3))) - 1u)) : base), T1[k], S1[k][j]);
        }
    }
    LDS_FENCE();
}
__device__ __forceinline__ void merge_part(const u32x4 N, const unsigned (&X)[8], u32x4& R0, u32x4& R1) {
    constexpr unsigned K = 0x87878787u;
    R0 = u32x4{(N.x & K) | X[0], (__builtin_amdgcn_alignbit(N.x, N.x, 4) & K) | X[1], (N.y & K) | X[2], (__builtin_amdgcn_alignbit(N.y, N.y, 4) & K) | X[3]};
    R1 = u32x4{(N.z & K) | X[4], (__builtin_amdgcn_alignbit(N.z, N.z, 4) & K) | X[5], (N.w & K) | X[6], (__builtin_amdgcn_alignbit(N.w, N.w, 4) & K) | X[7]};
}

constexpr int NSLOT = 8;
constexpr int CM_LUT = 0;
constexpr int CM_META = 1024;
constexpr int CM_DONE = CM_META + NSLOT * 16;
constexpr int CM_LAND = CM_DONE + NSLOT * 4;
constexpr int CM_NEXT = CM_LAND + 8;
constexpr int CM_TAB = 2048;
constexpr int CM_SLOT = 3072;
constexpr int CM_SC = CM_SLOT + NSLOT * UNIT;
static_assert(CM_SC + 14 * CL_BYTES <= LDS_BYTES && NPULLU >= NSTAGE && NPULLU <= 2 * NSTAGE, "config");

__device__ __forceinline__ bool timed_out(unsigned& spins, unsigned* tmo, unsigned call, unsigned long long t0) {
    if ((++spins & 1023u) != 0u) return false;
    if (__hip_atomic_load(tmo, __ATOMIC_RELAXED, __HIP_MEMORY_SCOPE_AGENT) != call &&
        __builtin_amdgcn_s_memrealtime() - t0 <= DEADLINE_TICKS) return false;
    __hip_atomic_store(tmo, call, __ATOMIC_RELAXED, __HIP_MEMORY_SCOPE_AGENT);
    return true;
}

__global__ void __launch_bounds__(BLOCK, 2) ag_gemm(Params p) {
    extern __shared__ __attribute__((aligned(16))) unsigned char smem[];
    const unsigned long long ts0 = NOW();
    const int lane = threadIdx.x & 63;
    const int wave = __builtin_amdgcn_readfirstlane(threadIdx.x >> 6);
    const int bid = blockIdx.x;
    unsigned char* const peer = p.ws[((bid >> 3) + 1) & 7];
    asm volatile("" ::"s"(p.x), "s"(p.w), "s"(p.sx), "s"(p.sw), "s"(p.out), "s"(p.rank), "s"(peer), "s"(p.ws[0]), "s"(p.ws[1]),
                 "s"(p.ws[2]), "s"(p.ws[3]), "s"(p.ws[4]), "s"(p.ws[5]), "s"(p.ws[6]), "s"(p.ws[7]));
    unsigned char* const myws = p.ws[0];
    unsigned* const cntp = (unsigned*)(myws + CNT_OFF + (size_t)bid * 128);
    unsigned* const tmo = (unsigned*)(myws + TMO_OFF);
    const unsigned char* const xrow = p.x + (size_t)(lane & 15) * KD + (lane >> 4) * 16;
    LDS_AS unsigned char* const lds = (LDS_AS unsigned char*)smem;
    const unsigned long long t0 = __builtin_amdgcn_s_memrealtime();
    unsigned call;
#define LOAD_COUNT() unsigned cv; asm volatile("global_load_dword %0, %1, off sc1" : "=&v"(cv) : "v"(cntp) : "memory")
    unsigned long long ts2 = 0, ts7 = 0, ts8 = 0, ts9 = 0;

    if (__builtin_expect(bid >= NCOMM && bid < NCOMM + NPUSHWG, 1)) {
        LOAD_COUNT();
        const int c_k = bid & 7;
        u32x4 A[PUSH_BLOCKS];
        const unsigned char* s = xrow + c_k * 512 + wave * 64;
        ld16<0>(A[0], s);
        if (PBAR) __builtin_amdgcn_s_barrier();
#pragma unroll
        for (int i = 1; i < PUSH_BLOCKS; ++i) ld16<0>(A[i], s + (size_t)i * (16 * KD));
        unsigned char* dst = peer + X_OFF + (size_t)p.rank * SHARD + (size_t)c_k * UNIT + wave * 1024 + lane * 16;
        asm volatile("s_waitcnt vmcnt(%3)" : "+v"(cv), "+v"(A[0]), "+v"(dst) : "i"(PUSH_BLOCKS - 1) : "memory");
        call = __builtin_amdgcn_readfirstlane(cv) + 1u;
        dst += (size_t)(call & 1u) * NR * SHARD;
        ts7 = NOW();
        st16_sys<0>(dst, A[0]);
#pragma unroll
        for (int i = 1; i < PUSH_BLOCKS; ++i) {
            if (wave < PW2) wait_vm<1>(); else wait_vm<0>();
            asm volatile("" : "+v"(A[i]));
            st16_sys<0>(dst + (size_t)i * (8 * UNIT), A[i]);
        }
        wait_vm<0>();
        ts2 = NOW();
        __builtin_amdgcn_s_barrier();
        if (wave == 0) {
            unsigned char* fp = peer + FLAG_OFF + ((size_t)p.rank * NUNIT + c_k + 8 * (lane >> 2)) * 128 + (lane & 3) * 8;
            asm volatile("s_mov_b64 exec, %2\n\tglobal_store_dword %0, %1, off sc0 sc1\n\ts_mov_b64 exec, -1"
                         ::"v"(fp), "v"(call), "i"((1 << (4 * PUSH_BLOCKS)) - 1) : "memory");
        }
    } else if (__builtin_expect(bid >= NCOMM, 1)) {
        LOAD_COUNT();
        const unsigned d = wave & 3, sn = bid - NCOMM - NPUSHWG;
        const unsigned su0 = NPUSHU + (wave < 4 ? (sn + 2 * NSTAGE - NPULLU) & (NSTAGE - 1) : NSTAGE + sn);
        const bool stage = su0 < NUNIT;
        const unsigned su = stage ? su0 : NUNIT - 1;
        u32x4 R0, R1;
        {
            const unsigned char* s = xrow + (size_t)(su >> 3) * (16 * KD) + (su & 7) * 512 + d * 128;
            ld16<0>(R0, s); ld16<64>(R1, s);
        }
        if (threadIdx.x >= 256) *(LDS_AS u32x2*)(lds + (threadIdx.x - 256) * 8) = lut_entry(threadIdx.x - 256, true);
        unsigned char* fp = myws + FLAG_OFF + (lane & 3) * 8;
#pragma unroll
        for (int k = 1; k < NR; ++k) if (lane == k) fp = p.ws[k] + RDY_OFF;
        fp += ((size_t)p.rank * NUNIT + su) * 128;
        asm volatile("s_waitcnt vmcnt(0) lgkmcnt(0)" : "+v"(cv), "+v"(R0), "+v"(R1), "+v"(fp) :: "memory");
        __builtin_amdgcn_s_barrier();
        call = __builtin_amdgcn_readfirstlane(cv) + 1u;
        ts7 = NOW();
        Packed o;
        o.hdr = HDR_NOFIT;
        unsigned char* const cp = myws + C_OFF + (size_t)(call & 1u) * SHARD + (size_t)su * UNIT;
        if (stage) {
            unsigned char* x = myws + X_OFF + ((size_t)(call & 1u) * NR + p.rank) * SHARD + (size_t)su * UNIT + d * 2048 + lane * 16;
            st16_sys<0>(x, R0); st16_sys<1024>(x, R1);
        }
        if (stage && su < NUNIT - RAW_UNITS) {
            encode_part(R0, R1, o, lds + 2048 + 64 + wave * CL_BYTES, lds, lane);
            st16_sys<0>(cp + d * 1024 + lane * 16, o.N);
            asm volatile("global_store_dwordx2 %0, %1, off sc0 sc1" ::"v"(cp + 4096 + d * 512 + lane * 8), "v"(o.L1) : "memory");
        }
        ts8 = NOW();
        if (lane == 0) *(LDS_AS unsigned*)(lds + 2048 + wave * 4) = o.hdr;
        wait_lgkm();
        __builtin_amdgcn_s_barrier();
        LDS_FENCE();
        const TailMap tm = tail_map(uniform4(*(const LDS_AS u32x4*)(lds + 2048 + (wave >> 2) * 16)), d);
        const bool fits = tm.total <= (unsigned)TAIL_MAX;
        LDS_AS unsigned char* const tb = lds + 2048 + 64 + 8 * CL_BYTES + (wave >> 2) * 1024;
        if (fits) {
            const unsigned h3 = (o.hdr >> 20) & 0x7fu, h4 = ((o.hdr >> 10) & 0x3ffu) - h3;
            if ((unsigned)lane < (o.hdr & 0x3ffu)) *(LDS_AS unsigned*)(tb + tm.o2 + lane * 4) = o.L2;
            if ((unsigned)lane < h3) *(LDS_AS unsigned short*)(tb + tm.o3 + lane * 2) = (unsigned short)o.L3;
            if ((unsigned)lane < h4) *(LDS_AS unsigned short*)(tb + tm.o4 + lane * 2) = (unsigned short)o.L4;
            if (lane == 0) *(LDS_AS unsigned*)(tb + d * 4) = tm.hdr;
        }
        if (lane < 8) fp += (size_t)(call & 1u) * NR * NUNIT * 128;
        u32x2 fv = u32x2{lane >= 8 ? call : (fits ? (tm.total + 15u) >> 4 : INFO_RAW), call};
        asm volatile("s_waitcnt vmcnt(0) lgkmcnt(0)" : "+v"(fp), "+v"(fv) :: "memory");
        __builtin_amdgcn_s_barrier();
        LDS_FENCE();
        if (d == 0 && fits) {
            st16_sys<0>(cp + 6144 + lane * 16, *(const LDS_AS u32x4*)(tb + lane * 16));
            wait_vm<0>();
        }
        ts9 = NOW();
        if (d == 0 && stage)
            asm volatile("s_mov_b64 exec, 0xffe\n\tglobal_store_dwordx2 %0, %1, off sc0 sc1\n\ts_mov_b64 exec, -1" ::"v"(fp), "v"(fv) : "memory");
        __builtin_amdgcn_s_barrier();
    } else {
        const int c_src = bid >> 3, c_k = bid & 7;
        const int src_rank = (p.rank + c_src + 1) & 7;
        if (wave == 1) {
            const unsigned k = lane & 7;
            const unsigned char* tp = k == 0 ? peer + X_OFF + (size_t)p.rank * SHARD : k == 1 ? peer + RDY_OFF
                                    : k == 2 ? peer + C_OFF : k == 3 ? peer + X_OFF + (size_t)src_rank * SHARD + (SHARD - 128)
                                    : k == 4 ? myws + X_OFF + (size_t)src_rank * SHARD : k == 5 ? (const unsigned char*)p.out + (size_t)src_rank * (MSH * NSH * 4)
                                    : k == 6 ? p.w : (const unsigned char*)p.sx;
            if (lane >= 8) tp = k == 2 ? tp + SHARD : (k == 0 || k == 3 || k == 4) ? tp + NR * SHARD : k == 1 ? peer + FLAG_OFF : k == 7 ? (const unsigned char*)p.sw : tp;
            asm volatile("s_mov_b32 m0, %1\n\ts_mov_b64 exec, 0xffff\n\tglobal_load_lds_dword %0, off sc0 sc1\n\ts_mov_b64 exec, -1"
                         ::"v"(tp), "s"(lds_addr(lds + CM_SC)) : "memory");
        }
        if (c_src == 7) return;
        LOAD_COUNT();
        if (threadIdx.x < 256) *(LDS_AS unsigned*)(lds + CM_LUT + threadIdx.x * 4) = lut_entry(threadIdx.x, false).x;
        else *(LDS_AS unsigned*)(lds + CM_TAB + (threadIdx.x - 256) * 4) = 0u;
        if (wave == 0) *(LDS_AS unsigned*)(lds + CM_META + lane * 4) = 0u;
        asm volatile("s_waitcnt vmcnt(0) lgkmcnt(0)" : "+v"(cv) :: "memory");
        __builtin_amdgcn_s_barrier();
        call = __builtin_amdgcn_readfirstlane(cv) + 1u;
        const size_t xo = X_OFF + ((size_t)(call & 1u) * NR + src_rank) * SHARD;
        if (wave == 0) {
            unsigned long long ts3 = 0;
            const unsigned char* const rawp = peer + xo + lane * 16;
            const unsigned char* const cmpp = peer + C_OFF + (size_t)(call & 1u) * SHARD + lane * 16;
            unsigned char* const tick = myws + TICK_OFF + c_src * 128;
            const unsigned tbase = (call - 1u) * (NPULLU - 8) - 16u - NPUSHU;
            unsigned q = 0;
            unsigned n = 16;
#define STEP() do { ++n; wait_vm<WIN - 1>(); if (lane == 0) *(volatile LDS_AS unsigned*)(lds + CM_LAND) = n - (WIN - 1); } while (0)
            auto staged = [&](unsigned u) -> unsigned {
                if (u - NPUSHU >= (unsigned)NPULLU) return INFO_FAIL;
                unsigned info;
                while ((info = __builtin_amdgcn_readfirstlane(*(volatile LDS_AS unsigned*)(lds + CM_TAB + u * 4))) == 0u) {}
                return info;
            };
            auto first_half = [&](unsigned u, unsigned info) -> const unsigned char* {
                const unsigned slot = q & (NSLOT - 1);
                if (lane == 0) *(volatile LDS_AS u32x4*)(lds + CM_META + slot * 16) = u32x4{q + 1u, u | (info & INFO_RAW), n, 0u};
                const unsigned la = lds_addr(lds + CM_SLOT) + slot * UNIT + 4096;
                const unsigned char* g = ((info & INFO_RAW) ? rawp : cmpp) + (size_t)u * UNIT;
                if (info & INFO_RAW) {
                    glds_sys<0>(g + 4096, la); STEP(); glds_sys<1024>(g + 4096, la); STEP();
                    glds_sys<2048>(g + 4096, la); STEP(); glds_sys<3072>(g + 4096, la); STEP();
                } else {
                    glds_sys_mask<2048>(g + 4096, la, info >= 64u ? ~0ull : (1ull << info) - 1ull); STEP();
                    glds_sys<0>(g + 4096, la); STEP(); glds_sys<1024>(g + 4096, la); STEP();
                }
                ++q;
                return g;
            };
            auto second_half = [&](unsigned qq, const unsigned char* g) {
                const unsigned slot = qq & (NSLOT - 1);
                if (lane == 0) *(volatile LDS_AS unsigned*)(lds + CM_META + slot * 16 + 12) = n;
                const unsigned la = lds_addr(lds + CM_SLOT) + slot * UNIT;
                glds_sys<0>(g, la); STEP(); glds_sys<1024>(g, la); STEP(); glds_sys<2048>(g, la); STEP(); glds_sys<3072>(g, la); STEP();
            };
            unsigned u = NPUSHU + c_k;
            unsigned info = staged(u);
            ts3 = NOW();
            if (info != INFO_FAIL) {
                const unsigned char* gcur = first_half(u, info);
                u += 8u;
                while ((info = staged(u)) != INFO_FAIL) {
                    if (q >= NSLOT)
                        while (*(volatile LDS_AS unsigned*)(lds + CM_DONE + (q & (NSLOT - 1)) * 4) != 2u * (q / NSLOT)) __builtin_amdgcn_s_sleep(1);
                    unsigned tv;
                    ticket(tv, tick); ++n;
                    const unsigned char* gnext = first_half(u, info);
                    second_half(q - 2u, gcur);
                    if (WIN > 8) wait_vm<7>();
                    asm volatile("" : "+v"(tv));
                    u = __builtin_amdgcn_readfirstlane(tv) - tbase;
                    gcur = gnext;
                }
                second_half(q - 1u, gcur);
            }
            if (lane == 0) *(volatile LDS_AS unsigned*)(lds + CM_LAND + 4) = q + 1u;
#define DRAIN(k) do { if (!OPENEND && (k) >= WIN - 1) break; wait_vm<k>(); if (lane == 0) *(volatile LDS_AS unsigned*)(lds + CM_LAND) = n - (k); } while (0)
            if (OPENEND || WIN > 8) { DRAIN(10); DRAIN(9); DRAIN(8); DRAIN(7); }
            DRAIN(6); DRAIN(5); DRAIN(4); DRAIN(3); DRAIN(2); DRAIN(1); DRAIN(0);
            STAMP(4); STAMPV(0, ts0); STAMPV(3, ts3); STAMPV(13, q);
            return;
        }
        if (wave == 7) {
            flag_store((unsigned char*)cntp, call);
            const unsigned char* rp = myws + RDY_OFF + ((size_t)(call & 1u) * NR + src_rank) * NUNIT * 128;
            unsigned spins = 0;
            for (int u0 = NPUSHU; u0 < NUNIT; u0 += 64) {
                const int u = u0 + lane < NUNIT ? u0 + lane : NUNIT - 1;
                while (true) {
                    u32x2 r;
                    asm volatile("global_load_dwordx2 %0, %1, off sc0 sc1\n\ts_waitcnt vmcnt(0)" : "=&v"(r) : "v"(rp + (size_t)u * 128) : "memory");
                    const unsigned t = r.y >= call ? r.x : 0u;
                    *(volatile LDS_AS unsigned*)(lds + CM_TAB + u * 4) = t;
                    if (__all(t != 0u)) break;
                    if (timed_out(spins, tmo, call, t0)) {
                        *(volatile LDS_AS u32x4*)(lds + CM_TAB + lane * 16) = u32x4{INFO_FAIL, INFO_FAIL, INFO_FAIL, INFO_FAIL};
                        u0 = NUNIT;
                        break;
                    }
                }
            }
            ts2 = NOW();
        }
        LDS_AS unsigned char* const sc = lds + CM_SC + (wave - 1) * (2 * CL_BYTES);
        unsigned char* const dst = myws + xo + lane * 16;
        unsigned char* const fl = myws + FLAG_OFF + (size_t)src_rank * NUNIT * 128;
        unsigned long long ts5 = 0, ts6 = 0;
        unsigned char* pend = nullptr;
        auto flush = [&]() {
            if (pend) {
                unsigned vm;
                asm volatile("s_getreg_b32 %0, hwreg(HW_REG_IB_STS, 0, 4)" : "=s"(vm));
                if (vm == 0u) { wait_vm<0>(); ts6 = NOW(); flag_store(pend, call); pend = nullptr; }
            } else if (DSLEEP) __builtin_amdgcn_s_sleep(DSLEEP);
        };
        while (true) {
            if (ACKWAIT && pend) { wait_vm<0>(); ts6 = NOW(); flag_store(pend, call);
                pend = nullptr; }
            unsigned rv = 0;
            if (lane == 0) rv = __hip_atomic_fetch_add((LDS_AS unsigned*)(lds + CM_NEXT), 1u, __ATOMIC_RELAXED, __HIP_MEMORY_SCOPE_WORKGROUP);
            const unsigned r = __builtin_amdgcn_readfirstlane(rv);
            const unsigned q = r >> 1, h = r & 1u;
            const unsigned slot = q & (NSLOT - 1);
            const LDS_AS unsigned char* const sl = lds + CM_SLOT + slot * UNIT;
            unsigned uw;
            bool end = false;
            while (true) {
                const u32x4 m = uniform4(*(volatile LDS_AS u32x4*)(lds + CM_META + slot * 16));
                const u32x2 lv = *(volatile LDS_AS u32x2*)(lds + CM_LAND);
                const unsigned land = __builtin_amdgcn_readfirstlane(lv.x), e = __builtin_amdgcn_readfirstlane(lv.y);
                uw = m.y;
                if (m.x == q + 1u) {
                    if (land >= m.z + ((uw & INFO_RAW) ? 4u * h : 2u + h)) break;
                } else if (e - 1u <= q) { end = true; break; }
                flush();
            }
            if (end) break;
            LDS_FENCE();
            const bool raw = (uw & INFO_RAW) != 0u;
            unsigned X[2][8];
            if (!raw) {
                decode_exps<2>(sl, 2u * h, X, sc, lds + CM_LUT, lane);
#pragma unroll
                for (int k = 0; k < 2; ++k)
                    asm volatile("" : "+v"(X[k][0]), "+v"(X[k][1]), "+v"(X[k][2]), "+v"(X[k][3]), "+v"(X[k][4]), "+v"(X[k][5]), "+v"(X[k][6]), "+v"(X[k][7]));
            }
            ts7 = NOW();
            if (!raw || h == 0u) {
                while (true) {
                    const unsigned ib = __builtin_amdgcn_readfirstlane(*(volatile LDS_AS unsigned*)(lds + CM_META + slot * 16 + 12));
                    const unsigned land = __builtin_amdgcn_readfirstlane(*(volatile LDS_AS unsigned*)(lds + CM_LAND));
                    if (ib != 0u && land >= ib + (raw ? 4u : 2u * h + 2u)) break;
                    if (pend) flush();
                }
                LDS_FENCE();
            }
            u32x4 R[4];
            if (raw) {
#pragma unroll
                for (int i = 0; i < 4; ++i) R[i] = *(const LDS_AS u32x4*)(sl + h * 4096 + i * 1024 + lane * 16);
            } else {
                const u32x4 N0 = *(const LDS_AS u32x4*)(sl + h * 2048 + lane * 16), N1 = *(const LDS_AS u32x4*)(sl + h * 2048 + 1024 + lane * 16);
                merge_part(N0, X[0], R[0], R[1]);
                merge_part(N1, X[1], R[2], R[3]);
            }
            ts5 = NOW();
            const unsigned u = uw & (NUNIT - 1);
            unsigned char* const o = dst + (size_t)u * UNIT + h * 4096;
            st16_sys<0>(o, R[0]); st16_sys<1024>(o, R[1]); st16_sys<2048>(o, R[2]); st16_sys<3072>(o, R[3]);
            if (lane == 0) __hip_atomic_fetch_add((LDS_AS unsigned*)(lds + CM_DONE + slot * 4), 1u, __ATOMIC_RELAXED, __HIP_MEMORY_SCOPE_WORKGROUP);
            if (pend) { wait_vm<4>(); flag_store(pend, call); }
            pend = fl + (size_t)u * 128 + h * 8;
        }
        if (pend) { wait_vm<0>(); ts6 = NOW(); flag_store(pend, call); }
        STAMPV(0, ts0); STAMPV(5, ts5); STAMPV(6, ts6); STAMPV(2, ts2); STAMPV(7, ts7);
        return;
    }

    const int cn = bid - NCOMM;
    const int cg = (cn >> 3) & 7;
    const int src_rank = (p.rank + cn + 3 * (cn >> 6)) & 7;
    if (wave == 7) flag_store((unsigned char*)cntp, call);

    i32x8 Wf[4][4];
    LDS_AS unsigned char* const wl = lds + wave * 16384;
    {
        const unsigned char* wsrc = p.w + (size_t)(wave * 512 + (lane >> 2)) * NSH + cg * 64 + (lane & 3) * 16;
        const int g = lane >> 4, q = (lane & 15) >> 1, h = lane & 1;
        LDS_AS unsigned char* const rd = wl + (16 * g + q) * 64 + h * 8;
#pragma unroll
        for (int half = 0; half < 2; ++half) {
#pragma unroll
            for (int i = 0; i < 16; ++i)
                __builtin_amdgcn_global_load_lds((const GLB_AS unsigned*)(wsrc + (size_t)(half * 256 + i * 16) * NSH),
                                                 (LDS_AS unsigned*)(wl + i * 1024), 16, 0, 0);
            wait_vm<0>();
#pragma unroll
            for (int s2 = 0; s2 < 2; ++s2)
#pragma unroll
                for (int t = 0; t < 4; ++t)
#pragma unroll
                    for (int pc = 0; pc < 4; ++pc) {
                        i32x2 v = __builtin_amdgcn_ds_read_tr8_b64_v2i32(
                            (LDS_AS i32x2*)(rd + (s2 * 128 + (pc >> 1) * 64 + (pc & 1) * 8) * 64 + t * 16));
                        Wf[half * 2 + s2][t][pc * 2] = v.x;
                        Wf[half * 2 + s2][t][pc * 2 + 1] = v.y;
                    }
            wait_lgkm();
        }
    }
    const float scale = p.sx[0] * p.sw[0];
    const unsigned char* const xl = myws + X_OFF + ((size_t)(call & 1u) * NR + src_rank) * SHARD + wave * UNIT + 4096 + lane * 16;
    const unsigned* const flags = (const unsigned*)(myws + FLAG_OFF + ((size_t)src_rank * NUNIT + wave) * 128 + (lane & 1) * 8);
    LDS_AS unsigned char* const pw = wl + (lane & 15) * PART_STRIDE + (lane >> 4) * 16;
    LDS_AS unsigned char* const pr = lds + (wave * 4 + (lane >> 4)) * PART_STRIDE + (lane & 15) * 16;
    float* const outp = p.out + (size_t)(src_rank * MSH + wave * 4 + (lane >> 4)) * NSH + cg * 64 + (lane & 15) * 4;
    int buf = 0;
    bool bad = false;
    const unsigned long long ts1 = NOW();
    unsigned long long ts3 = 0, ts4 = 0;
    for (int b = 2 - (cn >> 6); b < NBLK; b += 3) {
        u32x4 R[8];
        {
            const unsigned* f = flags + (size_t)b * (8 * 32);
            unsigned spins = 0;
            while (!bad && !__all(__hip_atomic_load(f, __ATOMIC_RELAXED, __HIP_MEMORY_SCOPE_SYSTEM) >= call)) {
                bad = timed_out(spins, tmo, call, t0);
                if (CSLEEP) __builtin_amdgcn_s_sleep(CSLEEP);
            }
            ts3 = NOW();
            const unsigned char* s = xl + (size_t)b * (8 * UNIT);
            asm volatile(
                "global_load_dwordx4 %0, %8, off offset:-4096 sc0 sc1\n\tglobal_load_dwordx4 %1, %8, off offset:-3072 sc0 sc1\n\t"
                "global_load_dwordx4 %2, %8, off offset:-2048 sc0 sc1\n\tglobal_load_dwordx4 %3, %8, off offset:-1024 sc0 sc1\n\t"
                "global_load_dwordx4 %4, %8, off sc0 sc1\n\tglobal_load_dwordx4 %5, %8, off offset:1024 sc0 sc1\n\t"
                "global_load_dwordx4 %6, %8, off offset:2048 sc0 sc1\n\tglobal_load_dwordx4 %7, %8, off offset:3072 sc0 sc1\n\t"
                "s_waitcnt vmcnt(0)"
                : "=&v"(R[0]), "=&v"(R[1]), "=&v"(R[2]), "=&v"(R[3]), "=&v"(R[4]), "=&v"(R[5]), "=&v"(R[6]), "=&v"(R[7])
                : "v"(s) : "memory");
        }
        ts4 = NOW();
        f32x4 acc[4] = {};
#pragma unroll
        for (int s = 0; s < 4; ++s) {
            i32x8 xf;
            xf[0] = R[2 * s].x; xf[1] = R[2 * s].y; xf[2] = R[2 * s].z; xf[3] = R[2 * s].w;
            xf[4] = R[2 * s + 1].x; xf[5] = R[2 * s + 1].y; xf[6] = R[2 * s + 1].z; xf[7] = R[2 * s + 1].w;
#pragma unroll
            for (int t = 0; t < 4; ++t)
                acc[t] = __builtin_amdgcn_mfma_scale_f32_16x16x128_f8f6f4(Wf[s][t], xf, acc[t], 1, 0, 0, 0, 0, 0);
        }
        if (bad) {
            const float nan = __builtin_nanf("");
#pragma unroll
            for (int t = 0; t < 4; ++t) acc[t] = f32x4{nan, nan, nan, nan};
        }
#pragma unroll
        for (int t = 0; t < 4; ++t) *(LDS_AS f32x4*)(pw + buf * PART_BYTES + t * 64) = acc[t];
        wait_lgkm();
        __builtin_amdgcn_s_barrier();
        asm volatile("" ::: "memory");
        if (wave < 4) {
            f32x4 sum = *(LDS_AS f32x4*)(pr + buf * PART_BYTES);
#pragma unroll
            for (int w2 = 1; w2 < 8; ++w2) sum += *(LDS_AS f32x4*)(pr + buf * PART_BYTES + w2 * 16384);
            sum *= scale;
            st16_sys<0>((unsigned char*)(outp + (size_t)b * (16 * NSH)), __builtin_bit_cast(u32x4, sum));
        }
        buf ^= 1;
    }
}

extern "C" void kernel_launch(int rank, int nranks, ncclComm_t comm, hipStream_t stream,
                              void* const* d_in, const int* in_sizes, int n_in,
                              void* d_out, int out_size,
                              const int* in_dtypes, int out_dtype,
                              void* const* d_out_all,
                              void* const* d_ws_all, size_t ws_bytes) {
    (void)comm; (void)in_dtypes; (void)out_dtype; (void)d_out_all;
    if (nranks != NR || (unsigned)rank >= (unsigned)NR || n_in != 4 || ws_bytes < WS_NEED ||
        in_sizes[0] != (int)SHARD || in_sizes[1] != KD * NSH || out_size != NR * MSH * NSH) {
        fprintf(stderr, "unexpected problem shape\n");
        return;
    }
    static bool init = false;
    if (!init) {
        HIP_CHECK(hipFuncSetAttribute((const void*)ag_gemm, hipFuncAttributeMaxDynamicSharedMemorySize, LDS_BYTES));
        init = true;
    }
    Params p;
    p.x = (const unsigned char*)d_in[0];
    p.w = (const unsigned char*)d_in[1];
    p.sx = (const float*)d_in[2];
    p.sw = (const float*)d_in[3];
    p.out = (float*)d_out;
    for (int d = 0; d < NR; ++d) p.ws[d] = (unsigned char*)d_ws_all[(rank + d) % NR];
    p.rank = rank;
    p.pad = 0;
    hipLaunchKernelGGL(ag_gemm, dim3(GRID), dim3(BLOCK), LDS_BYTES, stream, p);
    HIP_CHECK(hipGetLastError());
}
```

```cpp
#include <hip/hip_runtime.h>
#include <rccl/rccl.h>
#include <cstddef>
#include <cstdio>

#define HIP_CHECK(expr) do { hipError_t _e = (expr); if (_e != hipSuccess) { \
    fprintf(stderr, "HIP error %s at %s:%d\n", hipGetErrorString(_e), __FILE__, __LINE__); \
    return; } } while (0)

typedef unsigned u32x2 __attribute__((ext_vector_type(2)));
typedef unsigned u32x4 __attribute__((ext_vector_type(4)));
typedef int i32x2 __attribute__((ext_vector_type(2)));
typedef int i32x8 __attribute__((ext_vector_type(8)));
typedef float f32x4 __attribute__((ext_vector_type(4)));
#define LDS_AS __attribute__((address_space(3)))
#define GLB_AS __attribute__((address_space(1)))

constexpr int NR = 8;
constexpr int MSH = 512, KD = 4096, NSH = 512;
constexpr size_t SHARD = (size_t)MSH * KD;
constexpr int NBLK = MSH / 16;
constexpr int NUNIT = NBLK * 8;
constexpr int UNIT = 8192;
constexpr int GRID = 256, BLOCK = 512;
constexpr int NCOMM = 64, NPUSHWG = 64, NSTAGE = GRID - NCOMM - NPUSHWG;
constexpr int PUSH_BLOCKS = 3, PUSH_LAST = 4;
constexpr int NPUSHU = (PUSH_BLOCKS - 1) * 8 + PUSH_LAST;
constexpr int NPULLU = NUNIT - NPUSHU;
constexpr int PUSH_DEEP = 2;
constexpr int RAW_UNITS = 8;
constexpr int WIN = 8;
constexpr size_t X_OFF = 0;
constexpr size_t C_OFF = X_OFF + 2 * NR * SHARD;
constexpr size_t FLAG_OFF = C_OFF + 2 * SHARD;
constexpr size_t RDY_OFF = FLAG_OFF + (size_t)NR * NUNIT * 128;
constexpr size_t CNT_OFF = RDY_OFF + (size_t)2 * NR * NUNIT * 128;
constexpr size_t TICK_OFF = CNT_OFF + (size_t)GRID * 128;
constexpr size_t TMO_OFF = TICK_OFF + NR * 128;
constexpr size_t WS_NEED = TMO_OFF + 128;
constexpr unsigned long long DEADLINE_TICKS = 50000000ull;
constexpr int LDS_BYTES = 8 * 16384;
constexpr int PART_STRIDE = 272, PART_BYTES = 16 * PART_STRIDE;

struct Params {
    const unsigned char* x;
    const unsigned char* w;
    const float* sx;
    const float* sw;
    float* out;
    unsigned char* ws[NR];
    int rank;
    int pad;
};

#define LDS_FENCE() asm volatile("" ::: "memory")
template <int N> __device__ __forceinline__ void wait_vm() { asm volatile("s_waitcnt vmcnt(%0)" ::"i"(N) : "memory"); }
__device__ __forceinline__ void wait_lgkm() { asm volatile("s_waitcnt lgkmcnt(0)" ::: "memory"); }
__device__ __forceinline__ unsigned lds_addr(const LDS_AS void* p) { return (unsigned)(size_t)p; }

template <int OFF> __device__ __forceinline__ void ld16(u32x4& v, const unsigned char* p) {
    asm volatile("global_load_dwordx4 %0, %1, off offset:%2" : "=v"(v) : "v"(p), "i"(OFF) : "memory");
}
template <int OFF> __device__ __forceinline__ void st16_sys(unsigned char* p, u32x4 v) {
    asm volatile("global_store_dwordx4 %0, %1, off offset:%2 sc0 sc1\n\ts_nop 1" ::"v"(p), "v"(v), "i"(OFF) : "memory");
}
__device__ __forceinline__ void flag_store(unsigned char* p, unsigned v) {
    asm volatile("s_mov_b64 exec, 1\n\tglobal_store_dword %0, %1, off sc0 sc1\n\ts_mov_b64 exec, -1" ::"v"(p), "v"(v) : "memory");
}
__device__ __forceinline__ void ticket(unsigned& v, unsigned char* p) {
    asm volatile("s_mov_b64 exec, 1\n\tglobal_atomic_add %0, %1, %2, off sc0\n\ts_mov_b64 exec, -1" : "=&v"(v) : "v"(p), "v"(1u) : "memory");
}
__device__ __forceinline__ void lds_inc(LDS_AS unsigned char* p) {
    asm volatile("s_mov_b64 exec, 1\n\tds_add_u32 %0, %1\n\ts_mov_b64 exec, -1" ::"v"(lds_addr(p)), "v"(1u) : "memory");
}
__device__ __forceinline__ unsigned lds_ticket(LDS_AS unsigned char* p) {
    unsigned v;
    asm volatile("s_mov_b64 exec, 1\n\tds_add_rtn_u32 %0, %1, %2\n\ts_mov_b64 exec, -1\n\ts_waitcnt lgkmcnt(0)"
                 : "=&v"(v) : "v"(lds_addr(p)), "v"(1u) : "memory");
    return __builtin_amdgcn_readfirstlane(v);
}
template <int OFF> __device__ __forceinline__ void glds_sys(const unsigned char* g, unsigned lds) {
    asm volatile("s_mov_b32 m0, %1\n\ts_nop 0\n\tglobal_load_lds_dwordx4 %0, off offset:%2 sc0 sc1" ::"v"(g), "s"(lds), "i"(OFF) : "memory");
}
template <int OFF> __device__ __forceinline__ void glds_sys_mask(const unsigned char* g, unsigned lds, unsigned long long mask) {
    asm volatile("s_mov_b32 m0, %1\n\ts_mov_b64 exec, %3\n\tglobal_load_lds_dwordx4 %0, off offset:%2 sc0 sc1\n\ts_mov_b64 exec, -1"
                 ::"v"(g), "s"(lds), "i"(OFF), "s"(mask) : "memory");
}

constexpr int CAP1 = 1024, CAP2 = 512, CAP3 = 256, TAIL_MAX = 1024;
constexpr int CL_A2 = 0;
constexpr int CL_A3 = CAP1 + 16;
constexpr int CL_A4 = CL_A3 + CAP1 + 16;
constexpr int CL_BYTES = CL_A4 + CAP2 + 32;
constexpr unsigned INFO_RAW = 0x80000000u, INFO_FAIL = 0xffffffffu;
constexpr unsigned rel8(int a, int b, int c) { return (unsigned)(((a & 15) << 3) | ((b & 15) << 11) | ((c & 15) << 19)); }
constexpr unsigned map_byte(int r, bool var) {
    return r <= 2 ? (unsigned)r << 3 : r == 15 ? 0x18u : r == 14 ? 0x19u : r == (var ? 3 : 13) ? 0x1au
         : r == 12 ? 0x1bu : r == 11 ? 0x1fu : r == (var ? 13 : 3) ? 0x9bu : 0x9fu;
}
constexpr unsigned map_word(int r, bool var) {
    return map_byte(r, var) | (map_byte(r + 1, var) << 8) | (map_byte(r + 2, var) << 16) | (map_byte(r + 3, var) << 24);
}

__device__ __forceinline__ unsigned vperm(unsigned hi, unsigned lo, unsigned sel) { return __builtin_amdgcn_perm(hi, lo, sel); }
__device__ __forceinline__ unsigned bfi(unsigned m, unsigned a, unsigned b) { return (a & m) | (b & ~m); }
__device__ __forceinline__ unsigned wave_incl_scan(unsigned v) {
    v += __builtin_amdgcn_update_dpp(0u, v, 0x111, 0xf, 0xf, false);
    v += __builtin_amdgcn_update_dpp(0u, v, 0x112, 0xf, 0xf, false);
    v += __builtin_amdgcn_update_dpp(0u, v, 0x114, 0xf, 0xf, false);
    v += __builtin_amdgcn_update_dpp(0u, v, 0x118, 0xf, 0xf, false);
    v += __builtin_amdgcn_update_dpp(0u, v, 0x142, 0xa, 0xf, false);
    v += __builtin_amdgcn_update_dpp(0u, v, 0x143, 0xc, 0xf, false);
    return v;
}
__device__ __forceinline__ unsigned esc_bits(unsigned l) { return l & (l >> 1) & 0x55555555u; }

__device__ __forceinline__ u32x2 lut_entry(unsigned b, bool enc) {
    unsigned sel = 0, cnt = 0;
#pragma unroll
    for (int k = 0; k < 4; ++k) {
        const unsigned c = (b >> (2 * k)) & 3u;
        if (enc) { if (c == 3u) { sel |= (unsigned)k << (8 * cnt); ++cnt; } }
        else { sel |= (c == 3u ? 4u + cnt : c) << (8 * k); cnt += (c == 3u); }
    }
    if (enc && cnt < 4u) sel |= 0x07060504u << (8 * cnt);
    return u32x2{sel, 0x03020100u + (4u - cnt) * 0x01010101u};
}

template <int OFF> __device__ __forceinline__ void lds_or(unsigned a, unsigned v) {
    asm volatile("ds_or_b32 %0, %1 offset:%2" ::"v"(a), "v"(v), "i"(OFF) : "memory");
}
template <int NW, int J> __device__ __forceinline__ void compact_step(const unsigned (&P)[NW], const u32x2 (&E)[NW], unsigned& w, unsigned& w2, unsigned& u3) {
    w2 = vperm(w2, w, E[J].y);
    w = vperm(w, P[J], E[J].x);
    u3 += E[J].y & 0xffu;
    lds_or<4 * J>(u3 & ~3u, __builtin_amdgcn_alignbyte(w2, w, ~u3));
    if constexpr (J > 0) compact_step<NW, J - 1>(P, E, w, w2, u3);
}
template <int NW>
__device__ __forceinline__ void compact(const unsigned (&P)[NW], const u32x2 (&E)[NW], unsigned pos_end, LDS_AS unsigned char* dst) {
    unsigned w = 0, w2 = 0, u3 = lds_addr(dst) + pos_end + 3u - 4u * NW;
    compact_step<NW, NW - 1>(P, E, w, w2, u3);
    u3 -= 3u;
    lds_or<0>(u3 & ~3u, w << (8u * (u3 & 3u)));
}
template <int NW> __device__ __forceinline__ void lut_read(u32x2 (&E)[NW], const unsigned (&B8)[NW], const LDS_AS unsigned char* lut) {
#pragma unroll
    for (int j = NW - 1; j >= 0; --j) E[j] = *(const LDS_AS u32x2*)(lut + B8[j]);
    LDS_FENCE();
}

struct Packed { u32x4 N; u32x2 L1; };
__device__ __forceinline__ void tail_alloc(unsigned& v, LDS_AS unsigned* top, unsigned bytes) {
    asm volatile("s_mov_b64 exec, 1\n\tds_add_rtn_u32 %0, %1, %2\n\ts_mov_b64 exec, -1" : "=&v"(v) : "v"(lds_addr(top)), "v"(bytes) : "memory");
}

__device__ __forceinline__ void encode_clear(LDS_AS unsigned char* sc, int lane) {
    *(LDS_AS u32x4*)(sc + CL_A2 + lane * 16) = u32x4{0, 0, 0, 0};
    *(LDS_AS u32x2*)(sc + CL_A3 + lane * 8) = u32x2{0, 0};
    *(LDS_AS unsigned*)(sc + CL_A4 + lane * 4) = 0u;
}
template <class F>
__device__ __forceinline__ bool encode_part(const u32x4 R0, const u32x4 R1, unsigned d, LDS_AS unsigned char* sc, LDS_AS unsigned char* tb,
                                            LDS_AS unsigned* top, const LDS_AS unsigned char* lut, int lane, F planes) {
    Packed o;
    const unsigned X[8] = {R0.x, R0.y, R0.z, R0.w, R1.x, R1.y, R1.z, R1.w};
    unsigned s = 0;
#pragma unroll
    for (int j = 0; j < 8; ++j) s = __builtin_amdgcn_udot4(X[j] & 0x78787878u, 0x01010101u, s, false);
    s = __builtin_amdgcn_readlane(wave_incl_scan(s), 63);
    const unsigned mu = s + (16384u - 819u);
    const unsigned b = ((mu >> 14) - 1u) & 15u;
    const bool var = (mu & 16383u) >= 7373u;
    const unsigned K = ((16u - b) & 15u) * 0x08080808u;
    const unsigned M0 = var ? map_word(0, true) : map_word(0, false), M1 = map_word(4, true), M2 = map_word(8, true),
                   M3 = var ? map_word(12, true) : map_word(12, false);
    unsigned P[8], B8[8];
#pragma unroll
    for (int j = 0; j < 8; ++j) {
        const unsigned y = (X[j] & 0x78787878u) + K;
        const unsigned s7 = (y >> 3) & 0x07070707u;
        const unsigned hi = vperm(y << 1, y << 9, 0x0b090a08u);
        unsigned m = bfi(hi, vperm(M3, M2, s7), vperm(M1, M0, s7));
        asm("" : "+v"(m));
        B8[j] = __builtin_amdgcn_udot4(m & 0x18181818u, 0x40100401u, 0u, false);
        P[j] = bfi(0x78787878u, X[j], m);
    }
    u32x2 E1[8];
    lut_read<8>(E1, B8, lut);
    o.L1.x = ((B8[0] | (B8[1] << 8) | (B8[2] << 16)) >> 3) | (B8[3] << 21);
    o.L1.y = ((B8[4] | (B8[5] << 8) | (B8[6] << 16)) >> 3) | (B8[7] << 21);
    const unsigned c1 = __builtin_popcount(esc_bits(o.L1.x)) + __builtin_popcount(esc_bits(o.L1.y));
    const unsigned e1 = wave_incl_scan(c1);
    const unsigned n1 = __builtin_amdgcn_readlane(e1, 63);
    if (n1 > (unsigned)CAP1) return false;
    if (c1) compact<8>(P, E1, e1, sc + CL_A2);
    LDS_FENCE();
    const u32x4 v = *(const LDS_AS u32x4*)(sc + CL_A2 + lane * 16);
    LDS_FENCE();
    o.N.x = bfi(0x87878787u, X[0], __builtin_amdgcn_alignbit(X[1], X[1], 28));
    o.N.y = bfi(0x87878787u, X[2], __builtin_amdgcn_alignbit(X[3], X[3], 28));
    o.N.z = bfi(0x87878787u, X[4], __builtin_amdgcn_alignbit(X[5], X[5], 28));
    o.N.w = bfi(0x87878787u, X[6], __builtin_amdgcn_alignbit(X[7], X[7], 28));
    planes(o);
    const unsigned Q[4] = {v.x, v.y, v.z, v.w};
    unsigned B2[4];
#pragma unroll
    for (int j = 0; j < 4; ++j) B2[j] = __builtin_amdgcn_udot4(Q[j] & 0x03030303u, 0x40100401u, 0u, false);
    const unsigned L2 = B2[0] | (B2[1] << 8) | (B2[2] << 16) | (B2[3] << 24);
#pragma unroll
    for (int j = 0; j < 4; ++j) B2[j] <<= 3;
    u32x2 E2[4];
    lut_read<4>(E2, B2, lut);
    const unsigned c2 = __builtin_popcount(esc_bits(L2));
    const unsigned e2 = wave_incl_scan(c2);
    const unsigned n2 = __builtin_amdgcn_readlane(e2, 63);
    if (n2 > (unsigned)CAP2) return false;
    const unsigned h2 = (n1 + 7u) >> 3, h3 = (n2 + 7u) >> 3;
    const unsigned s23 = (2u * (h2 + h3) + 3u) & ~3u;
    unsigned av;
    tail_alloc(av, top, s23);
    if (c2) compact<4>(Q, E2, e2, sc + CL_A3);
    LDS_FENCE();
    u32x2 a3 = *(const LDS_AS u32x2*)(sc + CL_A3 + lane * 8);
    asm volatile("s_waitcnt lgkmcnt(0)" : "+v"(av), "+v"(a3) :: "memory");
    const unsigned o2 = __builtin_amdgcn_readfirstlane(av);
    if (o2 + s23 > (unsigned)TAIL_MAX) return true;
    if ((unsigned)lane < (h2 + 1u) >> 1) *(LDS_AS unsigned*)(tb + o2 + lane * 4) = L2;
    LDS_FENCE();
    const unsigned Q3[2] = {a3.x, a3.y};
    unsigned B3[2];
#pragma unroll
    for (int j = 0; j < 2; ++j) B3[j] = __builtin_amdgcn_udot4(bfi(0x01010101u, Q3[j] >> 2, Q3[j] >> 6) & 0x03030303u, 0x40100401u, 0u, false);
    const unsigned L3 = B3[0] | (B3[1] << 8);
    if ((unsigned)lane < h3) *(LDS_AS unsigned short*)(tb + o2 + 2u * h2 + lane * 2) = (unsigned short)L3;
    const unsigned c3 = __builtin_popcount(esc_bits(L3));
    unsigned o4 = 0;
    if (__any(c3 != 0u)) {
        B3[0] <<= 3; B3[1] <<= 3;
        u32x2 E3[2];
        lut_read<2>(E3, B3, lut);
        const unsigned e3 = wave_incl_scan(c3);
        const unsigned n3 = __builtin_amdgcn_readlane(e3, 63);
        if (n3 > (unsigned)CAP3) return false;
        const unsigned s4 = ((n3 + 7u) >> 3) << 2;
        unsigned a4v;
        tail_alloc(a4v, top, s4);
        if (c3) compact<2>(Q3, E3, e3, sc + CL_A4);
        LDS_FENCE();
        unsigned a4 = *(const LDS_AS unsigned*)(sc + CL_A4 + lane * 4);
        asm volatile("s_waitcnt lgkmcnt(0)" : "+v"(a4v), "+v"(a4) :: "memory");
        o4 = __builtin_amdgcn_readfirstlane(a4v);
        if (o4 + s4 > (unsigned)TAIL_MAX) return true;
        if ((unsigned)lane < s4 >> 1) *(LDS_AS unsigned short*)(tb + o4 + lane * 2) = (unsigned short)bfi(0x0f0fu, a4 >> 3, a4 >> 15);
    }
    if (lane == 0) *(LDS_AS unsigned*)(tb + d * 4) = ((o2 >> 2) & 0xffu) | ((((o2 >> 1) + h2) & 0x1ffu) << 8) | (o4 << 15) | (b << 27) | (var ? 0x80000000u : 0u);
    return true;
}
__device__ __forceinline__ u32x4 uniform4(const u32x4 v) {
    return u32x4{(unsigned)__builtin_amdgcn_readfirstlane(v.x), (unsigned)__builtin_amdgcn_readfirstlane(v.y),
                 (unsigned)__builtin_amdgcn_readfirstlane(v.z), (unsigned)__builtin_amdgcn_readfirstlane(v.w)};
}

__device__ __forceinline__ u32x2 win_read(unsigned q) {
    const LDS_AS unsigned* p = (const LDS_AS unsigned*)(size_t)(q & ~3u);
    return u32x2{p[0], p[1]};
}
__device__ __forceinline__ unsigned win_get(const u32x2 w, unsigned q) { return __builtin_amdgcn_alignbyte(w.y, w.x, q); }
__device__ __forceinline__ unsigned lut_sel(unsigned w, int byte) { return *(const LDS_AS unsigned*)(size_t)(((w >> (8 * byte)) & 0xffu) << 2); }
#define PHASE() do { LDS_FENCE(); __builtin_amdgcn_sched_barrier(0); } while (0)
__device__ __forceinline__ void decode_exps(const LDS_AS unsigned char* sl, unsigned d0, unsigned (&X)[2][8], LDS_AS unsigned char* sc0, int lane) {
    const LDS_AS unsigned char* const tb = sl + 6144;
    unsigned hd[2], L2[2], L3[2], S1[2][8], S2[2][4], S3[2][2], E0[2], E1[2], E2[2], T1[2], T2[2], T3[2], w3[2][2];
    u32x2 L1[2];
    const u32x2 hv = *(const LDS_AS u32x2*)(tb + d0 * 4);
#pragma unroll
    for (int k = 0; k < 2; ++k) L1[k] = *(const LDS_AS u32x2*)(sl + 4096 + (d0 + k) * 512 + lane * 8);
    PHASE();
    hd[0] = __builtin_amdgcn_readfirstlane(hv.x); hd[1] = __builtin_amdgcn_readfirstlane(hv.y);
#pragma unroll
    for (int k = 0; k < 2; ++k) {
        L3[k] = *(const LDS_AS unsigned short*)(tb + ((hd[k] >> 7) & 0x3feu) + lane * 2);
        L2[k] = *(const LDS_AS unsigned*)(tb + (hd[k] & 0xffu) * 4 + lane * 4);
    }
    PHASE();
    unsigned c1 = 0;
#pragma unroll
    for (int k = 0; k < 2; ++k) {
        const unsigned kb = ((hd[k] >> 27) & 15u) * 0x080808u;
        const bool var = (hd[k] >> 31) != 0u;
        T1[k] = (rel8(0, 1, 2) + kb) & 0x787878u;
        T2[k] = ((var ? rel8(-1, -2, 3) : rel8(-1, -2, -3)) + kb) & 0x787878u;
        T3[k] = ((var ? rel8(-4, -5, -3) : rel8(-4, -5, 3)) + kb) & 0x787878u;
        E0[k] = esc_bits(L1[k].x); E1[k] = esc_bits(L1[k].y);
        c1 |= (__builtin_popcount(E0[k]) + __builtin_popcount(E1[k])) << (16 * k);
    }
    const unsigned x1 = wave_incl_scan(c1) - c1;
    PHASE();
#pragma unroll
    for (int k = 0; k < 2; ++k) { S3[k][0] = lut_sel(L3[k], 0); S3[k][1] = lut_sel(L3[k], 1); }
#pragma unroll
    for (int k = 0; k < 2; ++k)
#pragma unroll
        for (int j = 0; j < 4; ++j) S2[k][j] = lut_sel(L2[k], j);
    PHASE();
    E2[0] = esc_bits(L2[0]); E2[1] = esc_bits(L2[1]);
    const unsigned c2 = __builtin_popcount(E2[0]) | (__builtin_popcount(E2[1]) << 16);
    const unsigned x2 = wave_incl_scan(c2) - c2;
#pragma unroll
    for (int k = 0; k < 2; ++k) {
        w3[k][0] = 0; w3[k][1] = 0;
        if (hd[k] & 0x1fe0000u) {
            LDS_AS unsigned char* const sc = sc0 + k * CL_BYTES;
            const unsigned l4 = *(const LDS_AS unsigned short*)(tb + ((hd[k] >> 15) & 0x3fcu) + lane * 2);
            const unsigned E3 = esc_bits(L3[k]);
            const unsigned c3 = __builtin_popcount(E3);
            const unsigned p3 = wave_incl_scan(c3) - c3 + lds_addr(sc + CL_A4), p3b = p3 + __builtin_popcount(E3 & 0xffu);
            *(LDS_AS unsigned*)(sc + CL_A4 + lane * 4) = ((l4 << 3) & 0x7878u) | ((l4 << 15) & 0x78780000u);
            LDS_FENCE();
            const u32x2 wa = win_read(p3), wb = win_read(p3b);
            w3[k][0] = win_get(wa, p3);
            w3[k][1] = win_get(wb, p3b);
        }
    }
    unsigned q2[2][4];
#pragma unroll
    for (int k = 0; k < 2; ++k) {
        *(LDS_AS u32x2*)(sc0 + k * CL_BYTES + CL_A3 + lane * 8) = u32x2{vperm(w3[k][0], T3[k], S3[k][0]), vperm(w3[k][1], T3[k], S3[k][1])};
        const unsigned p2 = ((x2 >> (16 * k)) & 0xffffu) + lds_addr(sc0 + k * CL_BYTES + CL_A3);
#pragma unroll
        for (int j = 0; j < 4; ++j) q2[k][j] = j ? p2 + __builtin_popcount(E2[k] & ((1u << (8 * j)) - 1u)) : p2;
    }
    PHASE();
    u32x2 W2[2][4];
#pragma unroll
    for (int k = 0; k < 2; ++k)
#pragma unroll
        for (int j = 0; j < 4; ++j) W2[k][j] = win_read(q2[k][j]);
    PHASE();
    unsigned q1[2][8];
#pragma unroll
    for (int k = 0; k < 2; ++k) {
        unsigned p1 = (x1 >> (16 * k)) & 0xffffu;
        p1 = (p1 > (unsigned)CAP1 ? (unsigned)CAP1 : p1) + lds_addr(sc0 + k * CL_BYTES + CL_A2);
        const unsigned p1b = p1 + __builtin_popcount(E0[k]);
#pragma unroll
        for (int j = 0; j < 8; ++j) {
            const unsigned base = (j & 4) ? p1b : p1, E = (j & 4) ? E1[k] : E0[k];
            q1[k][j] = (j & 3) ? base + __builtin_popcount(E & ((1u << (8 * (j & 3))) - 1u)) : base;
        }
    }
    PHASE();
#pragma unroll
    for (int k = 0; k < 2; ++k)
        *(LDS_AS u32x4*)(sc0 + k * CL_BYTES + CL_A2 + lane * 16) =
            u32x4{vperm(win_get(W2[k][0], q2[k][0]), T2[k], S2[k][0]), vperm(win_get(W2[k][1], q2[k][1]), T2[k], S2[k][1]),
                  vperm(win_get(W2[k][2], q2[k][2]), T2[k], S2[k][2]), vperm(win_get(W2[k][3], q2[k][3]), T2[k], S2[k][3])};
    PHASE();
    u32x2 W1[2][8];
#pragma unroll
    for (int k = 0; k < 2; ++k)
#pragma unroll
        for (int j = 0; j < 8; ++j) { W1[k][j] = win_read(q1[k][j]); S1[k][j] = lut_sel((j & 4) ? L1[k].y : L1[k].x, j & 3); }
    PHASE();
#pragma unroll
    for (int k = 0; k < 2; ++k)
#pragma unroll
        for (int j = 0; j < 8; ++j) X[k][j] = vperm(win_get(W1[k][j], q1[k][j]), T1[k], S1[k][j]);
}
__device__ __forceinline__ void merge_part(const u32x4 N, const unsigned (&X)[8], u32x4& R0, u32x4& R1) {
    constexpr unsigned K = 0x87878787u;
    R0 = u32x4{(N.x & K) | X[0], (__builtin_amdgcn_alignbit(N.x, N.x, 4) & K) | X[1], (N.y & K) | X[2], (__builtin_amdgcn_alignbit(N.y, N.y, 4) & K) | X[3]};
    R1 = u32x4{(N.z & K) | X[4], (__builtin_amdgcn_alignbit(N.z, N.z, 4) & K) | X[5], (N.w & K) | X[6], (__builtin_amdgcn_alignbit(N.w, N.w, 4) & K) | X[7]};
}

constexpr int NSLOT = 8;
constexpr int CM_LUT = 0;
constexpr int CM_META = 1024;
constexpr int CM_DONE = CM_META + NSLOT * 16;
constexpr int CM_LAND = CM_DONE + NSLOT * 4;
constexpr int CM_NEXT = CM_LAND + 8;
constexpr int CM_TAB = 2048;
constexpr int CM_SLOT = 3072;
constexpr int CM_SC = CM_SLOT + NSLOT * UNIT;
constexpr int ST_LUT = 0, ST_TOP = 2048, ST_SC = ST_TOP + 64, ST_TAIL = ST_SC + 8 * CL_BYTES;
static_assert(CM_SC + 14 * CL_BYTES <= LDS_BYTES && CM_NEXT + 4 <= CM_META + 256 && ST_TAIL + 2048 <= LDS_BYTES, "LDS map");
static_assert(NPULLU >= NSTAGE && NPULLU <= 2 * NSTAGE && WIN <= 8, "roles");

__device__ __forceinline__ bool timed_out(unsigned& spins, unsigned* tmo, unsigned call, unsigned long long t0) {
    if ((++spins & 1023u) != 0u) return false;
    if (__hip_atomic_load(tmo, __ATOMIC_RELAXED, __HIP_MEMORY_SCOPE_AGENT) != call &&
        __builtin_amdgcn_s_memrealtime() - t0 <= DEADLINE_TICKS) return false;
    __hip_atomic_store(tmo, call, __ATOMIC_RELAXED, __HIP_MEMORY_SCOPE_AGENT);
    return true;
}

__global__ void __launch_bounds__(BLOCK, 2) ag_gemm(Params p) {
    extern __shared__ __attribute__((aligned(16))) unsigned char smem[];
    const int lane = threadIdx.x & 63;
    const int wave = __builtin_amdgcn_readfirstlane(threadIdx.x >> 6);
    const int bid = blockIdx.x;
    unsigned char* const peer = p.ws[((bid >> 3) + 1) & 7];
    asm volatile("" ::"s"(p.x), "s"(p.w), "s"(p.sx), "s"(p.sw), "s"(p.out), "s"(p.rank), "s"(peer), "s"(p.ws[0]));
    unsigned char* const myws = p.ws[0];
    unsigned* const cntp = (unsigned*)(myws + CNT_OFF + (size_t)bid * 128);
    unsigned* const tmo = (unsigned*)(myws + TMO_OFF);
    const unsigned char* const xrow = p.x + (size_t)(lane & 15) * KD + (lane >> 4) * 16;
    LDS_AS unsigned char* const lds = (LDS_AS unsigned char*)smem;
    const unsigned long long t0 = __builtin_amdgcn_s_memrealtime();
    unsigned call;
#define LOAD_COUNT() unsigned cv; asm volatile("global_load_dword %0, %1, off sc1" : "=&v"(cv) : "v"(cntp) : "memory")

    if (__builtin_expect(bid >= NCOMM && bid < NCOMM + NPUSHWG, 1)) {
        LOAD_COUNT();
        const int c_k = bid & 7;
        u32x4 A[PUSH_BLOCKS];
        const unsigned char* s = xrow + c_k * 512 + wave * 64;
        ld16<0>(A[0], s);
        __builtin_amdgcn_s_barrier();
#pragma unroll
        for (int i = 1; i < PUSH_BLOCKS; ++i) ld16<0>(A[i], s + (size_t)i * (16 * KD));
        unsigned char* dst = peer + X_OFF + (size_t)p.rank * SHARD + (size_t)c_k * UNIT + wave * 1024 + lane * 16;
        asm volatile("s_waitcnt vmcnt(%3)" : "+v"(cv), "+v"(A[0]), "+v"(dst) : "i"(PUSH_BLOCKS - 1) : "memory");
        call = __builtin_amdgcn_readfirstlane(cv) + 1u;
        dst += (size_t)(call & 1u) * NR * SHARD;
        st16_sys<0>(dst, A[0]);
#pragma unroll
        for (int i = 1; i < PUSH_BLOCKS; ++i) {
            if (wave < PUSH_DEEP) wait_vm<1>(); else wait_vm<0>();
            asm volatile("" : "+v"(A[i]));
            if (i < PUSH_BLOCKS - 1 || c_k < PUSH_LAST) st16_sys<0>(dst + (size_t)i * (8 * UNIT), A[i]);
        }
        wait_vm<0>();
        __builtin_amdgcn_s_barrier();
        if (wave == 0) {
            unsigned char* fp = peer + FLAG_OFF + ((size_t)p.rank * NUNIT + c_k + 8 * (lane >> 1)) * 128 + (lane & 1) * 8;
            asm volatile("s_mov_b64 exec, %2\n\tglobal_store_dword %0, %1, off sc0 sc1\n\ts_mov_b64 exec, -1"
                         ::"v"(fp), "v"(call), "s"((1ull << (c_k < PUSH_LAST ? 2 * PUSH_BLOCKS : 2 * PUSH_BLOCKS - 2)) - 1ull) : "memory");
        }
    } else if (__builtin_expect(bid >= NCOMM, 1)) {
        LOAD_COUNT();
        const unsigned d = wave & 3, sn = bid - NCOMM - NPUSHWG;
        const unsigned su0 = NPUSHU + (wave < 4 ? (sn + 2 * NSTAGE - NPULLU) & (NSTAGE - 1) : NSTAGE + sn);
        const bool stage = su0 < NUNIT;
        const unsigned su = stage ? su0 : NUNIT - 1;
        u32x4 R0, R1;
        {
            const unsigned char* s = xrow + (size_t)(su >> 3) * (16 * KD) + (su & 7) * 512 + d * 128;
            ld16<0>(R0, s); ld16<64>(R1, s);
        }
        unsigned char* wsl;
        asm volatile("global_load_dwordx2 %0, %1, off" : "=&v"(wsl)
                     : "v"((const unsigned char*)__builtin_amdgcn_kernarg_segment_ptr() + offsetof(Params, ws) + (lane < 8 ? lane * 8 : 0)) : "memory");
        LDS_AS unsigned char* const sc = lds + ST_SC + wave * CL_BYTES;
        LDS_AS unsigned char* const tb = lds + ST_TAIL + (wave >> 2) * 1024;
        LDS_AS unsigned* const top = (LDS_AS unsigned*)(lds + ST_TOP) + (wave >> 2);
        encode_clear(sc, lane);
        *top = 16u;
        if (threadIdx.x >= 256) *(LDS_AS u32x2*)(lds + ST_LUT + (threadIdx.x - 256) * 8) = lut_entry(threadIdx.x - 256, true);
        wait_lgkm();
        __builtin_amdgcn_s_barrier();
        asm volatile("s_waitcnt vmcnt(1)" : "+v"(cv), "+v"(R0), "+v"(R1) :: "memory");
        call = __builtin_amdgcn_readfirstlane(cv) + 1u;
        unsigned char* const cp = myws + C_OFF + (size_t)(call & 1u) * SHARD + (size_t)su * UNIT;
        unsigned char* const x = myws + X_OFF + ((size_t)(call & 1u) * NR + p.rank) * SHARD + (size_t)su * UNIT + d * 2048 + lane * 16;
        bool rawdone = false, packed = false;
        if (stage && su < NUNIT - RAW_UNITS)
            packed = encode_part(R0, R1, d, sc, tb, top, lds + ST_LUT, lane, [&](const Packed& q) {
                st16_sys<0>(cp + d * 1024 + lane * 16, q.N);
                asm volatile("global_store_dwordx2 %0, %1, off sc0 sc1" ::"v"(cp + 4096 + d * 512 + lane * 8), "v"(q.L1) : "memory");
                st16_sys<0>(x, R0); st16_sys<1024>(x, R1);
                rawdone = true;
            });
        if (stage && !rawdone) { st16_sys<0>(x, R0); st16_sys<1024>(x, R1); }
        if (!packed) lds_or<0>(lds_addr(top), 0x10000u);
        asm volatile("s_waitcnt vmcnt(0) lgkmcnt(0)" : "+v"(wsl) :: "memory");
        __builtin_amdgcn_s_barrier();
        LDS_FENCE();
        const unsigned tv = *top;
        const u32x4 tl = *(const LDS_AS u32x4*)(tb + lane * 16);
        if (d == 0 && packed) st16_sys<0>(cp + 6144 + lane * 16, tl);
        const unsigned total = __builtin_amdgcn_readfirstlane(tv);
        unsigned char* fp = wsl + (lane < 8 ? RDY_OFF + (size_t)(call & 1u) * NR * NUNIT * 128 : FLAG_OFF + (lane & 1) * 8) + ((size_t)p.rank * NUNIT + su) * 128;
        u32x2 fv = u32x2{lane >= 8 ? call : (total <= (unsigned)TAIL_MAX ? (total + 15u) >> 4 : INFO_RAW), call};
        asm volatile("s_waitcnt vmcnt(0)" : "+v"(fp), "+v"(fv) :: "memory");
        if (d == 0 && stage)
            asm volatile("s_mov_b64 exec, 0x3fe\n\tglobal_store_dwordx2 %0, %1, off sc0 sc1\n\ts_mov_b64 exec, -1" ::"v"(fp), "v"(fv) : "memory");
        __builtin_amdgcn_s_barrier();
    } else {
        const int c_src = bid >> 3, c_k = bid & 7;
        const int src_rank = (p.rank + c_src + 1) & 7;
        if (wave == 1) {
            const unsigned k = lane & 7;
            const unsigned char* tp = k == 0 ? peer + X_OFF + (size_t)p.rank * SHARD : k == 1 ? peer + RDY_OFF
                                    : k == 2 ? peer + C_OFF : k == 3 ? peer + X_OFF + (size_t)src_rank * SHARD + (SHARD - 128)
                                    : k == 4 ? myws + X_OFF + (size_t)src_rank * SHARD : k == 5 ? (const unsigned char*)p.out + (size_t)src_rank * (MSH * NSH * 4)
                                    : k == 6 ? p.w : (const unsigned char*)p.sx;
            if (lane >= 8) tp = k == 2 ? tp + SHARD : (k == 0 || k == 3 || k == 4) ? tp + NR * SHARD : k == 1 ? peer + FLAG_OFF : k == 7 ? (const unsigned char*)p.sw : tp;
            asm volatile("s_mov_b32 m0, %1\n\ts_mov_b64 exec, 0xffff\n\tglobal_load_lds_dword %0, off sc0 sc1\n\ts_mov_b64 exec, -1"
                         ::"v"(tp), "s"(lds_addr(lds + CM_SC)) : "memory");
        }
        if (c_src == 7 || lds_addr(lds + CM_LUT) != 0u) return;
        LOAD_COUNT();
        if (threadIdx.x < 256) *(LDS_AS unsigned*)(lds + CM_LUT + threadIdx.x * 4) = lut_entry(threadIdx.x, false).x;
        else *(LDS_AS unsigned*)(lds + CM_TAB + (threadIdx.x - 256) * 4) = 0u;
        if (wave == 0) *(LDS_AS unsigned*)(lds + CM_META + lane * 4) = 0u;
        asm volatile("s_waitcnt vmcnt(0) lgkmcnt(0)" : "+v"(cv) :: "memory");
        __builtin_amdgcn_s_barrier();
        call = __builtin_amdgcn_readfirstlane(cv) + 1u;
        const size_t xo = X_OFF + ((size_t)(call & 1u) * NR + src_rank) * SHARD;
        LDS_AS unsigned* const tab = (LDS_AS unsigned*)(lds + CM_TAB);
        if (wave == 0) {
            const unsigned char* const rawp = peer + xo + lane * 16;
            const unsigned char* const cmpp = peer + C_OFF + (size_t)(call & 1u) * SHARD + lane * 16;
            unsigned char* const tick = myws + TICK_OFF + c_src * 128;
            const unsigned tbase = (call - 1u) * (NPULLU - 8) - 16u - NPUSHU;
            unsigned q = 0;
            unsigned n = 16;
#define LANDED(k) do { wait_vm<k>(); if (lane == 0) *(volatile LDS_AS unsigned*)(lds + CM_LAND) = n - (k); } while (0)
#define STEP() do { ++n; LANDED(WIN - 1); } while (0)
            auto staged = [&](unsigned u) -> unsigned {
                if (u - NPUSHU >= (unsigned)NPULLU) return INFO_FAIL;
                unsigned info;
                while ((info = __builtin_amdgcn_readfirstlane(__hip_atomic_load(tab + u, __ATOMIC_ACQUIRE, __HIP_MEMORY_SCOPE_WORKGROUP))) == 0u) {}
                return info;
            };
            auto first_half = [&](unsigned u, unsigned info) -> const unsigned char* {
                const unsigned slot = q & (NSLOT - 1);
                if (lane == 0) *(volatile LDS_AS u32x4*)(lds + CM_META + slot * 16) = u32x4{q + 1u, u | (info & INFO_RAW), n, 0u};
                const unsigned la = lds_addr(lds + CM_SLOT) + slot * UNIT + 4096;
                const unsigned char* g = ((info & INFO_RAW) ? rawp : cmpp) + (size_t)u * UNIT;
                if (info & INFO_RAW) {
                    glds_sys<0>(g + 4096, la); STEP(); glds_sys<1024>(g + 4096, la); STEP();
                    glds_sys<2048>(g + 4096, la); STEP(); glds_sys<3072>(g + 4096, la); STEP();
                } else {
                    glds_sys_mask<2048>(g + 4096, la, info >= 64u ? ~0ull : (1ull << info) - 1ull); STEP();
                    glds_sys<0>(g + 4096, la); STEP(); glds_sys<1024>(g + 4096, la); STEP();
                }
                ++q;
                return g;
            };
            auto second_half = [&](unsigned qq, const unsigned char* g) {
                const unsigned slot = qq & (NSLOT - 1);
                if (lane == 0) *(volatile LDS_AS unsigned*)(lds + CM_META + slot * 16 + 12) = n;
                const unsigned la = lds_addr(lds + CM_SLOT) + slot * UNIT;
                glds_sys<0>(g, la); STEP(); glds_sys<1024>(g, la); STEP(); glds_sys<2048>(g, la); STEP(); glds_sys<3072>(g, la); STEP();
            };
            unsigned u = NPUSHU + c_k;
            unsigned info = staged(u);
            if (info != INFO_FAIL) {
                const unsigned char* gcur = first_half(u, info);
                u += 8u;
                while ((info = staged(u)) != INFO_FAIL) {
                    if (q >= NSLOT)
                        while (*(volatile LDS_AS unsigned*)(lds + CM_DONE + (q & (NSLOT - 1)) * 4) != 2u * (q / NSLOT)) __builtin_amdgcn_s_sleep(1);
                    unsigned tv;
                    ticket(tv, tick); ++n;
                    const unsigned char* gnext = first_half(u, info);
                    second_half(q - 2u, gcur);
                    asm volatile("" : "+v"(tv));
                    u = __builtin_amdgcn_readfirstlane(tv) - tbase;
                    gcur = gnext;
                }
                second_half(q - 1u, gcur);
            }
            if (lane == 0) *(volatile LDS_AS unsigned*)(lds + CM_LAND + 4) = q + 1u;
            LANDED(6); LANDED(5); LANDED(4); LANDED(3); LANDED(2); LANDED(1); LANDED(0);
            return;
        }
        if (wave == 7) {
            flag_store((unsigned char*)cntp, call);
            const unsigned char* rp = myws + RDY_OFF + ((size_t)(call & 1u) * NR + src_rank) * NUNIT * 128;
            unsigned spins = 0;
            for (int u0 = NPUSHU + 16 - 64; u0 < NUNIT; u0 += 64) {
                const int u = u0 < NPUSHU ? NPUSHU + c_k + (lane & 1) * 8 : (u0 + lane < NUNIT ? u0 + lane : NUNIT - 1);
                while (true) {
                    u32x2 r;
                    asm volatile("global_load_dwordx2 %0, %1, off sc0 sc1\n\ts_waitcnt vmcnt(0)" : "=&v"(r) : "v"(rp + (size_t)u * 128) : "memory");
                    const unsigned t = r.y >= call ? r.x : 0u;
                    __hip_atomic_store(tab + u, t, __ATOMIC_RELEASE, __HIP_MEMORY_SCOPE_WORKGROUP);
                    if (__all(t != 0u)) break;
                    if (timed_out(spins, tmo, call, t0)) {
                        *(volatile LDS_AS u32x4*)(lds + CM_TAB + lane * 16) = u32x4{INFO_FAIL, INFO_FAIL, INFO_FAIL, INFO_FAIL};
                        u0 = NUNIT;
                        break;
                    }
                }
            }
        }
        LDS_AS unsigned char* const sc = lds + CM_SC + (wave - 1) * (2 * CL_BYTES);
        unsigned char* const dst = myws + xo + lane * 16;
        unsigned char* const fl = myws + FLAG_OFF + (size_t)src_rank * NUNIT * 128;
        while (true) {
            const unsigned r = lds_ticket(lds + CM_NEXT);
            const unsigned q = r >> 1, h = r & 1u;
            const unsigned slot = q & (NSLOT - 1);
            const LDS_AS unsigned char* const sl = lds + CM_SLOT + slot * UNIT;
            unsigned uw;
            bool end = false;
            while (true) {
                const u32x4 m = uniform4(*(volatile LDS_AS u32x4*)(lds + CM_META + slot * 16));
                const u32x2 lv = *(volatile LDS_AS u32x2*)(lds + CM_LAND);
                const unsigned land = __builtin_amdgcn_readfirstlane(lv.x), e = __builtin_amdgcn_readfirstlane(lv.y);
                uw = m.y;
                if (m.x == q + 1u) {
                    if (land >= m.z + ((uw & INFO_RAW) ? 4u * h : 2u + h)) break;
                } else if (e - 1u <= q) { end = true; break; }
                __builtin_amdgcn_s_sleep(1);
            }
            if (end) break;
            LDS_FENCE();
            const bool raw = (uw & INFO_RAW) != 0u;
            unsigned X[2][8];
            if (!raw) {
                decode_exps(sl, 2u * h, X, sc, lane);
#pragma unroll
                for (int k = 0; k < 2; ++k)
                    asm volatile("" : "+v"(X[k][0]), "+v"(X[k][1]), "+v"(X[k][2]), "+v"(X[k][3]), "+v"(X[k][4]), "+v"(X[k][5]), "+v"(X[k][6]), "+v"(X[k][7]));
            }
            if (!raw || h == 0u) {
                while (true) {
                    const unsigned ib = __builtin_amdgcn_readfirstlane(*(volatile LDS_AS unsigned*)(lds + CM_META + slot * 16 + 12));
                    const unsigned land = __builtin_amdgcn_readfirstlane(*(volatile LDS_AS unsigned*)(lds + CM_LAND));
                    if (ib != 0u && land >= ib + (raw ? 4u : 2u * h + 2u)) break;
                }
                LDS_FENCE();
            }
            u32x4 R[4];
            if (raw) {
#pragma unroll
                for (int i = 0; i < 4; ++i) R[i] = *(const LDS_AS u32x4*)(sl + h * 4096 + i * 1024 + lane * 16);
            } else {
                const u32x4 N0 = *(const LDS_AS u32x4*)(sl + h * 2048 + lane * 16), N1 = *(const LDS_AS u32x4*)(sl + h * 2048 + 1024 + lane * 16);
                merge_part(N0, X[0], R[0], R[1]);
                merge_part(N1, X[1], R[2], R[3]);
            }
            const unsigned u = uw & (NUNIT - 1);
            unsigned char* const o = dst + (size_t)u * UNIT + h * 4096;
            st16_sys<0>(o, R[0]); st16_sys<1024>(o, R[1]); st16_sys<2048>(o, R[2]); st16_sys<3072>(o, R[3]);
            lds_inc(lds + CM_DONE + slot * 4);
            wait_vm<0>();
            flag_store(fl + (size_t)u * 128 + h * 8, call);
        }
        return;
    }

    const int cn = bid - NCOMM;
    const int cg = (cn >> 3) & 7;
    const int src_rank = (p.rank + cn + 3 * (cn >> 6)) & 7;
    if (wave == 7) flag_store((unsigned char*)cntp, call);

    i32x8 Wf[4][4];
    LDS_AS unsigned char* const wl = lds + wave * 16384;
    {
        const unsigned char* wsrc = p.w + (size_t)(wave * 512 + (lane >> 2)) * NSH + cg * 64 + (lane & 3) * 16;
        const int g = lane >> 4, q = (lane & 15) >> 1, h = lane & 1;
        LDS_AS unsigned char* const rd = wl + (16 * g + q) * 64 + h * 8;
#pragma unroll
        for (int half = 0; half < 2; ++half) {
#pragma unroll
            for (int i = 0; i < 16; ++i)
                __builtin_amdgcn_global_load_lds((const GLB_AS unsigned*)(wsrc + (size_t)(half * 256 + i * 16) * NSH),
                                                 (LDS_AS unsigned*)(wl + i * 1024), 16, 0, 0);
            wait_vm<0>();
#pragma unroll
            for (int s2 = 0; s2 < 2; ++s2)
#pragma unroll
                for (int t = 0; t < 4; ++t)
#pragma unroll
                    for (int pc = 0; pc < 4; ++pc) {
                        i32x2 v = __builtin_amdgcn_ds_read_tr8_b64_v2i32(
                            (LDS_AS i32x2*)(rd + (s2 * 128 + (pc >> 1) * 64 + (pc & 1) * 8) * 64 + t * 16));
                        Wf[half * 2 + s2][t][pc * 2] = v.x;
                        Wf[half * 2 + s2][t][pc * 2 + 1] = v.y;
                    }
            wait_lgkm();
        }
    }
    const float scale = p.sx[0] * p.sw[0];
    const unsigned char* const xl = myws + X_OFF + ((size_t)(call & 1u) * NR + src_rank) * SHARD + wave * UNIT + 4096 + lane * 16;
    const unsigned* const flags = (const unsigned*)(myws + FLAG_OFF + ((size_t)src_rank * NUNIT + wave) * 128 + (lane & 1) * 8);
    LDS_AS unsigned char* const pw = lds + wave * PART_BYTES + (lane & 15) * PART_STRIDE + (lane >> 4) * 16;
    const LDS_AS unsigned char* const pr = lds + (wave * 4 + (lane >> 4)) * PART_STRIDE + (lane & 15) * 16;
    float* const outp = p.out + (size_t)(src_rank * MSH + wave * 4 + (lane >> 4)) * NSH + cg * 64 + (lane & 15) * 4;
    __builtin_amdgcn_s_barrier();
    int buf = 0;
    bool bad = false;
    for (int b = 2 - (cn >> 6); b < NBLK; b += 3) {
        const unsigned* f = flags + (size_t)b * (8 * 32);
        unsigned spins = 0;
        while (!bad && !__all(__hip_atomic_load(f, __ATOMIC_RELAXED, __HIP_MEMORY_SCOPE_SYSTEM) >= call)) {
            bad = timed_out(spins, tmo, call, t0);
            __builtin_amdgcn_s_sleep(1);
        }
        u32x4 R[8];
        asm volatile(
            "global_load_dwordx4 %0, %8, off offset:-4096 sc0 sc1\n\tglobal_load_dwordx4 %1, %8, off offset:-3072 sc0 sc1\n\t"
            "global_load_dwordx4 %2, %8, off offset:-2048 sc0 sc1\n\tglobal_load_dwordx4 %3, %8, off offset:-1024 sc0 sc1\n\t"
            "global_load_dwordx4 %4, %8, off sc0 sc1\n\tglobal_load_dwordx4 %5, %8, off offset:1024 sc0 sc1\n\t"
            "global_load_dwordx4 %6, %8, off offset:2048 sc0 sc1\n\tglobal_load_dwordx4 %7, %8, off offset:3072 sc0 sc1\n\t"
            "s_waitcnt vmcnt(0)"
            : "=&v"(R[0]), "=&v"(R[1]), "=&v"(R[2]), "=&v"(R[3]), "=&v"(R[4]), "=&v"(R[5]), "=&v"(R[6]), "=&v"(R[7])
            : "v"(xl + (size_t)b * (8 * UNIT)) : "memory");
        const float c0 = bad ? __builtin_nanf("") : 0.0f;
        f32x4 acc[4];
#pragma unroll
        for (int t = 0; t < 4; ++t) acc[t] = f32x4{c0, c0, c0, c0};
#pragma unroll
        for (int s = 0; s < 4; ++s) {
            i32x8 xf;
            xf[0] = R[2 * s].x; xf[1] = R[2 * s].y; xf[2] = R[2 * s].z; xf[3] = R[2 * s].w;
            xf[4] = R[2 * s + 1].x; xf[5] = R[2 * s + 1].y; xf[6] = R[2 * s + 1].z; xf[7] = R[2 * s + 1].w;
#pragma unroll
            for (int t = 0; t < 4; ++t)
                acc[t] = __builtin_amdgcn_mfma_scale_f32_16x16x128_f8f6f4(Wf[s][t], xf, acc[t], 1, 0, 0, 0, 0, 0);
        }
        __builtin_amdgcn_sched_barrier(0);
#pragma unroll
        for (int t = 0; t < 4; ++t) *(LDS_AS f32x4*)(pw + buf * (8 * PART_BYTES) + t * 64) = acc[t];
        wait_lgkm();
        __builtin_amdgcn_s_barrier();
        LDS_FENCE();
        if (wave < 4) {
            f32x4 v[8];
#pragma unroll
            for (int w2 = 0; w2 < 8; ++w2) v[w2] = *(const LDS_AS f32x4*)(pr + buf * (8 * PART_BYTES) + w2 * PART_BYTES);
            __builtin_amdgcn_sched_barrier(0);
            const f32x4 sum = (((v[0] + v[1]) + (v[2] + v[3])) + ((v[4] + v[5]) + (v[6] + v[7]))) * scale;
            st16_sys<0>((unsigned char*)(outp + (size_t)b * (16 * NSH)), __builtin_bit_cast(u32x4, sum));
        }
        buf ^= 1;
    }
}

extern "C" void kernel_launch(int rank, int nranks, ncclComm_t comm, hipStream_t stream,
                              void* const* d_in, const int* in_sizes, int n_in,
                              void* d_out, int out_size,
                              const int* in_dtypes, int out_dtype,
                              void* const* d_out_all,
                              void* const* d_ws_all, size_t ws_bytes) {
    (void)comm; (void)in_dtypes; (void)out_dtype; (void)d_out_all;
    if (nranks != NR || (unsigned)rank >= (unsigned)NR || n_in != 4 || ws_bytes < WS_NEED ||
        in_sizes[0] != (int)SHARD || in_sizes[1] != KD * NSH || out_size != NR * MSH * NSH) {
        fprintf(stderr, "unexpected problem shape\n");
        return;
    }
    static bool init = false;
    if (!init) {
        HIP_CHECK(hipFuncSetAttribute((const void*)ag_gemm, hipFuncAttributeMaxDynamicSharedMemorySize, LDS_BYTES));
        init = true;
    }
    Params p;
    p.x = (const unsigned char*)d_in[0];
    p.w = (const unsigned char*)d_in[1];
    p.sx = (const float*)d_in[2];
    p.sw = (const float*)d_in[3];
    p.out = (float*)d_out;
    for (int d = 0; d < NR; ++d) p.ws[d] = (unsigned char*)d_ws_all[(rank + d) % NR];
    p.rank = rank;
    p.pad = 0;
    hipLaunchKernelGGL(ag_gemm, dim3(GRID), dim3(BLOCK), LDS_BYTES, stream, p);
    HIP_CHECK(hipGetLastError());
}
```

```cpp
#include <hip/hip_runtime.h>
#include <rccl/rccl.h>
#include <cstddef>
#include <cstdio>

#define HIP_CHECK(expr) do { hipError_t _e = (expr); if (_e != hipSuccess) { \
    fprintf(stderr, "HIP error %s at %s:%d\n", hipGetErrorString(_e), __FILE__, __LINE__); \
    return; } } while (0)

typedef unsigned u32x2 __attribute__((ext_vector_type(2)));
typedef unsigned u32x4 __attribute__((ext_vector_type(4)));
typedef int i32x2 __attribute__((ext_vector_type(2)));
typedef int i32x8 __attribute__((ext_vector_type(8)));
typedef float f32x4 __attribute__((ext_vector_type(4)));
#define LDS_AS __attribute__((address_space(3)))
#define GLB_AS __attribute__((address_space(1)))

constexpr int NR = 8;
constexpr int MSH = 512, KD = 4096, NSH = 512;
constexpr size_t SHARD = (size_t)MSH * KD;
constexpr int NBLK = MSH / 16;
constexpr int NUNIT = NBLK * 8;
constexpr int UNIT = 8192;
constexpr int GRID = 256, BLOCK = 512;
constexpr int NCOMM = 64;
constexpr int NPB = 5, PBS = 5;
constexpr int PUSH_ROUNDS = (NPB + 1) / 2;
constexpr int NDYN = NUNIT - 8 * NPB;
constexpr int NRDY = NUNIT + 8;
constexpr int RAW_UNITS = 8;
constexpr int WIN = 8;
constexpr size_t X_OFF = 0;
constexpr size_t C_OFF = X_OFF + 2 * NR * SHARD;
constexpr size_t NI_OFF = C_OFF + 2 * SHARD;
constexpr size_t RDY_OFF = NI_OFF + (size_t)NR * 2 * 64 * 4096;
constexpr size_t FLAG_OFF = RDY_OFF + (size_t)2 * NR * NRDY * 128;
constexpr size_t CNT_OFF = FLAG_OFF + (size_t)NR * NUNIT * 128;
constexpr size_t TICK_OFF = CNT_OFF + (size_t)GRID * 128;
constexpr size_t TMO_OFF = TICK_OFF + NR * 128;
constexpr size_t WS_NEED = TMO_OFF + 128;
constexpr unsigned long long DEADLINE_TICKS = 100000000ull;
constexpr unsigned long long LDS_DEADLINE_TICKS = 150000000ull;
constexpr int LDS_BYTES = 8 * 16384;
constexpr int PART_STRIDE = 272, PART_BYTES = 16 * PART_STRIDE;

struct Params {
    const unsigned char* x;
    const unsigned char* w;
    const float* sx;
    const float* sw;
    float* out;
    unsigned char* ws[NR];
    int rank;
    int pad;
};

#define LDS_FENCE() asm volatile("" ::: "memory")
template <int N> __device__ __forceinline__ void wait_vm() { asm volatile("s_waitcnt vmcnt(%0)" ::"i"(N) : "memory"); }
__device__ __forceinline__ void wait_lgkm() { asm volatile("s_waitcnt lgkmcnt(0)" ::: "memory"); }
__device__ __forceinline__ unsigned lds_addr(const LDS_AS void* p) { return (unsigned)(size_t)p; }

template <int OFF> __device__ __forceinline__ void ld16(u32x4& v, const unsigned char* base, unsigned off) {
    asm volatile("global_load_dwordx4 %0, %1, %2 offset:%3" : "=&v"(v) : "v"(off), "s"(base), "i"(OFF) : "memory");
}
template <int OFF> __device__ __forceinline__ void st16_sys(unsigned char* p, u32x4 v) {
    asm volatile("global_store_dwordx4 %0, %1, off offset:%2 sc0 sc1\n\ts_nop 1" ::"v"(p), "v"(v), "i"(OFF) : "memory");
}
__device__ __forceinline__ void flag_store(unsigned char* p, unsigned v) {
    asm volatile("s_mov_b64 exec, 1\n\tglobal_store_dword %0, %1, off sc0 sc1\n\ts_mov_b64 exec, -1" ::"v"(p), "v"(v) : "memory");
}
__device__ __forceinline__ void ticket(unsigned& v, unsigned char* p) {
    asm volatile("s_mov_b64 exec, 1\n\tglobal_atomic_add %0, %1, %2, off sc0\n\ts_mov_b64 exec, -1" : "=&v"(v) : "v"(p), "v"(1u) : "memory");
}
__device__ __forceinline__ void lds_inc(LDS_AS unsigned char* p) {
    asm volatile("s_mov_b64 exec, 1\n\tds_add_u32 %0, %1\n\ts_mov_b64 exec, -1" ::"v"(lds_addr(p)), "v"(1u) : "memory");
}
__device__ __forceinline__ unsigned lds_ticket(LDS_AS unsigned char* p) {
    unsigned v;
    asm volatile("s_mov_b64 exec, 1\n\tds_add_rtn_u32 %0, %1, %2\n\ts_mov_b64 exec, -1\n\ts_waitcnt lgkmcnt(0)"
                 : "=&v"(v) : "v"(lds_addr(p)), "v"(1u) : "memory");
    return __builtin_amdgcn_readfirstlane(v);
}
template <int OFF> __device__ __forceinline__ void glds_sys(const unsigned char* g, unsigned lds) {
    asm volatile("s_mov_b32 m0, %1\n\ts_nop 0\n\tglobal_load_lds_dwordx4 %0, off offset:%2 sc0 sc1" ::"v"(g), "s"(lds), "i"(OFF) : "memory");
}

constexpr int CAP1 = 1024, CAP2 = 512, CAP3 = 256, TAIL_MAX = 1024;
constexpr int CL_A2 = 0;
constexpr int CL_A3 = CAP1 + 16;
constexpr int CL_A4 = CL_A3 + CAP1 + 16;
constexpr int CL_BYTES = CL_A4 + CAP2 + 32;
constexpr unsigned INFO_RAW = 0x80u, INFO_FAIL = 0xffu;
constexpr unsigned rel8(int a, int b, int c) { return (unsigned)(((a & 15) << 3) | ((b & 15) << 11) | ((c & 15) << 19)); }
constexpr unsigned map_byte(int r, bool var) {
    return r <= 2 ? (unsigned)r << 3 : r == 15 ? 0x18u : r == 14 ? 0x19u : r == (var ? 3 : 13) ? 0x1au
         : r == 12 ? 0x1bu : r == 11 ? 0x1fu : r == (var ? 13 : 3) ? 0x9bu : 0x9fu;
}
constexpr unsigned map_word(int r, bool var) {
    return map_byte(r, var) | (map_byte(r + 1, var) << 8) | (map_byte(r + 2, var) << 16) | (map_byte(r + 3, var) << 24);
}

__device__ __forceinline__ unsigned vperm(unsigned hi, unsigned lo, unsigned sel) { return __builtin_amdgcn_perm(hi, lo, sel); }
__device__ __forceinline__ unsigned bfi(unsigned m, unsigned a, unsigned b) { return (a & m) | (b & ~m); }
__device__ __forceinline__ unsigned wave_incl_scan(unsigned v) {
    v += __builtin_amdgcn_update_dpp(0u, v, 0x111, 0xf, 0xf, false);
    v += __builtin_amdgcn_update_dpp(0u, v, 0x112, 0xf, 0xf, false);
    v += __builtin_amdgcn_update_dpp(0u, v, 0x114, 0xf, 0xf, false);
    v += __builtin_amdgcn_update_dpp(0u, v, 0x118, 0xf, 0xf, false);
    v += __builtin_amdgcn_update_dpp(0u, v, 0x142, 0xa, 0xf, false);
    v += __builtin_amdgcn_update_dpp(0u, v, 0x143, 0xc, 0xf, false);
    return v;
}
__device__ __forceinline__ unsigned esc_bits(unsigned l) { return l & (l >> 1) & 0x55555555u; }

__device__ __forceinline__ u32x2 lut_entry(unsigned b, bool enc) {
    unsigned sel = 0, cnt = 0;
#pragma unroll
    for (int k = 0; k < 4; ++k) {
        const unsigned c = (b >> (2 * k)) & 3u;
        if (enc) { if (c == 3u) { sel |= (unsigned)k << (8 * cnt); ++cnt; } }
        else { sel |= (c == 3u ? 4u + cnt : c) << (8 * k); cnt += (c == 3u); }
    }
    if (enc && cnt < 4u) sel |= 0x07060504u << (8 * cnt);
    return u32x2{sel, 0x03020100u + (4u - cnt) * 0x01010101u};
}

template <int OFF> __device__ __forceinline__ void lds_or(unsigned a, unsigned v) {
    asm volatile("ds_or_b32 %0, %1 offset:%2" ::"v"(a), "v"(v), "i"(OFF) : "memory");
}
template <int NW, int J> __device__ __forceinline__ void compact_step(const unsigned (&P)[NW], const u32x2 (&E)[NW], unsigned& w, unsigned& w2, unsigned& u3) {
    w2 = vperm(w2, w, E[J].y);
    w = vperm(w, P[J], E[J].x);
    u3 += E[J].y & 0xffu;
    lds_or<4 * J>(u3 & ~3u, __builtin_amdgcn_alignbyte(w2, w, ~u3));
    if constexpr (J > 0) compact_step<NW, J - 1>(P, E, w, w2, u3);
}
template <int NW>
__device__ __forceinline__ void compact(const unsigned (&P)[NW], const u32x2 (&E)[NW], unsigned pos_end, LDS_AS unsigned char* dst) {
    unsigned w = 0, w2 = 0, u3 = lds_addr(dst) + pos_end + 3u - 4u * NW;
    compact_step<NW, NW - 1>(P, E, w, w2, u3);
    u3 -= 3u;
    lds_or<0>(u3 & ~3u, w << (8u * (u3 & 3u)));
}
template <int NW> __device__ __forceinline__ void lut_read(u32x2 (&E)[NW], const unsigned (&B8)[NW], const LDS_AS unsigned char* lut) {
#pragma unroll
    for (int j = NW - 1; j >= 0; --j) E[j] = *(const LDS_AS u32x2*)(lut + B8[j]);
    LDS_FENCE();
}

struct Packed { u32x4 N; u32x2 L1; };
__device__ __forceinline__ u32x4 nibble_plane(const u32x4 a, const u32x4 b) {
    constexpr unsigned K = 0x87878787u;
    return u32x4{bfi(K, a.x, __builtin_amdgcn_alignbit(a.y, a.y, 28)), bfi(K, a.z, __builtin_amdgcn_alignbit(a.w, a.w, 28)),
                 bfi(K, b.x, __builtin_amdgcn_alignbit(b.y, b.y, 28)), bfi(K, b.z, __builtin_amdgcn_alignbit(b.w, b.w, 28))};
}
__device__ __forceinline__ void tail_alloc(unsigned& v, LDS_AS unsigned* top, unsigned bytes) {
    asm volatile("s_mov_b64 exec, 1\n\tds_add_rtn_u32 %0, %1, %2\n\ts_mov_b64 exec, -1" : "=&v"(v) : "v"(lds_addr(top)), "v"(bytes) : "memory");
}

__device__ __forceinline__ void encode_clear(LDS_AS unsigned char* sc, int lane) {
    *(LDS_AS u32x4*)(sc + CL_A2 + lane * 16) = u32x4{0, 0, 0, 0};
    *(LDS_AS u32x2*)(sc + CL_A3 + lane * 8) = u32x2{0, 0};
    *(LDS_AS unsigned*)(sc + CL_A4 + lane * 4) = 0u;
}
template <class F>
__device__ __forceinline__ bool encode_part(const u32x4 R0, const u32x4 R1, unsigned d, LDS_AS unsigned char* sc, LDS_AS unsigned char* tb,
                                            LDS_AS unsigned* top, const LDS_AS unsigned char* lut, int lane, F planes) {
    Packed o;
    const unsigned X[8] = {R0.x, R0.y, R0.z, R0.w, R1.x, R1.y, R1.z, R1.w};
    unsigned s = 0;
#pragma unroll
    for (int j = 0; j < 8; ++j) s = __builtin_amdgcn_udot4(X[j] & 0x78787878u, 0x01010101u, s, false);
    s = __builtin_amdgcn_readlane(wave_incl_scan(s), 63);
    const unsigned mu = s + (16384u - 819u);
    const unsigned b = ((mu >> 14) - 1u) & 15u;
    const bool var = (mu & 16383u) >= 7373u;
    const unsigned K = ((16u - b) & 15u) * 0x08080808u;
    const unsigned M0 = var ? map_word(0, true) : map_word(0, false), M1 = map_word(4, true), M2 = map_word(8, true),
                   M3 = var ? map_word(12, true) : map_word(12, false);
    unsigned P[8], B8[8];
#pragma unroll
    for (int j = 0; j < 8; ++j) {
        const unsigned y = (X[j] & 0x78787878u) + K;
        const unsigned s7 = (y >> 3) & 0x07070707u;
        const unsigned hi = vperm(y << 1, y << 9, 0x0b090a08u);
        unsigned m = bfi(hi, vperm(M3, M2, s7), vperm(M1, M0, s7));
        asm("" : "+v"(m));
        B8[j] = __builtin_amdgcn_udot4(m & 0x18181818u, 0x40100401u, 0u, false);
        P[j] = bfi(0x78787878u, X[j], m);
    }
    u32x2 E1[8];
    lut_read<8>(E1, B8, lut);
    o.L1.x = ((B8[0] | (B8[1] << 8) | (B8[2] << 16)) >> 3) | (B8[3] << 21);
    o.L1.y = ((B8[4] | (B8[5] << 8) | (B8[6] << 16)) >> 3) | (B8[7] << 21);
    const unsigned c1 = __builtin_popcount(esc_bits(o.L1.x)) + __builtin_popcount(esc_bits(o.L1.y));
    const unsigned e1 = wave_incl_scan(c1);
    const unsigned n1 = __builtin_amdgcn_readlane(e1, 63);
    if (n1 > (unsigned)CAP1) return false;
    if (c1) compact<8>(P, E1, e1, sc + CL_A2);
    LDS_FENCE();
    const u32x4 v = *(const LDS_AS u32x4*)(sc + CL_A2 + lane * 16);
    LDS_FENCE();
    o.N = nibble_plane(R0, R1);
    planes(o);
    const unsigned Q[4] = {v.x, v.y, v.z, v.w};
    unsigned B2[4];
#pragma unroll
    for (int j = 0; j < 4; ++j) B2[j] = __builtin_amdgcn_udot4(Q[j] & 0x03030303u, 0x40100401u, 0u, false);
    const unsigned L2 = B2[0] | (B2[1] << 8) | (B2[2] << 16) | (B2[3] << 24);
#pragma unroll
    for (int j = 0; j < 4; ++j) B2[j] <<= 3;
    u32x2 E2[4];
    lut_read<4>(E2, B2, lut);
    const unsigned c2 = __builtin_popcount(esc_bits(L2));
    const unsigned e2 = wave_incl_scan(c2);
    const unsigned n2 = __builtin_amdgcn_readlane(e2, 63);
    if (n2 > (unsigned)CAP2) return false;
    const unsigned h2 = (n1 + 7u) >> 3, h3 = (n2 + 7u) >> 3;
    const unsigned s23 = (2u * (h2 + h3) + 3u) & ~3u;
    unsigned av;
    tail_alloc(av, top, s23);
    if (c2) compact<4>(Q, E2, e2, sc + CL_A3);
    LDS_FENCE();
    u32x2 a3 = *(const LDS_AS u32x2*)(sc + CL_A3 + lane * 8);
    asm volatile("s_waitcnt lgkmcnt(0)" : "+v"(av), "+v"(a3) :: "memory");
    const unsigned o2 = __builtin_amdgcn_readfirstlane(av);
    if (o2 + s23 > (unsigned)TAIL_MAX) return true;
    if ((unsigned)lane < (h2 + 1u) >> 1) *(LDS_AS unsigned*)(tb + o2 + lane * 4) = L2;
    LDS_FENCE();
    const unsigned Q3[2] = {a3.x, a3.y};
    unsigned B3[2];
#pragma unroll
    for (int j = 0; j < 2; ++j) B3[j] = __builtin_amdgcn_udot4(bfi(0x01010101u, Q3[j] >> 2, Q3[j] >> 6) & 0x03030303u, 0x40100401u, 0u, false);
    const unsigned L3 = B3[0] | (B3[1] << 8);
    if ((unsigned)lane < h3) *(LDS_AS unsigned short*)(tb + o2 + 2u * h2 + lane * 2) = (unsigned short)L3;
    const unsigned c3 = __builtin_popcount(esc_bits(L3));
    unsigned o4 = 0;
    if (__any(c3 != 0u)) {
        B3[0] <<= 3; B3[1] <<= 3;
        u32x2 E3[2];
        lut_read<2>(E3, B3, lut);
        const unsigned e3 = wave_incl_scan(c3);
        const unsigned n3 = __builtin_amdgcn_readlane(e3, 63);
        if (n3 > (unsigned)CAP3) return false;
        const unsigned s4 = ((n3 + 7u) >> 3) << 2;
        unsigned a4v;
        tail_alloc(a4v, top, s4);
        if (c3) compact<2>(Q3, E3, e3, sc + CL_A4);
        LDS_FENCE();
        unsigned a4 = *(const LDS_AS unsigned*)(sc + CL_A4 + lane * 4);
        asm volatile("s_waitcnt lgkmcnt(0)" : "+v"(a4v), "+v"(a4) :: "memory");
        o4 = __builtin_amdgcn_readfirstlane(a4v);
        if (o4 + s4 > (unsigned)TAIL_MAX) return true;
        if ((unsigned)lane < s4 >> 1) *(LDS_AS unsigned short*)(tb + o4 + lane * 2) = (unsigned short)bfi(0x0f0fu, a4 >> 3, a4 >> 15);
    }
    if (lane == 0) *(LDS_AS unsigned*)(tb + d * 4) = ((o2 >> 2) & 0xffu) | ((((o2 >> 1) + h2) & 0x1ffu) << 8) | (o4 << 15) | (b << 27) | (var ? 0x80000000u : 0u);
    return true;
}
__device__ __forceinline__ u32x4 uniform4(const u32x4 v) {
    return u32x4{(unsigned)__builtin_amdgcn_readfirstlane(v.x), (unsigned)__builtin_amdgcn_readfirstlane(v.y),
                 (unsigned)__builtin_amdgcn_readfirstlane(v.z), (unsigned)__builtin_amdgcn_readfirstlane(v.w)};
}

__device__ __forceinline__ u32x2 win_read(unsigned q) {
    const LDS_AS unsigned* p = (const LDS_AS unsigned*)(size_t)(q & ~3u);
    return u32x2{p[0], p[1]};
}
__device__ __forceinline__ unsigned win_get(const u32x2 w, unsigned q) { return __builtin_amdgcn_alignbyte(w.y, w.x, q); }
__device__ __forceinline__ unsigned lut_sel(unsigned w, int byte) { return *(const LDS_AS unsigned*)(size_t)(((w >> (8 * byte)) & 0xffu) << 2); }
#define PHASE() do { LDS_FENCE(); __builtin_amdgcn_sched_barrier(0); } while (0)
__device__ __forceinline__ void decode_exps(const LDS_AS unsigned char* sl, unsigned d0, unsigned (&X)[2][8], LDS_AS unsigned char* sc0, int lane) {
    const LDS_AS unsigned char* const tb = sl + 6144;
    unsigned hd[2], L2[2], L3[2], S1[2][8], S2[2][4], S3[2][2], E0[2], E1[2], E2[2], T1[2], T2[2], T3[2], w3[2][2];
    u32x2 L1[2];
    const u32x2 hv = *(const LDS_AS u32x2*)(tb + d0 * 4);
#pragma unroll
    for (int k = 0; k < 2; ++k) L1[k] = *(const LDS_AS u32x2*)(sl + 4096 + (d0 + k) * 512 + lane * 8);
    PHASE();
    hd[0] = __builtin_amdgcn_readfirstlane(hv.x); hd[1] = __builtin_amdgcn_readfirstlane(hv.y);
#pragma unroll
    for (int k = 0; k < 2; ++k) {
        L3[k] = *(const LDS_AS unsigned short*)(tb + ((hd[k] >> 7) & 0x3feu) + lane * 2);
        L2[k] = *(const LDS_AS unsigned*)(tb + (hd[k] & 0xffu) * 4 + lane * 4);
    }
    PHASE();
    unsigned c1 = 0;
#pragma unroll
    for (int k = 0; k < 2; ++k) {
        const unsigned kb = ((hd[k] >> 27) & 15u) * 0x080808u;
        const bool var = (hd[k] >> 31) != 0u;
        T1[k] = (rel8(0, 1, 2) + kb) & 0x787878u;
        T2[k] = ((var ? rel8(-1, -2, 3) : rel8(-1, -2, -3)) + kb) & 0x787878u;
        T3[k] = ((var ? rel8(-4, -5, -3) : rel8(-4, -5, 3)) + kb) & 0x787878u;
        E0[k] = esc_bits(L1[k].x); E1[k] = esc_bits(L1[k].y);
        c1 |= (__builtin_popcount(E0[k]) + __builtin_popcount(E1[k])) << (16 * k);
    }
    const unsigned x1 = wave_incl_scan(c1) - c1;
    PHASE();
#pragma unroll
    for (int k = 0; k < 2; ++k) { S3[k][0] = lut_sel(L3[k], 0); S3[k][1] = lut_sel(L3[k], 1); }
#pragma unroll
    for (int k = 0; k < 2; ++k)
#pragma unroll
        for (int j = 0; j < 4; ++j) S2[k][j] = lut_sel(L2[k], j);
    PHASE();
    E2[0] = esc_bits(L2[0]); E2[1] = esc_bits(L2[1]);
    const unsigned c2 = __builtin_popcount(E2[0]) | (__builtin_popcount(E2[1]) << 16);
    const unsigned x2 = wave_incl_scan(c2) - c2;
#pragma unroll
    for (int k = 0; k < 2; ++k) {
        w3[k][0] = 0; w3[k][1] = 0;
        if (hd[k] & 0x1fe0000u) {
            LDS_AS unsigned char* const sc = sc0 + k * CL_BYTES;
            const unsigned l4 = *(const LDS_AS unsigned short*)(tb + ((hd[k] >> 15) & 0x3fcu) + lane * 2);
            const unsigned E3 = esc_bits(L3[k]);
            const unsigned c3 = __builtin_popcount(E3);
            const unsigned p3 = wave_incl_scan(c3) - c3 + lds_addr(sc + CL_A4), p3b = p3 + __builtin_popcount(E3 & 0xffu);
            *(LDS_AS unsigned*)(sc + CL_A4 + lane * 4) = ((l4 << 3) & 0x7878u) | ((l4 << 15) & 0x78780000u);
            LDS_FENCE();
            const u32x2 wa = win_read(p3), wb = win_read(p3b);
            w3[k][0] = win_get(wa, p3);
            w3[k][1] = win_get(wb, p3b);
        }
    }
    unsigned q2[2][4];
#pragma unroll
    for (int k = 0; k < 2; ++k) {
        *(LDS_AS u32x2*)(sc0 + k * CL_BYTES + CL_A3 + lane * 8) = u32x2{vperm(w3[k][0], T3[k], S3[k][0]), vperm(w3[k][1], T3[k], S3[k][1])};
        const unsigned p2 = ((x2 >> (16 * k)) & 0xffffu) + lds_addr(sc0 + k * CL_BYTES + CL_A3);
#pragma unroll
        for (int j = 0; j < 4; ++j) q2[k][j] = j ? p2 + __builtin_popcount(E2[k] & ((1u << (8 * j)) - 1u)) : p2;
    }
    PHASE();
    u32x2 W2[2][4];
#pragma unroll
    for (int k = 0; k < 2; ++k)
#pragma unroll
        for (int j = 0; j < 4; ++j) W2[k][j] = win_read(q2[k][j]);
    PHASE();
    unsigned q1[2][8];
#pragma unroll
    for (int k = 0; k < 2; ++k) {
        unsigned p1 = (x1 >> (16 * k)) & 0xffffu;
        p1 = (p1 > (unsigned)CAP1 ? (unsigned)CAP1 : p1) + lds_addr(sc0 + k * CL_BYTES + CL_A2);
        const unsigned p1b = p1 + __builtin_popcount(E0[k]);
#pragma unroll
        for (int j = 0; j < 8; ++j) {
            const unsigned base = (j & 4) ? p1b : p1, E = (j & 4) ? E1[k] : E0[k];
            q1[k][j] = (j & 3) ? base + __builtin_popcount(E & ((1u << (8 * (j & 3))) - 1u)) : base;
        }
    }
    PHASE();
#pragma unroll
    for (int k = 0; k < 2; ++k)
        *(LDS_AS u32x4*)(sc0 + k * CL_BYTES + CL_A2 + lane * 16) =
            u32x4{vperm(win_get(W2[k][0], q2[k][0]), T2[k], S2[k][0]), vperm(win_get(W2[k][1], q2[k][1]), T2[k], S2[k][1]),
                  vperm(win_get(W2[k][2], q2[k][2]), T2[k], S2[k][2]), vperm(win_get(W2[k][3], q2[k][3]), T2[k], S2[k][3])};
    PHASE();
    u32x2 W1[2][8];
#pragma unroll
    for (int k = 0; k < 2; ++k)
#pragma unroll
        for (int j = 0; j < 8; ++j) { W1[k][j] = win_read(q1[k][j]); S1[k][j] = lut_sel((j & 4) ? L1[k].y : L1[k].x, j & 3); }
    PHASE();
#pragma unroll
    for (int k = 0; k < 2; ++k)
#pragma unroll
        for (int j = 0; j < 8; ++j) X[k][j] = vperm(win_get(W1[k][j], q1[k][j]), T1[k], S1[k][j]);
}
__device__ __forceinline__ void merge_part(const u32x4 N, const unsigned (&X)[8], u32x4& R0, u32x4& R1) {
    constexpr unsigned K = 0x87878787u;
    R0 = u32x4{(N.x & K) | X[0], (__builtin_amdgcn_alignbit(N.x, N.x, 4) & K) | X[1], (N.y & K) | X[2], (__builtin_amdgcn_alignbit(N.y, N.y, 4) & K) | X[3]};
    R1 = u32x4{(N.z & K) | X[4], (__builtin_amdgcn_alignbit(N.z, N.z, 4) & K) | X[5], (N.w & K) | X[6], (__builtin_amdgcn_alignbit(N.w, N.w, 4) & K) | X[7]};
}

constexpr int NSLOT = 8;
constexpr unsigned META_RAW = INFO_RAW << 24;
constexpr unsigned META_INBOX = 0x40000000u;
constexpr int CM_LUT = 0;
constexpr int CM_META = 1024;
constexpr int CM_DONE = CM_META + NSLOT * 16;
constexpr int CM_NEXT = CM_DONE + NSLOT * 4;
constexpr int CM_END = CM_META + 248;
constexpr int CM_LAND = CM_END + 4;
constexpr int CM_TAB = 2048;
constexpr int CM_SLOT = 4096;
constexpr int CM_SC = CM_SLOT + NSLOT * UNIT;
constexpr int ST_LUT = 0, ST_TOP = 2048, ST_SC = ST_TOP + 64, ST_TAIL = ST_SC + 8 * CL_BYTES;
static_assert(CM_SC + 14 * CL_BYTES <= LDS_BYTES && CM_NEXT + 4 <= CM_END && CM_LAND + 256 <= CM_TAB && ST_TAIL + 2048 <= LDS_BYTES, "LDS map");
static_assert(NPB + 2 < NSLOT && PBS >= 4 && NPB * PBS <= NBLK - 1 && WIN == 8, "roles");

__device__ __forceinline__ bool timed_out(unsigned& spins, unsigned* tmo, unsigned call, unsigned long long t0) {
    if ((++spins & 1023u) != 0u) return false;
    if (__hip_atomic_load(tmo, __ATOMIC_RELAXED, __HIP_MEMORY_SCOPE_AGENT) != call &&
        __builtin_amdgcn_s_memrealtime() - t0 <= DEADLINE_TICKS) return false;
    __hip_atomic_store(tmo, call, __ATOMIC_RELAXED, __HIP_MEMORY_SCOPE_AGENT);
    return true;
}

__device__ __forceinline__ bool stuck(unsigned& spins, unsigned long long t0) {
    return (++spins & 0xffffu) == 0u && __builtin_amdgcn_s_memrealtime() - t0 > LDS_DEADLINE_TICKS;
}

__global__ void __launch_bounds__(BLOCK, 2) ag_gemm(Params p) {
    extern __shared__ __attribute__((aligned(16))) unsigned char smem[];
    unsigned pf0, pf1;
    asm volatile("s_load_dword %0, %2, 0x0\n\ts_load_dword %1, %2, 0x40" : "=&s"(pf0), "=&s"(pf1) : "s"(__builtin_amdgcn_kernarg_segment_ptr()));
    __builtin_amdgcn_sched_barrier(0);
    const int lane = threadIdx.x & 63;
    const int wave = __builtin_amdgcn_readfirstlane(threadIdx.x >> 6);
    const int bid = blockIdx.x;
    unsigned char* const peer = p.ws[((bid >> 3) + 1) & 7];
    const unsigned su0 = (bid & 63) + 64u * ((0x21404300u >> (8 * (bid >> 6) + (wave & 4))) & 7u);
    unsigned soff = ((su0 >> 3) * 16 + (lane & 15)) * KD + (su0 & 7) * 512 + (wave & 3) * 128 + (lane >> 4) * 16;
    unsigned poff = (((wave >> 2) * PBS + PBS - 1) * 16 + (lane >> 3)) * KD + (bid & 7) * 512 + (wave & 3) * 128 + (lane & 7) * 16;
    unsigned poff2 = poff + 8 * KD, coff = CNT_OFF + bid * 128;
    asm volatile("" : "+v"(soff), "+v"(poff), "+v"(poff2), "+v"(coff));
    asm volatile("s_waitcnt lgkmcnt(0)" ::"s"(pf0), "s"(pf1), "s"(p.x), "s"(p.w), "s"(p.sx), "s"(p.sw), "s"(p.out), "s"(p.rank), "s"(peer), "s"(p.ws[0]));
    unsigned char* const myws = p.ws[0];
    unsigned* const cntp = (unsigned*)(myws + CNT_OFF + (size_t)bid * 128);
    unsigned* const tmo = (unsigned*)(myws + TMO_OFF);
    LDS_AS unsigned char* const lds = (LDS_AS unsigned char*)smem;
    unsigned call;
#define LOAD_COUNT() unsigned cv; asm volatile("global_load_dword %0, %1, %2 sc1" : "=&v"(cv) : "v"(coff), "s"(myws) : "memory")

    if (__builtin_expect(bid >= NCOMM, 1)) {
      if (bid < NCOMM + 56) {
        LOAD_COUNT();
        const int c_k = bid & 7, i0 = wave >> 2;
        u32x4 A[PUSH_ROUNDS][2];
#pragma unroll
        for (int i = 0; i < PUSH_ROUNDS; ++i) {
            const unsigned char* s = p.x + (i0 + 2 * i < NPB ? (size_t)i * (2 * PBS * 16 * KD) : 0);
            ld16<0>(A[i][0], s, poff); ld16<0>(A[i][1], s, poff2);
            if (i == 0) __builtin_amdgcn_s_barrier();
        }
        unsigned char* dst = peer + NI_OFF + ((size_t)p.rank * 128 + i0 * 8 + c_k) * 4096 + (wave & 3) * 1024 + lane * 16;
        asm volatile("s_waitcnt vmcnt(%4)" : "+v"(cv), "+v"(A[0][0]), "+v"(A[0][1]), "+v"(dst) : "i"(2 * PUSH_ROUNDS - 2) : "memory");
        call = __builtin_amdgcn_readfirstlane(cv) + 1u;
        dst += (size_t)(call & 1u) * 64 * 4096;
        st16_sys<0>(dst, nibble_plane(A[0][0], A[0][1]));
#pragma unroll
        for (int i = 1; i < PUSH_ROUNDS; ++i) {
            wait_vm<0>();
            asm volatile("" : "+v"(A[i][0]), "+v"(A[i][1]));
            if (i0 + 2 * i < NPB) st16_sys<0>(dst + (size_t)i * (16 * 4096), nibble_plane(A[i][0], A[i][1]));
        }
        wait_vm<0>();
        __builtin_amdgcn_s_barrier();
        if (wave == 0)
            asm volatile("s_mov_b64 exec, 1\n\tglobal_store_dwordx2 %0, %1, off sc0 sc1\n\ts_mov_b64 exec, -1"
                         ::"v"(peer + RDY_OFF + (((size_t)(call & 1u) * NR + p.rank) * NRDY + NUNIT + (bid & 7)) * 128), "v"(u32x2{1u, call}) : "memory");
      }
      {
        LOAD_COUNT();
        const unsigned d = wave & 3;
        const bool stage = su0 < NUNIT;
        const unsigned su = stage ? su0 : NUNIT - 1;
        u32x4 R0, R1;
        if (stage) { ld16<0>(R0, p.x, soff); ld16<64>(R1, p.x, soff); }
        unsigned char* wsl;
        asm volatile("global_load_dwordx2 %0, %1, off" : "=&v"(wsl)
                     : "v"((const unsigned char*)__builtin_amdgcn_kernarg_segment_ptr() + offsetof(Params, ws) + (lane < 8 ? lane * 8 : 0)) : "memory");
        LDS_AS unsigned char* const sc = lds + ST_SC + wave * CL_BYTES;
        LDS_AS unsigned char* const tb = lds + ST_TAIL + (wave >> 2) * 1024;
        LDS_AS unsigned* const top = (LDS_AS unsigned*)(lds + ST_TOP) + (wave >> 2);
        encode_clear(sc, lane);
        *top = 16u;
        if (threadIdx.x >= 256) *(LDS_AS u32x2*)(lds + ST_LUT + (threadIdx.x - 256) * 8) = lut_entry(threadIdx.x - 256, true);
        wait_lgkm();
        __builtin_amdgcn_s_barrier();
        asm volatile("s_waitcnt vmcnt(1)" : "+v"(cv), "+v"(R0), "+v"(R1) :: "memory");
        call = __builtin_amdgcn_readfirstlane(cv) + 1u;
        unsigned char* const cp = myws + C_OFF + (size_t)(call & 1u) * SHARD + (size_t)su * UNIT;
        unsigned char* const x = myws + X_OFF + ((size_t)(call & 1u) * NR + p.rank) * SHARD + (size_t)su * UNIT + d * 2048 + lane * 16;
        bool rawdone = false, packed = false;
        if (stage && su < NUNIT - RAW_UNITS)
            packed = encode_part(R0, R1, d, sc, tb, top, lds + ST_LUT, lane, [&](const Packed& q) {
                st16_sys<0>(cp + d * 1024 + lane * 16, q.N);
                asm volatile("global_store_dwordx2 %0, %1, off sc0 sc1" ::"v"(cp + 4096 + d * 512 + lane * 8), "v"(q.L1) : "memory");
                st16_sys<0>(x, R0); st16_sys<1024>(x, R1);
                rawdone = true;
            });
        if (stage && !rawdone) { st16_sys<0>(x, R0); st16_sys<1024>(x, R1); }
        if (!packed) lds_or<0>(lds_addr(top), 0x10000u);
        asm volatile("s_waitcnt vmcnt(0) lgkmcnt(0)" : "+v"(wsl) :: "memory");
        __builtin_amdgcn_s_barrier();
        LDS_FENCE();
        const unsigned tv = *top;
        const u32x4 tl = *(const LDS_AS u32x4*)(tb + lane * 16);
        if (d == 0 && packed) st16_sys<0>(cp + 6144 + lane * 16, tl);
        const unsigned total = __builtin_amdgcn_readfirstlane(tv);
        unsigned char* fp = wsl + (lane < 8 ? RDY_OFF + ((size_t)(call & 1u) * NR + p.rank) * NRDY * 128 : FLAG_OFF + (size_t)p.rank * NUNIT * 128 + (lane & 1) * 8) + (size_t)su * 128;
        u32x2 fv = u32x2{lane >= 8 ? call : (total <= (unsigned)TAIL_MAX ? (total + 15u) >> 4 : INFO_RAW), call};
        asm volatile("s_waitcnt vmcnt(0)" : "+v"(fp), "+v"(fv) :: "memory");
        if (d == 0 && stage)
            asm volatile("s_mov_b64 exec, 0x3fe\n\tglobal_store_dwordx2 %0, %1, off sc0 sc1\n\ts_mov_b64 exec, -1" ::"v"(fp), "v"(fv) : "memory");
        __builtin_amdgcn_s_barrier();
      }
    } else {
        const int c_src = bid >> 3, c_k = bid & 7;
        const int src_rank = (p.rank + c_src + 1) & 7;
        if (wave == 1) {
            const unsigned k = lane & 7;
            const unsigned char* tp = k == 0 ? peer + NI_OFF + (size_t)p.rank * 128 * 4096 : k == 1 ? peer + RDY_OFF
                                    : k == 2 ? peer + C_OFF : k == 3 ? peer + X_OFF + (size_t)src_rank * SHARD + (SHARD - 128)
                                    : k == 4 ? myws + X_OFF + (size_t)src_rank * SHARD : k == 5 ? (const unsigned char*)p.out + (size_t)src_rank * (MSH * NSH * 4)
                                    : k == 6 ? p.w : (const unsigned char*)p.sx;
            if (lane >= 8) tp = k == 2 ? tp + SHARD : (k == 3 || k == 4) ? tp + NR * SHARD : k == 0 ? myws + NI_OFF + (size_t)src_rank * 128 * 4096 : k == 7 ? (const unsigned char*)p.sw : tp;
            asm volatile("s_mov_b32 m0, %1\n\ts_mov_b64 exec, 0xffff\n\tglobal_load_lds_dword %0, off sc0 sc1\n\ts_mov_b64 exec, -1"
                         ::"v"(tp), "s"(lds_addr(lds + CM_SC)) : "memory");
        }
        if (c_src == 7 || lds_addr(lds + CM_LUT) != 0u) return;
        LOAD_COUNT();
        if (threadIdx.x < 256) *(LDS_AS unsigned*)(lds + CM_LUT + threadIdx.x * 4) = lut_entry(threadIdx.x, false).x;
        else *(LDS_AS u32x2*)(lds + CM_TAB + (threadIdx.x - 256) * 8) = u32x2{0u, 0u};
        if (wave == 0) *(LDS_AS unsigned*)(lds + CM_META + lane * 4) = 0u;
        asm volatile("s_waitcnt vmcnt(0) lgkmcnt(0)" : "+v"(cv) :: "memory");
        __builtin_amdgcn_s_barrier();
        call = __builtin_amdgcn_readfirstlane(cv) + 1u;
        const size_t xo = X_OFF + ((size_t)(call & 1u) * NR + src_rank) * SHARD;
        LDS_AS unsigned char* const tab = lds + CM_TAB;
        const unsigned long long t0 = __builtin_amdgcn_s_memrealtime();
        unsigned spins = 0;
        if (wave == 0) {
            const unsigned char* const cmpp = peer + C_OFF + (size_t)(call & 1u) * SHARD + lane * 16;
            const size_t rawd = xo - (C_OFF + (size_t)(call & 1u) * SHARD);
            unsigned char* const tick = myws + TICK_OFF + c_src * 128;
            const unsigned tbase = (call - 1u) * (NDYN - 8) - 16u;
            unsigned q = 0;
            unsigned n = 16;
            unsigned lv = n - (WIN - 1);
            const unsigned landp = lds_addr(lds + CM_LAND) + lane * 4;
#define STEP "s_waitcnt vmcnt(%[w])\n\tv_add_u32 %[lv], 1, %[lv]\n\tds_write_b32 %[lp], %[lv]\n\t"
            unsigned tabv = 0;
            auto ready = [&](unsigned u) -> unsigned {
                unsigned info = ((unsigned)__builtin_amdgcn_readlane(tabv, u >> 2) >> (8 * (u & 3u))) & 0xffu;
                while (__builtin_expect(info == 0u, 0)) {
                    tabv = __hip_atomic_load((LDS_AS unsigned*)tab + lane, __ATOMIC_ACQUIRE, __HIP_MEMORY_SCOPE_WORKGROUP);
                    info = ((unsigned)__builtin_amdgcn_readlane(tabv, u >> 2) >> (8 * (u & 3u))) & 0xffu;
                    if (stuck(spins, t0)) info = INFO_FAIL;
                }
                return info;
            };
            bool ok = true;
            auto slot_free = [&](unsigned seen) {
                if (__builtin_expect((unsigned)__builtin_amdgcn_readfirstlane(seen) != 2u * (q / NSLOT), 0))
                    while (ok && *(volatile LDS_AS unsigned*)(lds + CM_DONE + (q & (NSLOT - 1)) * 4) != 2u * (q / NSLOT)) {
                        wait_vm<0>();
                        *(volatile LDS_AS unsigned*)(size_t)landp = n;
                        ok = !stuck(spins, t0);
                    }
            };
            auto first_half = [&](unsigned u, unsigned info, unsigned mark) -> const unsigned char* {
                const unsigned slot = q & (NSLOT - 1);
                const unsigned char* g = cmpp + ((size_t)u * UNIT + ((info & INFO_RAW) ? rawd : 0));
                const unsigned la = lds_addr(lds + CM_SLOT) + slot * UNIT + 4096;
                asm volatile("s_mov_b32 m0, %[la]\n\ts_mov_b64 exec, %[mk]\n\t"
                             "global_load_lds_dwordx4 %[g], off offset:2048 sc0 sc1\n\t"
                             "s_mov_b64 exec, 1\n\tds_write_b128 %[ma], %[mv]\n\ts_mov_b64 exec, -1\n\t" STEP
                             "global_load_lds_dwordx4 %[g], off sc0 sc1\n\t" STEP
                             "global_load_lds_dwordx4 %[g], off offset:1024 sc0 sc1\n\t" STEP
                             : [lv] "+&v"(lv)
                             : [g] "v"(g + 4096), [la] "s"(la), [mk] "s"(info >= 64u ? ~0ull : (1ull << info) - 1ull), [lp] "v"(landp), [w] "i"(WIN - 1),
                               [ma] "v"(lds_addr(lds + CM_META) + slot * 16), [mv] "v"(u32x4{q + 1u, u | mark | ((info & INFO_RAW) << 24), n, 0u})
                             : "memory");
                n += 3;
                if (__builtin_expect((info & INFO_RAW) != 0u, 0)) {
                    asm volatile("s_mov_b32 m0, %[la]\n\ts_nop 0\n\tglobal_load_lds_dwordx4 %[g], off offset:3072 sc0 sc1\n\t" STEP
                                 : [lv] "+&v"(lv) : [g] "v"(g + 4096), [la] "s"(la), [lp] "v"(landp), [w] "i"(WIN - 1) : "memory");
                    ++n;
                }
                ++q;
                return g;
            };
            auto second_half = [&](unsigned qq, const unsigned char* g) {
                const unsigned slot = qq & (NSLOT - 1);
                asm volatile("s_mov_b32 m0, %[la]\n\ts_nop 0\n\t"
                             "global_load_lds_dwordx4 %[g], off sc0 sc1\n\t"
                             "s_mov_b64 exec, 1\n\tds_write_b32 %[ma], %[mv]\n\ts_mov_b64 exec, -1\n\t" STEP
                             "global_load_lds_dwordx4 %[g], off offset:1024 sc0 sc1\n\t" STEP
                             "global_load_lds_dwordx4 %[g], off offset:2048 sc0 sc1\n\t" STEP
                             "global_load_lds_dwordx4 %[g], off offset:3072 sc0 sc1\n\t" STEP
                             : [lv] "+&v"(lv)
                             : [g] "v"(g), [la] "s"(lds_addr(lds + CM_SLOT) + slot * UNIT), [lp] "v"(landp), [w] "i"(WIN - 1),
                               [ma] "v"(lds_addr(lds + CM_META) + slot * 16 + 12), [mv] "v"(n)
                             : "memory");
                n += 4;
            };
            unsigned up = (PBS - 1) * 8 + c_k;
            auto pushed_below = [&](unsigned u) {
                while (ok && up < u) {
                    const unsigned info = ready(up);
                    unsigned pl;
                    while ((pl = __builtin_amdgcn_readfirstlane(__hip_atomic_load(tab + NUNIT + c_k, __ATOMIC_ACQUIRE, __HIP_MEMORY_SCOPE_WORKGROUP))) == 0u &&
                           !stuck(spins, t0)) {}
                    ok = info != INFO_FAIL && pl == 1u;
                    slot_free(~0u);
                    if (ok) {
                        const unsigned char* g = first_half(up, info, META_INBOX | ((up / (8 * PBS)) << 8));
                        if (info & INFO_RAW) second_half(q - 1u, g);
                        up = up + 8 * PBS < (unsigned)(NPB * PBS * 8) ? up + 8 * PBS : ~0u;
                    }
                }
            };
            unsigned info = ready(c_k);
            if (info != INFO_FAIL) {
                const unsigned char* gcur = first_half(c_k, info, 0u);
                unsigned qcur = 0, k = c_k + 8u, seen = 0;
                while (k < (unsigned)NDYN) {
                    const unsigned kb = (k >> 3) / (PBS - 1);
                    const unsigned u = k + 8u * (kb < (unsigned)NPB ? kb : NPB);
                    if (__builtin_expect(up < u, 0)) { pushed_below(u); seen = ~0u; }
                    info = ready(u);
                    slot_free(seen);
                    if (__builtin_expect(!ok || info == INFO_FAIL, 0)) break;
                    unsigned tv;
                    ticket(tv, tick); ++n; ++lv;
                    const unsigned char* gnext = first_half(u, info, 0u);
                    seen = *(volatile LDS_AS unsigned*)(lds + CM_DONE + (q & (NSLOT - 1)) * 4);
                    second_half(qcur, gcur);
                    asm volatile("" : "+v"(tv));
                    k = __builtin_amdgcn_readfirstlane(tv) - tbase;
                    gcur = gnext; qcur = q - 1u;
                }
                second_half(qcur, gcur);
                pushed_below(NUNIT);
            }
            if (lane == 0) *(volatile LDS_AS unsigned*)(lds + CM_END) = q + 1u;
#define LANDED(k) do { wait_vm<k>(); *(volatile LDS_AS unsigned*)(size_t)landp = n - (k); } while (0)
            LANDED(6); LANDED(5); LANDED(4); LANDED(3); LANDED(2); LANDED(1); LANDED(0);
            return;
        }
        if (wave == 7) {
            flag_store((unsigned char*)cntp, call);
            const unsigned char* rp = myws + RDY_OFF + ((size_t)(call & 1u) * NR + src_rank) * NRDY * 128;
            for (int u0 = -56; u0 < NUNIT; u0 += 64) {
                const int u = u0 < 0 ? c_k + (lane & 1) * 8 : (u0 + lane < 16 ? NUNIT + c_k : (u0 + lane < NUNIT ? u0 + lane : NUNIT - 1));
                while (true) {
                    u32x2 r;
                    asm volatile("global_load_dwordx2 %0, %1, off sc0 sc1\n\ts_waitcnt vmcnt(0)" : "=&v"(r) : "v"(rp + (size_t)u * 128) : "memory");
                    const unsigned t = r.y >= call ? r.x : 0u;
                    __hip_atomic_store(tab + u, (unsigned char)t, __ATOMIC_RELEASE, __HIP_MEMORY_SCOPE_WORKGROUP);
                    if (__all(t != 0u)) break;
                    if (timed_out(spins, tmo, call, t0)) {
                        *(volatile LDS_AS u32x2*)(lds + CM_TAB + lane * 8) = u32x2{~0u, ~0u};
                        u0 = NUNIT;
                        break;
                    }
                }
            }
        }
        LDS_AS unsigned char* const sc = lds + CM_SC + (wave - 1) * (2 * CL_BYTES);
        unsigned char* const dst = myws + xo + lane * 16;
        unsigned char* const fl = myws + FLAG_OFF + (size_t)src_rank * NUNIT * 128;
        const unsigned char* const inbox = myws + NI_OFF + (((size_t)src_rank * 2 + (call & 1u)) * 64 + c_k) * 4096 + lane * 16;
        while (true) {
            const unsigned r = lds_ticket(lds + CM_NEXT);
            const unsigned q = r >> 1, h = r & 1u;
            const unsigned slot = q & (NSLOT - 1);
            const LDS_AS unsigned char* const sl = lds + CM_SLOT + slot * UNIT;
            unsigned uw;
            bool end = false;
            while (true) {
                const u32x4 m = uniform4(*(volatile LDS_AS u32x4*)(lds + CM_META + slot * 16));
                const u32x2 lv = *(volatile LDS_AS u32x2*)(lds + CM_END);
                const unsigned land = __builtin_amdgcn_readfirstlane(lv.y), e = __builtin_amdgcn_readfirstlane(lv.x);
                uw = m.y;
                if (m.x == q + 1u) {
                    if (land >= m.z + ((uw & META_RAW) ? 4u * h : 2u + h)) break;
                } else if (e - 1u <= q) { end = true; break; }
                if (stuck(spins, t0)) return;
                __builtin_amdgcn_s_sleep(1);
            }
            if (end) break;
            LDS_FENCE();
            const bool raw = (uw & META_RAW) != 0u;
            const bool inb = !raw && (uw & META_INBOX) != 0u;
            if (inb) {
                const unsigned char* g = inbox + (size_t)((uw >> 8) & 7u) * (8 * 4096) + h * 2048;
                glds_sys<0>(g, lds_addr(sl) + h * 2048); glds_sys<1024>(g, lds_addr(sl) + h * 2048);
            }
            unsigned X[2][8];
            if (!raw) {
                decode_exps(sl, 2u * h, X, sc, lane);
#pragma unroll
                for (int k = 0; k < 2; ++k)
                    asm volatile("" : "+v"(X[k][0]), "+v"(X[k][1]), "+v"(X[k][2]), "+v"(X[k][3]), "+v"(X[k][4]), "+v"(X[k][5]), "+v"(X[k][6]), "+v"(X[k][7]));
            }
            if (inb) wait_vm<0>();
            else if (!raw || h == 0u) {
                while (true) {
                    const unsigned ib = __builtin_amdgcn_readfirstlane(*(volatile LDS_AS unsigned*)(lds + CM_META + slot * 16 + 12));
                    const unsigned land = __builtin_amdgcn_readfirstlane(*(volatile LDS_AS unsigned*)(lds + CM_LAND));
                    if (ib != 0u && land >= ib + (raw ? 4u : 2u * h + 2u)) break;
                    if (stuck(spins, t0)) return;
                }
                LDS_FENCE();
            }
            u32x4 R[4];
            if (raw) {
#pragma unroll
                for (int i = 0; i < 4; ++i) R[i] = *(const LDS_AS u32x4*)(sl + h * 4096 + i * 1024 + lane * 16);
            } else {
                const LDS_AS unsigned char* const np = sl + h * 2048 + (inb ? (lane & 7) * 128 + (lane >> 4) * 16 + (lane & 8) : lane * 16);
                const LDS_AS unsigned char* const nq = np + (inb ? 64 : 8);
                const u32x2 a0 = *(const LDS_AS u32x2*)np, b0 = *(const LDS_AS u32x2*)nq, a1 = *(const LDS_AS u32x2*)(np + 1024), b1 = *(const LDS_AS u32x2*)(nq + 1024);
                const u32x4 N0 = u32x4{a0.x, a0.y, b0.x, b0.y}, N1 = u32x4{a1.x, a1.y, b1.x, b1.y};
                merge_part(N0, X[0], R[0], R[1]);
                merge_part(N1, X[1], R[2], R[3]);
            }
            const unsigned u = uw & (NUNIT - 1);
            unsigned char* const o = dst + (size_t)u * UNIT + h * 4096;
            st16_sys<0>(o, R[0]); st16_sys<1024>(o, R[1]); st16_sys<2048>(o, R[2]); st16_sys<3072>(o, R[3]);
            lds_inc(lds + CM_DONE + slot * 4);
            wait_vm<0>();
            flag_store(fl + (size_t)u * 128 + h * 8, call);
        }
        return;
    }

    const int cn = bid - NCOMM;
    const int cg = (cn >> 3) & 7;
    const int src_rank = (p.rank + cn + 3 * (cn >> 6)) & 7;
    if (wave == 7) flag_store((unsigned char*)cntp, call);

    i32x8 Wf[4][4];
    LDS_AS unsigned char* const wl = lds + wave * 16384;
    {
        const unsigned char* wsrc = p.w + (size_t)(wave * 512 + (lane >> 2)) * NSH + cg * 64 + (lane & 3) * 16;
        const int g = lane >> 4, q = (lane & 15) >> 1, h = lane & 1;
        LDS_AS unsigned char* const rd = wl + (16 * g + q) * 64 + h * 8;
#pragma unroll
        for (int half = 0; half < 2; ++half) {
#pragma unroll
            for (int i = 0; i < 16; ++i)
                __builtin_amdgcn_global_load_lds((const GLB_AS unsigned*)(wsrc + (size_t)(half * 256 + i * 16) * NSH),
                                                 (LDS_AS unsigned*)(wl + i * 1024), 16, 0, 0);
            wait_vm<0>();
#pragma unroll
            for (int s2 = 0; s2 < 2; ++s2)
#pragma unroll
                for (int t = 0; t < 4; ++t)
#pragma unroll
                    for (int pc = 0; pc < 4; ++pc) {
                        i32x2 v = __builtin_amdgcn_ds_read_tr8_b64_v2i32(
                            (LDS_AS i32x2*)(rd + (s2 * 128 + (pc >> 1) * 64 + (pc & 1) * 8) * 64 + t * 16));
                        Wf[half * 2 + s2][t][pc * 2] = v.x;
                        Wf[half * 2 + s2][t][pc * 2 + 1] = v.y;
                    }
            wait_lgkm();
        }
    }
    const float scale = p.sx[0] * p.sw[0];
    const unsigned char* const xl = myws + X_OFF + ((size_t)(call & 1u) * NR + src_rank) * SHARD + wave * UNIT + 4096 + lane * 16;
    const unsigned* const flags = (const unsigned*)(myws + FLAG_OFF + ((size_t)src_rank * NUNIT + wave) * 128 + (lane & 1) * 8);
    LDS_AS unsigned char* const pw = lds + wave * PART_BYTES + (lane & 15) * PART_STRIDE + (lane >> 4) * 16;
    const LDS_AS unsigned char* const pr = lds + (wave * 4 + (lane >> 4)) * PART_STRIDE + (lane & 15) * 16;
    float* const outp = p.out + (size_t)(src_rank * MSH + wave * 4 + (lane >> 4)) * NSH + cg * 64 + (lane & 15) * 4;
    __builtin_amdgcn_s_barrier();
    const unsigned long long t0 = __builtin_amdgcn_s_memrealtime();
    int buf = 0;
    bool bad = false;
    for (int b = 2 - (cn >> 6); b < NBLK; b += 3) {
        const unsigned* f = flags + (size_t)b * (8 * 32);
        unsigned spins = 0;
        while (!bad && !__all(__hip_atomic_load(f, __ATOMIC_RELAXED, __HIP_MEMORY_SCOPE_SYSTEM) >= call)) {
            bad = timed_out(spins, tmo, call, t0);
            __builtin_amdgcn_s_sleep(1);
        }
        u32x4 R[8];
        asm volatile(
            "global_load_dwordx4 %0, %8, off offset:-4096 sc0 sc1\n\tglobal_load_dwordx4 %1, %8, off offset:-3072 sc0 sc1\n\t"
            "global_load_dwordx4 %2, %8, off offset:-2048 sc0 sc1\n\tglobal_load_dwordx4 %3, %8, off offset:-1024 sc0 sc1\n\t"
            "global_load_dwordx4 %4, %8, off sc0 sc1\n\tglobal_load_dwordx4 %5, %8, off offset:1024 sc0 sc1\n\t"
            "global_load_dwordx4 %6, %8, off offset:2048 sc0 sc1\n\tglobal_load_dwordx4 %7, %8, off offset:3072 sc0 sc1\n\t"
            "s_waitcnt vmcnt(6)"
            : "=&v"(R[0]), "=&v"(R[1]), "=&v"(R[2]), "=&v"(R[3]), "=&v"(R[4]), "=&v"(R[5]), "=&v"(R[6]), "=&v"(R[7])
            : "v"(xl + (size_t)b * (8 * UNIT)) : "memory");
        const float c0 = bad ? __builtin_nanf("") : 0.0f;
        f32x4 acc[4];
#pragma unroll
        for (int t = 0; t < 4; ++t) acc[t] = f32x4{c0, c0, c0, c0};
#pragma unroll
        for (int s = 0; s < 4; ++s) {
            if (s > 0) {
                __builtin_amdgcn_sched_barrier(0);
                if (s == 1) asm volatile("s_waitcnt vmcnt(4)" : "+v"(R[2]), "+v"(R[3]) :: "memory");
                if (s == 2) asm volatile("s_waitcnt vmcnt(2)" : "+v"(R[4]), "+v"(R[5]) :: "memory");
                if (s == 3) asm volatile("s_waitcnt vmcnt(0)" : "+v"(R[6]), "+v"(R[7]) :: "memory");
            }
            i32x8 xf;
            xf[0] = R[2 * s].x; xf[1] = R[2 * s].y; xf[2] = R[2 * s].z; xf[3] = R[2 * s].w;
            xf[4] = R[2 * s + 1].x; xf[5] = R[2 * s + 1].y; xf[6] = R[2 * s + 1].z; xf[7] = R[2 * s + 1].w;
#pragma unroll
            for (int t = 0; t < 4; ++t)
                acc[t] = __builtin_amdgcn_mfma_scale_f32_16x16x128_f8f6f4(Wf[s][t], xf, acc[t], 1, 0, 0, 0, 0, 0);
        }
        __builtin_amdgcn_sched_barrier(0);
#pragma unroll
        for (int t = 0; t < 4; ++t) *(LDS_AS f32x4*)(pw + buf * (8 * PART_BYTES) + t * 64) = acc[t];
        wait_lgkm();
        __builtin_amdgcn_s_barrier();
        LDS_FENCE();
        if (wave < 4) {
            f32x4 v[8];
#pragma unroll
            for (int w2 = 0; w2 < 8; ++w2) v[w2] = *(const LDS_AS f32x4*)(pr + buf * (8 * PART_BYTES) + w2 * PART_BYTES);
            __builtin_amdgcn_sched_barrier(0);
            const f32x4 sum = (((v[0] + v[1]) + (v[2] + v[3])) + ((v[4] + v[5]) + (v[6] + v[7]))) * scale;
            st16_sys<0>((unsigned char*)(outp + (size_t)b * (16 * NSH)), __builtin_bit_cast(u32x4, sum));
        }
        buf ^= 1;
    }
}

extern "C" void kernel_launch(int rank, int nranks, ncclComm_t comm, hipStream_t stream,
                              void* const* d_in, const int* in_sizes, int n_in,
                              void* d_out, int out_size,
                              const int* in_dtypes, int out_dtype,
                              void* const* d_out_all,
                              void* const* d_ws_all, size_t ws_bytes) {
    (void)comm; (void)in_dtypes; (void)out_dtype; (void)d_out_all;
    if (nranks != NR || (unsigned)rank >= (unsigned)NR || n_in != 4 || ws_bytes < WS_NEED ||
        in_sizes[0] != (int)SHARD || in_sizes[1] != KD * NSH || out_size != NR * MSH * NSH) {
        fprintf(stderr, "unexpected problem shape\n");
        return;
    }
    static bool init = false;
    if (!init) {
        HIP_CHECK(hipFuncSetAttribute((const void*)ag_gemm, hipFuncAttributeMaxDynamicSharedMemorySize, LDS_BYTES));
        init = true;
    }
    Params p;
    p.x = (const unsigned char*)d_in[0];
    p.w = (const unsigned char*)d_in[1];
    p.sx = (const float*)d_in[2];
    p.sw = (const float*)d_in[3];
    p.out = (float*)d_out;
    for (int d = 0; d < NR; ++d) p.ws[d] = (unsigned char*)d_ws_all[(rank + d) % NR];
    p.rank = rank;
    p.pad = 0;
    hipLaunchKernelGGL(ag_gemm, dim3(GRID), dim3(BLOCK), LDS_BYTES, stream, p);
    HIP_CHECK(hipGetLastError());
}
```

```cpp
#include <hip/hip_runtime.h>
#include <rccl/rccl.h>
#include <cstddef>
#include <cstdio>

#define HIP_CHECK(expr) do { hipError_t _e = (expr); if (_e != hipSuccess) { \
    fprintf(stderr, "HIP error %s at %s:%d\n", hipGetErrorString(_e), __FILE__, __LINE__); \
    return; } } while (0)

typedef unsigned u32x2 __attribute__((ext_vector_type(2)));
typedef unsigned u32x4 __attribute__((ext_vector_type(4)));
typedef int i32x2 __attribute__((ext_vector_type(2)));
typedef int i32x8 __attribute__((ext_vector_type(8)));
typedef float f32x4 __attribute__((ext_vector_type(4)));
#define LDS_AS __attribute__((address_space(3)))
#define GLB_AS __attribute__((address_space(1)))

constexpr int NR = 8;
constexpr int MSH = 512, KD = 4096, NSH = 512;
constexpr size_t SHARD = (size_t)MSH * KD;
constexpr int NBLK = MSH / 16;
constexpr int NUNIT = NBLK * 8;
constexpr int UNIT = 8192;
constexpr int GRID = 256, BLOCK = 512;
constexpr int NCOMM = 64;
constexpr int NPB = 5, PB0 = 7, PBS = 4;
constexpr int PUSH_ROUNDS = (NPB + 1) / 2;
constexpr int NDYN = NUNIT - 8 * NPB;
constexpr int NRDY = NUNIT + 8;
constexpr int RAW_UNITS = 8;
constexpr unsigned HOLD_TICKS = 280;
constexpr int WIN = 8;
constexpr size_t X_OFF = 0;
constexpr size_t C_OFF = X_OFF + 2 * NR * SHARD;
constexpr size_t NI_OFF = C_OFF + 2 * SHARD;
constexpr size_t RDY_OFF = NI_OFF + (size_t)NR * 2 * 64 * 4096;
constexpr size_t FLAG_OFF = RDY_OFF + (size_t)2 * NR * NRDY * 128;
constexpr size_t CNT_OFF = FLAG_OFF + (size_t)NR * NUNIT * 128;
constexpr size_t TICK_OFF = CNT_OFF + (size_t)GRID * 128;
constexpr size_t TMO_OFF = TICK_OFF + NR * 128;
constexpr size_t WS_NEED = TMO_OFF + 128;
constexpr unsigned long long DEADLINE_TICKS = 100000000ull;
constexpr unsigned long long LDS_DEADLINE_TICKS = 150000000ull;
constexpr int LDS_BYTES = 8 * 16384;
constexpr int PART_STRIDE = 272, PART_BYTES = 16 * PART_STRIDE;

struct Params {
    const unsigned char* x;
    const unsigned char* w;
    const float* sx;
    const float* sw;
    float* out;
    unsigned char* ws[NR];
    int rank;
    int pad;
};

#define LDS_FENCE() asm volatile("" ::: "memory")
template <int N> __device__ __forceinline__ void wait_vm() { asm volatile("s_waitcnt vmcnt(%0)" ::"i"(N) : "memory"); }
__device__ __forceinline__ void wait_lgkm() { asm volatile("s_waitcnt lgkmcnt(0)" ::: "memory"); }
__device__ __forceinline__ unsigned lds_addr(const LDS_AS void* p) { return (unsigned)(size_t)p; }

template <int OFF> __device__ __forceinline__ void ld16(u32x4& v, const unsigned char* base, unsigned off) {
    asm volatile("global_load_dwordx4 %0, %1, %2 offset:%3" : "=&v"(v) : "v"(off), "s"(base), "i"(OFF) : "memory");
}
template <int OFF> __device__ __forceinline__ void st16_sys(unsigned char* p, u32x4 v) {
    asm volatile("global_store_dwordx4 %0, %1, off offset:%2 sc0 sc1\n\ts_nop 1" ::"v"(p), "v"(v), "i"(OFF) : "memory");
}
__device__ __forceinline__ void flag_store(unsigned char* p, unsigned v) {
    asm volatile("s_mov_b64 exec, 1\n\tglobal_store_dword %0, %1, off sc0 sc1\n\ts_mov_b64 exec, -1" ::"v"(p), "v"(v) : "memory");
}
__device__ __forceinline__ void ticket(unsigned& v, unsigned char* p) {
    asm volatile("s_mov_b64 exec, 1\n\tglobal_atomic_add %0, %1, %2, off sc0\n\ts_mov_b64 exec, -1" : "=&v"(v) : "v"(p), "v"(1u) : "memory");
}
__device__ __forceinline__ void lds_inc(LDS_AS unsigned char* p) {
    asm volatile("s_mov_b64 exec, 1\n\tds_add_u32 %0, %1\n\ts_mov_b64 exec, -1" ::"v"(lds_addr(p)), "v"(1u) : "memory");
}
__device__ __forceinline__ unsigned lds_ticket(LDS_AS unsigned char* p) {
    unsigned v;
    asm volatile("s_mov_b64 exec, 1\n\tds_add_rtn_u32 %0, %1, %2\n\ts_mov_b64 exec, -1\n\ts_waitcnt lgkmcnt(0)"
                 : "=&v"(v) : "v"(lds_addr(p)), "v"(1u) : "memory");
    return __builtin_amdgcn_readfirstlane(v);
}
template <int OFF> __device__ __forceinline__ void glds_sys(const unsigned char* g, unsigned lds) {
    asm volatile("s_mov_b32 m0, %1\n\ts_nop 0\n\tglobal_load_lds_dwordx4 %0, off offset:%2 sc0 sc1" ::"v"(g), "s"(lds), "i"(OFF) : "memory");
}

constexpr int CAP1 = 1024, CAP2 = 512, CAP3 = 256, TAIL_MAX = 1024;
constexpr int CL_A2 = 0;
constexpr int CL_A3 = CAP1 + 16;
constexpr int CL_A4 = CL_A3 + CAP1 + 16;
constexpr int CL_BYTES = CL_A4 + CAP2 + 32;
constexpr unsigned INFO_RAW = 0x80u, INFO_FAIL = 0xffu;
constexpr unsigned rel8(int a, int b, int c) { return (unsigned)(((a & 15) << 3) | ((b & 15) << 11) | ((c & 15) << 19)); }
constexpr unsigned map_byte(int r, bool var) {
    return r <= 2 ? (unsigned)r << 3 : r == 15 ? 0x18u : r == 14 ? 0x19u : r == (var ? 3 : 13) ? 0x1au
         : r == 12 ? 0x1bu : r == 11 ? 0x1fu : r == (var ? 13 : 3) ? 0x9bu : 0x9fu;
}
constexpr unsigned map_word(int r, bool var) {
    return map_byte(r, var) | (map_byte(r + 1, var) << 8) | (map_byte(r + 2, var) << 16) | (map_byte(r + 3, var) << 24);
}

__device__ __forceinline__ unsigned vperm(unsigned hi, unsigned lo, unsigned sel) { return __builtin_amdgcn_perm(hi, lo, sel); }
__device__ __forceinline__ unsigned bfi(unsigned m, unsigned a, unsigned b) { return (a & m) | (b & ~m); }
__device__ __forceinline__ unsigned wave_incl_scan(unsigned v) {
    v += __builtin_amdgcn_update_dpp(0u, v, 0x111, 0xf, 0xf, false);
    v += __builtin_amdgcn_update_dpp(0u, v, 0x112, 0xf, 0xf, false);
    v += __builtin_amdgcn_update_dpp(0u, v, 0x114, 0xf, 0xf, false);
    v += __builtin_amdgcn_update_dpp(0u, v, 0x118, 0xf, 0xf, false);
    v += __builtin_amdgcn_update_dpp(0u, v, 0x142, 0xa, 0xf, false);
    v += __builtin_amdgcn_update_dpp(0u, v, 0x143, 0xc, 0xf, false);
    return v;
}
__device__ __forceinline__ unsigned esc_bits(unsigned l) { return l & (l >> 1) & 0x55555555u; }

__device__ __forceinline__ u32x2 lut_entry(unsigned b, bool enc) {
    unsigned sel = 0, cnt = 0;
#pragma unroll
    for (int k = 0; k < 4; ++k) {
        const unsigned c = (b >> (2 * k)) & 3u;
        if (enc) { if (c == 3u) { sel |= (unsigned)k << (8 * cnt); ++cnt; } }
        else { sel |= (c == 3u ? 4u + cnt : c) << (8 * k); cnt += (c == 3u); }
    }
    if (enc && cnt < 4u) sel |= 0x07060504u << (8 * cnt);
    return u32x2{sel, 0x03020100u + (4u - cnt) * 0x01010101u};
}

template <int OFF> __device__ __forceinline__ void lds_or(unsigned a, unsigned v) {
    asm volatile("ds_or_b32 %0, %1 offset:%2" ::"v"(a), "v"(v), "i"(OFF) : "memory");
}
template <int NW, int J> __device__ __forceinline__ void compact_step(const unsigned (&P)[NW], const u32x2 (&E)[NW], unsigned& w, unsigned& w2, unsigned& u3) {
    w2 = vperm(w2, w, E[J].y);
    w = vperm(w, P[J], E[J].x);
    u3 += E[J].y & 0xffu;
    lds_or<4 * J>(u3 & ~3u, __builtin_amdgcn_alignbyte(w2, w, ~u3));
    if constexpr (J > 0) compact_step<NW, J - 1>(P, E, w, w2, u3);
}
template <int NW>
__device__ __forceinline__ void compact(const unsigned (&P)[NW], const u32x2 (&E)[NW], unsigned pos_end, LDS_AS unsigned char* dst) {
    unsigned w = 0, w2 = 0, u3 = lds_addr(dst) + pos_end + 3u - 4u * NW;
    compact_step<NW, NW - 1>(P, E, w, w2, u3);
    u3 -= 3u;
    lds_or<0>(u3 & ~3u, w << (8u * (u3 & 3u)));
}
template <int NW> __device__ __forceinline__ void lut_read(u32x2 (&E)[NW], const unsigned (&B8)[NW], const LDS_AS unsigned char* lut) {
#pragma unroll
    for (int j = NW - 1; j >= 0; --j) E[j] = *(const LDS_AS u32x2*)(lut + B8[j]);
    LDS_FENCE();
}

struct Packed { u32x4 N; u32x2 L1; };
__device__ __forceinline__ u32x4 nibble_plane(const u32x4 a, const u32x4 b) {
    constexpr unsigned K = 0x87878787u;
    return u32x4{bfi(K, a.x, __builtin_amdgcn_alignbit(a.y, a.y, 28)), bfi(K, a.z, __builtin_amdgcn_alignbit(a.w, a.w, 28)),
                 bfi(K, b.x, __builtin_amdgcn_alignbit(b.y, b.y, 28)), bfi(K, b.z, __builtin_amdgcn_alignbit(b.w, b.w, 28))};
}
__device__ __forceinline__ void tail_alloc(unsigned& v, LDS_AS unsigned* top, unsigned bytes) {
    asm volatile("s_mov_b64 exec, 1\n\tds_add_rtn_u32 %0, %1, %2\n\ts_mov_b64 exec, -1" : "=&v"(v) : "v"(lds_addr(top)), "v"(bytes) : "memory");
}

__device__ __forceinline__ void encode_clear(LDS_AS unsigned char* sc, int lane) {
    *(LDS_AS u32x4*)(sc + CL_A2 + lane * 16) = u32x4{0, 0, 0, 0};
    *(LDS_AS u32x2*)(sc + CL_A3 + lane * 8) = u32x2{0, 0};
    *(LDS_AS unsigned*)(sc + CL_A4 + lane * 4) = 0u;
}
template <class F>
__device__ __forceinline__ bool encode_part(const u32x4 R0, const u32x4 R1, unsigned d, LDS_AS unsigned char* sc, LDS_AS unsigned char* tb,
                                            LDS_AS unsigned* top, const LDS_AS unsigned char* lut, int lane, F planes) {
    Packed o;
    const unsigned X[8] = {R0.x, R0.y, R0.z, R0.w, R1.x, R1.y, R1.z, R1.w};
    unsigned s = 0;
#pragma unroll
    for (int j = 0; j < 8; ++j) s = __builtin_amdgcn_udot4(X[j] & 0x78787878u, 0x01010101u, s, false);
    s = __builtin_amdgcn_readlane(wave_incl_scan(s), 63);
    const unsigned mu = s + (16384u - 819u);
    const unsigned b = ((mu >> 14) - 1u) & 15u;
    const bool var = (mu & 16383u) >= 7373u;
    const unsigned K = ((16u - b) & 15u) * 0x08080808u;
    const unsigned M0 = var ? map_word(0, true) : map_word(0, false), M1 = map_word(4, true), M2 = map_word(8, true),
                   M3 = var ? map_word(12, true) : map_word(12, false);
    unsigned P[8], B8[8];
#pragma unroll
    for (int j = 0; j < 8; ++j) {
        const unsigned y = (X[j] & 0x78787878u) + K;
        const unsigned s7 = (y >> 3) & 0x07070707u;
        const unsigned hi = vperm(y << 1, y << 9, 0x0b090a08u);
        unsigned m = bfi(hi, vperm(M3, M2, s7), vperm(M1, M0, s7));
        asm("" : "+v"(m));
        B8[j] = __builtin_amdgcn_udot4(m & 0x18181818u, 0x40100401u, 0u, false);
        P[j] = bfi(0x78787878u, X[j], m);
    }
    u32x2 E1[8];
    lut_read<8>(E1, B8, lut);
    o.L1.x = ((B8[0] | (B8[1] << 8) | (B8[2] << 16)) >> 3) | (B8[3] << 21);
    o.L1.y = ((B8[4] | (B8[5] << 8) | (B8[6] << 16)) >> 3) | (B8[7] << 21);
    const unsigned c1 = __builtin_popcount(esc_bits(o.L1.x)) + __builtin_popcount(esc_bits(o.L1.y));
    const unsigned e1 = wave_incl_scan(c1);
    const unsigned n1 = __builtin_amdgcn_readlane(e1, 63);
    if (n1 > (unsigned)CAP1) return false;
    if (c1) compact<8>(P, E1, e1, sc + CL_A2);
    LDS_FENCE();
    const u32x4 v = *(const LDS_AS u32x4*)(sc + CL_A2 + lane * 16);
    LDS_FENCE();
    o.N = nibble_plane(R0, R1);
    planes(o);
    const unsigned Q[4] = {v.x, v.y, v.z, v.w};
    unsigned B2[4];
#pragma unroll
    for (int j = 0; j < 4; ++j) B2[j] = __builtin_amdgcn_udot4(Q[j] & 0x03030303u, 0x40100401u, 0u, false);
    const unsigned L2 = B2[0] | (B2[1] << 8) | (B2[2] << 16) | (B2[3] << 24);
#pragma unroll
    for (int j = 0; j < 4; ++j) B2[j] <<= 3;
    u32x2 E2[4];
    lut_read<4>(E2, B2, lut);
    const unsigned c2 = __builtin_popcount(esc_bits(L2));
    const unsigned e2 = wave_incl_scan(c2);
    const unsigned n2 = __builtin_amdgcn_readlane(e2, 63);
    if (n2 > (unsigned)CAP2) return false;
    const unsigned h2 = (n1 + 7u) >> 3, h3 = (n2 + 7u) >> 3;
    const unsigned s23 = (2u * (h2 + h3) + 3u) & ~3u;
    unsigned av;
    tail_alloc(av, top, s23);
    if (c2) compact<4>(Q, E2, e2, sc + CL_A3);
    LDS_FENCE();
    u32x2 a3 = *(const LDS_AS u32x2*)(sc + CL_A3 + lane * 8);
    asm volatile("s_waitcnt lgkmcnt(0)" : "+v"(av), "+v"(a3) :: "memory");
    const unsigned o2 = __builtin_amdgcn_readfirstlane(av);
    if (o2 + s23 > (unsigned)TAIL_MAX) return true;
    if ((unsigned)lane < (h2 + 1u) >> 1) *(LDS_AS unsigned*)(tb + o2 + lane * 4) = L2;
    LDS_FENCE();
    const unsigned Q3[2] = {a3.x, a3.y};
    unsigned B3[2];
#pragma unroll
    for (int j = 0; j < 2; ++j) B3[j] = __builtin_amdgcn_udot4(bfi(0x01010101u, Q3[j] >> 2, Q3[j] >> 6) & 0x03030303u, 0x40100401u, 0u, false);
    const unsigned L3 = B3[0] | (B3[1] << 8);
    if ((unsigned)lane < h3) *(LDS_AS unsigned short*)(tb + o2 + 2u * h2 + lane * 2) = (unsigned short)L3;
    const unsigned c3 = __builtin_popcount(esc_bits(L3));
    unsigned o4 = 0;
    if (__any(c3 != 0u)) {
        B3[0] <<= 3; B3[1] <<= 3;
        u32x2 E3[2];
        lut_read<2>(E3, B3, lut);
        const unsigned e3 = wave_incl_scan(c3);
        const unsigned n3 = __builtin_amdgcn_readlane(e3, 63);
        if (n3 > (unsigned)CAP3) return false;
        const unsigned s4 = ((n3 + 7u) >> 3) << 2;
        unsigned a4v;
        tail_alloc(a4v, top, s4);
        if (c3) compact<2>(Q3, E3, e3, sc + CL_A4);
        LDS_FENCE();
        unsigned a4 = *(const LDS_AS unsigned*)(sc + CL_A4 + lane * 4);
        asm volatile("s_waitcnt lgkmcnt(0)" : "+v"(a4v), "+v"(a4) :: "memory");
        o4 = __builtin_amdgcn_readfirstlane(a4v);
        if (o4 + s4 > (unsigned)TAIL_MAX) return true;
        if ((unsigned)lane < s4 >> 1) *(LDS_AS unsigned short*)(tb + o4 + lane * 2) = (unsigned short)bfi(0x0f0fu, a4 >> 3, a4 >> 15);
    }
    if (lane == 0) *(LDS_AS unsigned*)(tb + d * 4) = ((o2 >> 2) & 0xffu) | ((((o2 >> 1) + h2) & 0x1ffu) << 8) | (o4 << 15) | (b << 27) | (var ? 0x80000000u : 0u);
    return true;
}
__device__ __forceinline__ u32x4 uniform4(const u32x4 v) {
    return u32x4{(unsigned)__builtin_amdgcn_readfirstlane(v.x), (unsigned)__builtin_amdgcn_readfirstlane(v.y),
                 (unsigned)__builtin_amdgcn_readfirstlane(v.z), (unsigned)__builtin_amdgcn_readfirstlane(v.w)};
}

__device__ __forceinline__ u32x2 win_read(unsigned q) {
    const LDS_AS unsigned* p = (const LDS_AS unsigned*)(size_t)(q & ~3u);
    return u32x2{p[0], p[1]};
}
__device__ __forceinline__ unsigned win_get(const u32x2 w, unsigned q) { return __builtin_amdgcn_alignbyte(w.y, w.x, q); }
__device__ __forceinline__ unsigned lut_sel(unsigned w, int byte) { return *(const LDS_AS unsigned*)(size_t)(((w >> (8 * byte)) & 0xffu) << 2); }
#define PHASE() do { LDS_FENCE(); __builtin_amdgcn_sched_barrier(0); } while (0)
__device__ __forceinline__ void decode_exps(const LDS_AS unsigned char* sl, unsigned d0, unsigned (&X)[2][8], LDS_AS unsigned char* sc0, int lane) {
    const LDS_AS unsigned char* const tb = sl + 6144;
    unsigned hd[2], L2[2], L3[2], S1[2][8], S2[2][4], S3[2][2], E0[2], E1[2], E2[2], T1[2], T2[2], T3[2], w3[2][2];
    u32x2 L1[2];
    const u32x2 hv = *(const LDS_AS u32x2*)(tb + d0 * 4);
#pragma unroll
    for (int k = 0; k < 2; ++k) L1[k] = *(const LDS_AS u32x2*)(sl + 4096 + (d0 + k) * 512 + lane * 8);
    PHASE();
    hd[0] = __builtin_amdgcn_readfirstlane(hv.x); hd[1] = __builtin_amdgcn_readfirstlane(hv.y);
#pragma unroll
    for (int k = 0; k < 2; ++k) {
        L3[k] = *(const LDS_AS unsigned short*)(tb + ((hd[k] >> 7) & 0x3feu) + lane * 2);
        L2[k] = *(const LDS_AS unsigned*)(tb + (hd[k] & 0xffu) * 4 + lane * 4);
    }
    PHASE();
    unsigned c1 = 0;
#pragma unroll
    for (int k = 0; k < 2; ++k) {
        const unsigned kb = ((hd[k] >> 27) & 15u) * 0x080808u;
        const bool var = (hd[k] >> 31) != 0u;
        T1[k] = (rel8(0, 1, 2) + kb) & 0x787878u;
        T2[k] = ((var ? rel8(-1, -2, 3) : rel8(-1, -2, -3)) + kb) & 0x787878u;
        T3[k] = ((var ? rel8(-4, -5, -3) : rel8(-4, -5, 3)) + kb) & 0x787878u;
        E0[k] = esc_bits(L1[k].x); E1[k] = esc_bits(L1[k].y);
        c1 |= (__builtin_popcount(E0[k]) + __builtin_popcount(E1[k])) << (16 * k);
    }
    const unsigned x1 = wave_incl_scan(c1) - c1;
    PHASE();
#pragma unroll
    for (int k = 0; k < 2; ++k) { S3[k][0] = lut_sel(L3[k], 0); S3[k][1] = lut_sel(L3[k], 1); }
#pragma unroll
    for (int k = 0; k < 2; ++k)
#pragma unroll
        for (int j = 0; j < 4; ++j) S2[k][j] = lut_sel(L2[k], j);
    PHASE();
    E2[0] = esc_bits(L2[0]); E2[1] = esc_bits(L2[1]);
    const unsigned c2 = __builtin_popcount(E2[0]) | (__builtin_popcount(E2[1]) << 16);
    const unsigned x2 = wave_incl_scan(c2) - c2;
#pragma unroll
    for (int k = 0; k < 2; ++k) {
        w3[k][0] = 0; w3[k][1] = 0;
        if (hd[k] & 0x1fe0000u) {
            LDS_AS unsigned char* const sc = sc0 + k * CL_BYTES;
            const unsigned l4 = *(const LDS_AS unsigned short*)(tb + ((hd[k] >> 15) & 0x3fcu) + lane * 2);
            const unsigned E3 = esc_bits(L3[k]);
            const unsigned c3 = __builtin_popcount(E3);
            const unsigned p3 = wave_incl_scan(c3) - c3 + lds_addr(sc + CL_A4), p3b = p3 + __builtin_popcount(E3 & 0xffu);
            *(LDS_AS unsigned*)(sc + CL_A4 + lane * 4) = ((l4 << 3) & 0x7878u) | ((l4 << 15) & 0x78780000u);
            LDS_FENCE();
            const u32x2 wa = win_read(p3), wb = win_read(p3b);
            w3[k][0] = win_get(wa, p3);
            w3[k][1] = win_get(wb, p3b);
        }
    }
    unsigned q2[2][4];
#pragma unroll
    for (int k = 0; k < 2; ++k) {
        *(LDS_AS u32x2*)(sc0 + k * CL_BYTES + CL_A3 + lane * 8) = u32x2{vperm(w3[k][0], T3[k], S3[k][0]), vperm(w3[k][1], T3[k], S3[k][1])};
        const unsigned p2 = ((x2 >> (16 * k)) & 0xffffu) + lds_addr(sc0 + k * CL_BYTES + CL_A3);
#pragma unroll
        for (int j = 0; j < 4; ++j) q2[k][j] = j ? p2 + __builtin_popcount(E2[k] & ((1u << (8 * j)) - 1u)) : p2;
    }
    PHASE();
    u32x2 W2[2][4];
#pragma unroll
    for (int k = 0; k < 2; ++k)
#pragma unroll
        for (int j = 0; j < 4; ++j) W2[k][j] = win_read(q2[k][j]);
    PHASE();
    unsigned q1[2][8];
#pragma unroll
    for (int k = 0; k < 2; ++k) {
        unsigned p1 = (x1 >> (16 * k)) & 0xffffu;
        p1 = (p1 > (unsigned)CAP1 ? (unsigned)CAP1 : p1) + lds_addr(sc0 + k * CL_BYTES + CL_A2);
        const unsigned p1b = p1 + __builtin_popcount(E0[k]);
#pragma unroll
        for (int j = 0; j < 8; ++j) {
            const unsigned base = (j & 4) ? p1b : p1, E = (j & 4) ? E1[k] : E0[k];
            q1[k][j] = (j & 3) ? base + __builtin_popcount(E & ((1u << (8 * (j & 3))) - 1u)) : base;
        }
    }
    PHASE();
#pragma unroll
    for (int k = 0; k < 2; ++k)
        *(LDS_AS u32x4*)(sc0 + k * CL_BYTES + CL_A2 + lane * 16) =
            u32x4{vperm(win_get(W2[k][0], q2[k][0]), T2[k], S2[k][0]), vperm(win_get(W2[k][1], q2[k][1]), T2[k], S2[k][1]),
                  vperm(win_get(W2[k][2], q2[k][2]), T2[k], S2[k][2]), vperm(win_get(W2[k][3], q2[k][3]), T2[k], S2[k][3])};
    PHASE();
    u32x2 W1[2][8];
#pragma unroll
    for (int k = 0; k < 2; ++k)
#pragma unroll
        for (int j = 0; j < 8; ++j) { W1[k][j] = win_read(q1[k][j]); S1[k][j] = lut_sel((j & 4) ? L1[k].y : L1[k].x, j & 3); }
    PHASE();
#pragma unroll
    for (int k = 0; k < 2; ++k)
#pragma unroll
        for (int j = 0; j < 8; ++j) X[k][j] = vperm(win_get(W1[k][j], q1[k][j]), T1[k], S1[k][j]);
}
__device__ __forceinline__ void merge_part(const u32x4 N, const unsigned (&X)[8], u32x4& R0, u32x4& R1) {
    constexpr unsigned K = 0x87878787u;
    R0 = u32x4{(N.x & K) | X[0], (__builtin_amdgcn_alignbit(N.x, N.x, 4) & K) | X[1], (N.y & K) | X[2], (__builtin_amdgcn_alignbit(N.y, N.y, 4) & K) | X[3]};
    R1 = u32x4{(N.z & K) | X[4], (__builtin_amdgcn_alignbit(N.z, N.z, 4) & K) | X[5], (N.w & K) | X[6], (__builtin_amdgcn_alignbit(N.w, N.w, 4) & K) | X[7]};
}

constexpr int NSLOT = 8;
constexpr unsigned META_RAW = INFO_RAW << 24;
constexpr unsigned META_INBOX = 0x40000000u;
constexpr int CM_LUT = 0;
constexpr int CM_META = 1024;
constexpr int CM_DONE = CM_META + NSLOT * 16;
constexpr int CM_NEXT = CM_DONE + NSLOT * 4;
constexpr int CM_END = CM_META + 248;
constexpr int CM_LAND = CM_END + 4;
constexpr int CM_TAB = 2048;
constexpr int CM_SLOT = 4096;
constexpr int CM_SC = CM_SLOT + NSLOT * UNIT;
constexpr int ST_LUT = 0, ST_TOP = 2048, ST_SC = ST_TOP + 64, ST_TAIL = ST_SC + 8 * CL_BYTES;
static_assert(CM_SC + 14 * CL_BYTES <= LDS_BYTES && CM_NEXT + 4 <= CM_END && CM_LAND + 256 <= CM_TAB && ST_TAIL + 2048 <= LDS_BYTES, "LDS map");
static_assert(NPB + 2 < NSLOT && PBS >= 2 && PB0 >= 2 && PB0 + (NPB - 1) * PBS < NBLK - 1 && WIN == 8, "roles");

__device__ __forceinline__ bool timed_out(unsigned& spins, unsigned* tmo, unsigned call, unsigned long long t0) {
    if ((++spins & 1023u) != 0u) return false;
    if (__hip_atomic_load(tmo, __ATOMIC_RELAXED, __HIP_MEMORY_SCOPE_AGENT) != call &&
        __builtin_amdgcn_s_memrealtime() - t0 <= DEADLINE_TICKS) return false;
    __hip_atomic_store(tmo, call, __ATOMIC_RELAXED, __HIP_MEMORY_SCOPE_AGENT);
    return true;
}

__device__ __forceinline__ bool stuck(unsigned& spins, unsigned long long t0) {
    return (++spins & 0xffffu) == 0u && __builtin_amdgcn_s_memrealtime() - t0 > LDS_DEADLINE_TICKS;
}

__global__ void __launch_bounds__(BLOCK, 2) ag_gemm(Params p) {
    extern __shared__ __attribute__((aligned(16))) unsigned char smem[];
    unsigned pf0, pf1;
    asm volatile("s_load_dword %0, %2, 0x0\n\ts_load_dword %1, %2, 0x40" : "=&s"(pf0), "=&s"(pf1) : "s"(__builtin_amdgcn_kernarg_segment_ptr()));
    __builtin_amdgcn_sched_barrier(0);
    const int lane = threadIdx.x & 63;
    const int wave = __builtin_amdgcn_readfirstlane(threadIdx.x >> 6);
    const int bid = blockIdx.x;
    unsigned char* const peer = p.ws[((bid >> 3) + 1) & 7];
    const unsigned su0 = (bid & 63) + 64u * ((0x21404300u >> (8 * (bid >> 6) + (wave & 4))) & 7u);
    unsigned soff = ((su0 >> 3) * 16 + (lane & 15)) * KD + (su0 & 7) * 512 + (wave & 3) * 128 + (lane >> 4) * 16;
    unsigned poff = (((wave >> 2) * PBS + PB0) * 16 + (lane >> 3)) * KD + (bid & 7) * 512 + (wave & 3) * 128 + (lane & 7) * 16;
    unsigned poff2 = poff + 8 * KD, coff = CNT_OFF + bid * 128;
    asm volatile("" : "+v"(soff), "+v"(poff), "+v"(poff2), "+v"(coff));
    asm volatile("s_waitcnt lgkmcnt(0)" ::"s"(pf0), "s"(pf1), "s"(p.x), "s"(p.w), "s"(p.sx), "s"(p.sw), "s"(p.out), "s"(p.rank), "s"(peer), "s"(p.ws[0]));
    unsigned char* const myws = p.ws[0];
    unsigned* const cntp = (unsigned*)(myws + CNT_OFF + (size_t)bid * 128);
    unsigned* const tmo = (unsigned*)(myws + TMO_OFF);
    LDS_AS unsigned char* const lds = (LDS_AS unsigned char*)smem;
    unsigned call;
#define LOAD_COUNT() unsigned cv; asm volatile("global_load_dword %0, %1, %2 sc1" : "=&v"(cv) : "v"(coff), "s"(myws) : "memory")

    if (__builtin_expect(bid >= NCOMM, 1)) {
      if (bid < NCOMM + 56) {
        LOAD_COUNT();
        const int c_k = bid & 7, i0 = wave >> 2;
        u32x4 A[PUSH_ROUNDS][2];
#pragma unroll
        for (int i = 0; i < PUSH_ROUNDS; ++i) {
            const unsigned char* s = p.x + (i0 + 2 * i < NPB ? (size_t)i * (2 * PBS * 16 * KD) : 0);
            ld16<0>(A[i][0], s, poff); ld16<0>(A[i][1], s, poff2);
            if (i == 0) __builtin_amdgcn_s_barrier();
        }
        unsigned char* dst = peer + NI_OFF + ((size_t)p.rank * 128 + i0 * 8 + c_k) * 4096 + (wave & 3) * 1024 + lane * 16;
        asm volatile("s_waitcnt vmcnt(%4)" : "+v"(cv), "+v"(A[0][0]), "+v"(A[0][1]), "+v"(dst) : "i"(2 * PUSH_ROUNDS - 2) : "memory");
        call = __builtin_amdgcn_readfirstlane(cv) + 1u;
        dst += (size_t)(call & 1u) * 64 * 4096;
        st16_sys<0>(dst, nibble_plane(A[0][0], A[0][1]));
#pragma unroll
        for (int i = 1; i < PUSH_ROUNDS; ++i) {
            wait_vm<0>();
            asm volatile("" : "+v"(A[i][0]), "+v"(A[i][1]));
            if (i0 + 2 * i < NPB) st16_sys<0>(dst + (size_t)i * (16 * 4096), nibble_plane(A[i][0], A[i][1]));
        }
        wait_vm<0>();
        __builtin_amdgcn_s_barrier();
        if (wave == 0)
            asm volatile("s_mov_b64 exec, 1\n\tglobal_store_dwordx2 %0, %1, off sc0 sc1\n\ts_mov_b64 exec, -1"
                         ::"v"(peer + RDY_OFF + (((size_t)(call & 1u) * NR + p.rank) * NRDY + NUNIT + (bid & 7)) * 128), "v"(u32x2{1u, call}) : "memory");
      }
      {
        LOAD_COUNT();
        const unsigned d = wave & 3;
        const bool stage = su0 < NUNIT;
        const unsigned su = stage ? su0 : NUNIT - 1;
        u32x4 R0, R1;
        if (stage) { ld16<0>(R0, p.x, soff); ld16<64>(R1, p.x, soff); }
        unsigned char* wsl;
        asm volatile("global_load_dwordx2 %0, %1, off" : "=&v"(wsl)
                     : "v"((const unsigned char*)__builtin_amdgcn_kernarg_segment_ptr() + offsetof(Params, ws) + (lane < 8 ? lane * 8 : 0)) : "memory");
        LDS_AS unsigned char* const sc = lds + ST_SC + wave * CL_BYTES;
        LDS_AS unsigned char* const tb = lds + ST_TAIL + (wave >> 2) * 1024;
        LDS_AS unsigned* const top = (LDS_AS unsigned*)(lds + ST_TOP) + (wave >> 2);
        encode_clear(sc, lane);
        *top = 16u;
        if (threadIdx.x >= 256) *(LDS_AS u32x2*)(lds + ST_LUT + (threadIdx.x - 256) * 8) = lut_entry(threadIdx.x - 256, true);
        wait_lgkm();
        __builtin_amdgcn_s_barrier();
        asm volatile("s_waitcnt vmcnt(1)" : "+v"(cv), "+v"(R0), "+v"(R1) :: "memory");
        call = __builtin_amdgcn_readfirstlane(cv) + 1u;
        unsigned char* const cp = myws + C_OFF + (size_t)(call & 1u) * SHARD + (size_t)su * UNIT;
        unsigned char* const x = myws + X_OFF + ((size_t)(call & 1u) * NR + p.rank) * SHARD + (size_t)su * UNIT + d * 2048 + lane * 16;
        bool rawdone = false, packed = false;
        if (stage && su < NUNIT - RAW_UNITS)
            packed = encode_part(R0, R1, d, sc, tb, top, lds + ST_LUT, lane, [&](const Packed& q) {
                st16_sys<0>(cp + d * 1024 + lane * 16, q.N);
                asm volatile("global_store_dwordx2 %0, %1, off sc0 sc1" ::"v"(cp + 4096 + d * 512 + lane * 8), "v"(q.L1) : "memory");
                st16_sys<0>(x, R0); st16_sys<1024>(x, R1);
                rawdone = true;
            });
        if (stage && !rawdone) { st16_sys<0>(x, R0); st16_sys<1024>(x, R1); }
        if (!packed) lds_or<0>(lds_addr(top), 0x10000u);
        asm volatile("s_waitcnt vmcnt(0) lgkmcnt(0)" : "+v"(wsl) :: "memory");
        __builtin_amdgcn_s_barrier();
        LDS_FENCE();
        const unsigned tv = *top;
        const u32x4 tl = *(const LDS_AS u32x4*)(tb + lane * 16);
        if (d == 0 && packed) st16_sys<0>(cp + 6144 + lane * 16, tl);
        const unsigned total = __builtin_amdgcn_readfirstlane(tv);
        unsigned char* fp = wsl + (lane < 8 ? RDY_OFF + ((size_t)(call & 1u) * NR + p.rank) * NRDY * 128 : FLAG_OFF + (size_t)p.rank * NUNIT * 128 + (lane & 1) * 8) + (size_t)su * 128;
        u32x2 fv = u32x2{lane >= 8 ? call : (total <= (unsigned)TAIL_MAX ? (total + 15u) >> 4 : INFO_RAW), call};
        asm volatile("s_waitcnt vmcnt(0)" : "+v"(fp), "+v"(fv) :: "memory");
        if (d == 0 && stage)
            asm volatile("s_mov_b64 exec, 0x3fe\n\tglobal_store_dwordx2 %0, %1, off sc0 sc1\n\ts_mov_b64 exec, -1" ::"v"(fp), "v"(fv) : "memory");
        __builtin_amdgcn_s_barrier();
      }
    } else {
        const unsigned long long t0 = __builtin_amdgcn_s_memrealtime();
        const int c_src = bid >> 3, c_k = bid & 7;
        const int src_rank = (p.rank + c_src + 1) & 7;
        if (wave == 1) {
            const unsigned k = lane & 7;
            const unsigned char* tp = k == 0 ? peer + NI_OFF + (size_t)p.rank * 128 * 4096 : k == 1 ? peer + RDY_OFF
                                    : k == 2 ? peer + C_OFF : k == 3 ? peer + X_OFF + (size_t)src_rank * SHARD + (SHARD - 128)
                                    : k == 4 ? myws + X_OFF + (size_t)src_rank * SHARD : k == 5 ? (const unsigned char*)p.out + (size_t)src_rank * (MSH * NSH * 4)
                                    : k == 6 ? p.w : (const unsigned char*)p.sx;
            if (lane >= 8) tp = k == 2 ? tp + SHARD : (k == 3 || k == 4) ? tp + NR * SHARD : k == 0 ? myws + NI_OFF + (size_t)src_rank * 128 * 4096 : k == 7 ? (const unsigned char*)p.sw : tp;
            asm volatile("s_mov_b32 m0, %1\n\ts_mov_b64 exec, 0xffff\n\tglobal_load_lds_dword %0, off sc0 sc1\n\ts_mov_b64 exec, -1"
                         ::"v"(tp), "s"(lds_addr(lds + CM_SC)) : "memory");
        }
        if (c_src == 7 || lds_addr(lds + CM_LUT) != 0u) return;
        LOAD_COUNT();
        if (threadIdx.x < 256) *(LDS_AS unsigned*)(lds + CM_LUT + threadIdx.x * 4) = lut_entry(threadIdx.x, false).x;
        else *(LDS_AS u32x2*)(lds + CM_TAB + (threadIdx.x - 256) * 8) = u32x2{0u, 0u};
        if (wave == 0) *(LDS_AS unsigned*)(lds + CM_META + lane * 4) = 0u;
        asm volatile("s_waitcnt vmcnt(0) lgkmcnt(0)" : "+v"(cv) :: "memory");
        __builtin_amdgcn_s_barrier();
        call = __builtin_amdgcn_readfirstlane(cv) + 1u;
        const size_t xo = X_OFF + ((size_t)(call & 1u) * NR + src_rank) * SHARD;
        LDS_AS unsigned char* const tab = lds + CM_TAB;
        unsigned spins = 0;
        if (wave == 0) {
            const unsigned char* const cmpp = peer + C_OFF + (size_t)(call & 1u) * SHARD + lane * 16;
            const size_t rawd = xo - (C_OFF + (size_t)(call & 1u) * SHARD);
            unsigned char* const tick = myws + TICK_OFF + c_src * 128;
            const unsigned tbase = (call - 1u) * (NDYN - 8) - 16u;
            unsigned q = 0;
            unsigned n = 16;
            unsigned lv = n - (WIN - 1);
            const unsigned landp = lds_addr(lds + CM_LAND) + lane * 4;
#define STEP "s_waitcnt vmcnt(%[w])\n\tv_add_u32 %[lv], 1, %[lv]\n\tds_write_b32 %[lp], %[lv]\n\t"
            unsigned tabv = 0;
            auto ready = [&](unsigned u) -> unsigned {
                unsigned info = ((unsigned)__builtin_amdgcn_readlane(tabv, u >> 2) >> (8 * (u & 3u))) & 0xffu;
                while (__builtin_expect(info == 0u, 0)) {
                    tabv = __hip_atomic_load((LDS_AS unsigned*)tab + lane, __ATOMIC_ACQUIRE, __HIP_MEMORY_SCOPE_WORKGROUP);
                    info = ((unsigned)__builtin_amdgcn_readlane(tabv, u >> 2) >> (8 * (u & 3u))) & 0xffu;
                    if (stuck(spins, t0)) info = INFO_FAIL;
                }
                return info;
            };
            bool ok = true;
            auto slot_free = [&](unsigned seen) {
                if (__builtin_expect((unsigned)__builtin_amdgcn_readfirstlane(seen) != 2u * (q / NSLOT), 0))
                    while (ok && *(volatile LDS_AS unsigned*)(lds + CM_DONE + (q & (NSLOT - 1)) * 4) != 2u * (q / NSLOT)) {
                        wait_vm<0>();
                        *(volatile LDS_AS unsigned*)(size_t)landp = n;
                        ok = !stuck(spins, t0);
                    }
            };
            auto first_half = [&](unsigned u, unsigned info, unsigned mark) -> const unsigned char* {
                const unsigned slot = q & (NSLOT - 1);
                const unsigned char* g = cmpp + ((size_t)u * UNIT + ((info & INFO_RAW) ? rawd : 0));
                const unsigned la = lds_addr(lds + CM_SLOT) + slot * UNIT + 4096;
                asm volatile("s_mov_b32 m0, %[la]\n\ts_mov_b64 exec, %[mk]\n\t"
                             "global_load_lds_dwordx4 %[g], off offset:2048 sc0 sc1\n\t"
                             "s_mov_b64 exec, 1\n\tds_write_b128 %[ma], %[mv]\n\ts_mov_b64 exec, -1\n\t" STEP
                             "global_load_lds_dwordx4 %[g], off sc0 sc1\n\t" STEP
                             "global_load_lds_dwordx4 %[g], off offset:1024 sc0 sc1\n\t" STEP
                             : [lv] "+&v"(lv)
                             : [g] "v"(g + 4096), [la] "s"(la), [mk] "s"(info >= 64u ? ~0ull : (1ull << info) - 1ull), [lp] "v"(landp), [w] "i"(WIN - 1),
                               [ma] "v"(lds_addr(lds + CM_META) + slot * 16), [mv] "v"(u32x4{q + 1u, u | mark | ((info & INFO_RAW) << 24), n, 0u})
                             : "memory");
                n += 3;
                if (__builtin_expect((info & INFO_RAW) != 0u, 0)) {
                    asm volatile("s_mov_b32 m0, %[la]\n\ts_nop 0\n\tglobal_load_lds_dwordx4 %[g], off offset:3072 sc0 sc1\n\t" STEP
                                 : [lv] "+&v"(lv) : [g] "v"(g + 4096), [la] "s"(la), [lp] "v"(landp), [w] "i"(WIN - 1) : "memory");
                    ++n;
                }
                ++q;
                return g;
            };
            auto second_half = [&](unsigned qq, const unsigned char* g) {
                const unsigned slot = qq & (NSLOT - 1);
                asm volatile("s_mov_b32 m0, %[la]\n\ts_nop 0\n\t"
                             "global_load_lds_dwordx4 %[g], off sc0 sc1\n\t"
                             "s_mov_b64 exec, 1\n\tds_write_b32 %[ma], %[mv]\n\ts_mov_b64 exec, -1\n\t" STEP
                             "global_load_lds_dwordx4 %[g], off offset:1024 sc0 sc1\n\t" STEP
                             "global_load_lds_dwordx4 %[g], off offset:2048 sc0 sc1\n\t" STEP
                             "global_load_lds_dwordx4 %[g], off offset:3072 sc0 sc1\n\t" STEP
                             : [lv] "+&v"(lv)
                             : [g] "v"(g), [la] "s"(lds_addr(lds + CM_SLOT) + slot * UNIT), [lp] "v"(landp), [w] "i"(WIN - 1),
                               [ma] "v"(lds_addr(lds + CM_META) + slot * 16 + 12), [mv] "v"(n)
                             : "memory");
                n += 4;
            };
            unsigned up = PB0 * 8 + c_k, ip = 0;
            auto pushed_below = [&](unsigned u) {
                while (ok && up < u) {
                    const unsigned info = ready(up);
                    unsigned pl;
                    while ((pl = __builtin_amdgcn_readfirstlane(__hip_atomic_load(tab + NUNIT + c_k, __ATOMIC_ACQUIRE, __HIP_MEMORY_SCOPE_WORKGROUP))) == 0u &&
                           !stuck(spins, t0)) {}
                    ok = info != INFO_FAIL && pl == 1u;
                    slot_free(~0u);
                    if (ok) {
                        const unsigned char* g = first_half(up, info, META_INBOX | (ip << 8));
                        if (info & INFO_RAW) second_half(q - 1u, g);
                        up = ++ip < (unsigned)NPB ? up + 8 * PBS : ~0u;
                    }
                }
            };
            while (__builtin_amdgcn_s_memrealtime() - t0 < HOLD_TICKS) __builtin_amdgcn_s_sleep(1);
            unsigned info = ready(c_k);
            if (info != INFO_FAIL) {
                const unsigned char* gcur = first_half(c_k, info, 0u);
                unsigned qcur = 0, k = c_k + 8u, seen = 0;
                while (k < (unsigned)NDYN) {
                    const unsigned kb = (k >> 3) < (unsigned)PB0 ? 0u : ((k >> 3) - PB0) / (PBS - 1) + 1u;
                    const unsigned u = k + 8u * (kb < (unsigned)NPB ? kb : NPB);
                    if (__builtin_expect(up < u, 0)) { pushed_below(u); seen = ~0u; }
                    info = ready(u);
                    slot_free(seen);
                    if (__builtin_expect(!ok || info == INFO_FAIL, 0)) break;
                    unsigned tv;
                    ticket(tv, tick); ++n; ++lv;
                    const unsigned char* gnext = first_half(u, info, 0u);
                    seen = *(volatile LDS_AS unsigned*)(lds + CM_DONE + (q & (NSLOT - 1)) * 4);
                    second_half(qcur, gcur);
                    asm volatile("" : "+v"(tv));
                    k = __builtin_amdgcn_readfirstlane(tv) - tbase;
                    gcur = gnext; qcur = q - 1u;
                }
                second_half(qcur, gcur);
                pushed_below(NUNIT);
            }
            if (lane == 0) *(volatile LDS_AS unsigned*)(lds + CM_END) = q + 1u;
#define LANDED(k) do { wait_vm<k>(); *(volatile LDS_AS unsigned*)(size_t)landp = n - (k); } while (0)
            LANDED(6); LANDED(5); LANDED(4); LANDED(3); LANDED(2); LANDED(1); LANDED(0);
            return;
        }
        if (wave == 7) {
            flag_store((unsigned char*)cntp, call);
            const unsigned char* rp = myws + RDY_OFF + ((size_t)(call & 1u) * NR + src_rank) * NRDY * 128;
            for (int u0 = -56; u0 < NUNIT; u0 += 64) {
                const int u = u0 < 0 ? c_k + (lane & 1) * 8 : (u0 + lane < 16 ? NUNIT + c_k : (u0 + lane < NUNIT ? u0 + lane : NUNIT - 1));
                while (true) {
                    u32x2 r;
                    asm volatile("global_load_dwordx2 %0, %1, off sc0 sc1\n\ts_waitcnt vmcnt(0)" : "=&v"(r) : "v"(rp + (size_t)u * 128) : "memory");
                    const unsigned t = r.y >= call ? r.x : 0u;
                    __hip_atomic_store(tab + u, (unsigned char)t, __ATOMIC_RELEASE, __HIP_MEMORY_SCOPE_WORKGROUP);
                    if (__all(t != 0u)) break;
                    if (timed_out(spins, tmo, call, t0)) {
                        *(volatile LDS_AS u32x2*)(lds + CM_TAB + lane * 8) = u32x2{~0u, ~0u};
                        u0 = NUNIT;
                        break;
                    }
                }
            }
        }
        LDS_AS unsigned char* const sc = lds + CM_SC + (wave - 1) * (2 * CL_BYTES);
        unsigned char* const dst = myws + xo + lane * 16;
        unsigned char* const fl = myws + FLAG_OFF + (size_t)src_rank * NUNIT * 128;
        const unsigned char* const inbox = myws + NI_OFF + (((size_t)src_rank * 2 + (call & 1u)) * 64 + c_k) * 4096 + lane * 16;
        while (true) {
            const unsigned r = lds_ticket(lds + CM_NEXT);
            const unsigned q = r >> 1, h = r & 1u;
            const unsigned slot = q & (NSLOT - 1);
            const LDS_AS unsigned char* const sl = lds + CM_SLOT + slot * UNIT;
            unsigned uw;
            bool end = false;
            while (true) {
                const u32x4 m = uniform4(*(volatile LDS_AS u32x4*)(lds + CM_META + slot * 16));
                const u32x2 lv = *(volatile LDS_AS u32x2*)(lds + CM_END);
                const unsigned land = __builtin_amdgcn_readfirstlane(lv.y), e = __builtin_amdgcn_readfirstlane(lv.x);
                uw = m.y;
                if (m.x == q + 1u) {
                    if (land >= m.z + ((uw & META_RAW) ? 4u * h : 2u + h)) break;
                } else if (e - 1u <= q) { end = true; break; }
                if (stuck(spins, t0)) return;
                __builtin_amdgcn_s_sleep(1);
            }
            if (end) break;
            LDS_FENCE();
            const bool raw = (uw & META_RAW) != 0u;
            const bool inb = !raw && (uw & META_INBOX) != 0u;
            if (inb) {
                const unsigned char* g = inbox + (size_t)((uw >> 8) & 7u) * (8 * 4096) + h * 2048;
                glds_sys<0>(g, lds_addr(sl) + h * 2048); glds_sys<1024>(g, lds_addr(sl) + h * 2048);
            }
            unsigned X[2][8];
            if (!raw) {
                decode_exps(sl, 2u * h, X, sc, lane);
#pragma unroll
                for (int k = 0; k < 2; ++k)
                    asm volatile("" : "+v"(X[k][0]), "+v"(X[k][1]), "+v"(X[k][2]), "+v"(X[k][3]), "+v"(X[k][4]), "+v"(X[k][5]), "+v"(X[k][6]), "+v"(X[k][7]));
            }
            if (inb) wait_vm<0>();
            else if (!raw || h == 0u) {
                while (true) {
                    const unsigned ib = __builtin_amdgcn_readfirstlane(*(volatile LDS_AS unsigned*)(lds + CM_META + slot * 16 + 12));
                    const unsigned land = __builtin_amdgcn_readfirstlane(*(volatile LDS_AS unsigned*)(lds + CM_LAND));
                    if (ib != 0u && land >= ib + (raw ? 4u : 2u * h + 2u)) break;
                    if (stuck(spins, t0)) return;
                }
                LDS_FENCE();
            }
            u32x4 R[4];
            if (raw) {
#pragma unroll
                for (int i = 0; i < 4; ++i) R[i] = *(const LDS_AS u32x4*)(sl + h * 4096 + i * 1024 + lane * 16);
            } else {
                const LDS_AS unsigned char* const np = sl + h * 2048 + (inb ? (lane & 7) * 128 + (lane >> 4) * 16 + (lane & 8) : lane * 16);
                const LDS_AS unsigned char* const nq = np + (inb ? 64 : 8);
                const u32x2 a0 = *(const LDS_AS u32x2*)np, b0 = *(const LDS_AS u32x2*)nq, a1 = *(const LDS_AS u32x2*)(np + 1024), b1 = *(const LDS_AS u32x2*)(nq + 1024);
                const u32x4 N0 = u32x4{a0.x, a0.y, b0.x, b0.y}, N1 = u32x4{a1.x, a1.y, b1.x, b1.y};
                merge_part(N0, X[0], R[0], R[1]);
                merge_part(N1, X[1], R[2], R[3]);
            }
            const unsigned u = uw & (NUNIT - 1);
            unsigned char* const o = dst + (size_t)u * UNIT + h * 4096;
            st16_sys<0>(o, R[0]); st16_sys<1024>(o, R[1]); st16_sys<2048>(o, R[2]); st16_sys<3072>(o, R[3]);
            lds_inc(lds + CM_DONE + slot * 4);
            wait_vm<0>();
            flag_store(fl + (size_t)u * 128 + h * 8, call);
        }
        return;
    }

    const int cn = bid - NCOMM;
    const int cg = (cn >> 3) & 7;
    const int src_rank = (p.rank + cn + 3 * (cn >> 6)) & 7;
    if (wave == 7) flag_store((unsigned char*)cntp, call);

    i32x8 Wf[4][4];
    LDS_AS unsigned char* const wl = lds + wave * 16384;
    {
        const unsigned char* wsrc = p.w + (size_t)(wave * 512 + (lane >> 2)) * NSH + cg * 64 + (lane & 3) * 16;
        const int g = lane >> 4, q = (lane & 15) >> 1, h = lane & 1;
        LDS_AS unsigned char* const rd = wl + (16 * g + q) * 64 + h * 8;
#pragma unroll
        for (int half = 0; half < 2; ++half) {
#pragma unroll
            for (int i = 0; i < 16; ++i)
                __builtin_amdgcn_global_load_lds((const GLB_AS unsigned*)(wsrc + (size_t)(half * 256 + i * 16) * NSH),
                                                 (LDS_AS unsigned*)(wl + i * 1024), 16, 0, 0);
            wait_vm<0>();
#pragma unroll
            for (int s2 = 0; s2 < 2; ++s2)
#pragma unroll
                for (int t = 0; t < 4; ++t)
#pragma unroll
                    for (int pc = 0; pc < 4; ++pc) {
                        i32x2 v = __builtin_amdgcn_ds_read_tr8_b64_v2i32(
                            (LDS_AS i32x2*)(rd + (s2 * 128 + (pc >> 1) * 64 + (pc & 1) * 8) * 64 + t * 16));
                        Wf[half * 2 + s2][t][pc * 2] = v.x;
                        Wf[half * 2 + s2][t][pc * 2 + 1] = v.y;
                    }
            wait_lgkm();
        }
    }
    const float scale = p.sx[0] * p.sw[0];
    const unsigned char* const xl = myws + X_OFF + ((size_t)(call & 1u) * NR + src_rank) * SHARD + wave * UNIT + 4096 + lane * 16;
    const unsigned* const flags = (const unsigned*)(myws + FLAG_OFF + ((size_t)src_rank * NUNIT + wave) * 128 + (lane & 1) * 8);
    LDS_AS unsigned char* const pw = lds + wave * PART_BYTES + (lane & 15) * PART_STRIDE + (lane >> 4) * 16;
    const LDS_AS unsigned char* const pr = lds + (wave * 4 + (lane >> 4)) * PART_STRIDE + (lane & 15) * 16;
    float* const outp = p.out + (size_t)(src_rank * MSH + wave * 4 + (lane >> 4)) * NSH + cg * 64 + (lane & 15) * 4;
    __builtin_amdgcn_s_barrier();
    const unsigned long long t0 = __builtin_amdgcn_s_memrealtime();
    int buf = 0;
    bool bad = false;
    for (int b = 2 - (cn >> 6); b < NBLK; b += 3) {
        const unsigned* f = flags + (size_t)b * (8 * 32);
        unsigned spins = 0;
        while (!bad && !__all(__hip_atomic_load(f, __ATOMIC_RELAXED, __HIP_MEMORY_SCOPE_SYSTEM) >= call)) {
            bad = timed_out(spins, tmo, call, t0);
            __builtin_amdgcn_s_sleep(1);
        }
        u32x4 R[8];
        asm volatile(
            "global_load_dwordx4 %0, %8, off offset:-4096 sc0 sc1\n\tglobal_load_dwordx4 %1, %8, off offset:-3072 sc0 sc1\n\t"
            "global_load_dwordx4 %2, %8, off offset:-2048 sc0 sc1\n\tglobal_load_dwordx4 %3, %8, off offset:-1024 sc0 sc1\n\t"
            "global_load_dwordx4 %4, %8, off sc0 sc1\n\tglobal_load_dwordx4 %5, %8, off offset:1024 sc0 sc1\n\t"
            "global_load_dwordx4 %6, %8, off offset:2048 sc0 sc1\n\tglobal_load_dwordx4 %7, %8, off offset:3072 sc0 sc1\n\t"
            "s_waitcnt vmcnt(6)"
            : "=&v"(R[0]), "=&v"(R[1]), "=&v"(R[2]), "=&v"(R[3]), "=&v"(R[4]), "=&v"(R[5]), "=&v"(R[6]), "=&v"(R[7])
            : "v"(xl + (size_t)b * (8 * UNIT)) : "memory");
        const float c0 = bad ? __builtin_nanf("") : 0.0f;
        f32x4 acc[4];
#pragma unroll
        for (int t = 0; t < 4; ++t) acc[t] = f32x4{c0, c0, c0, c0};
#pragma unroll
        for (int s = 0; s < 4; ++s) {
            if (s > 0) {
                __builtin_amdgcn_sched_barrier(0);
                if (s == 1) asm volatile("s_waitcnt vmcnt(4)" : "+v"(R[2]), "+v"(R[3]) :: "memory");
                if (s == 2) asm volatile("s_waitcnt vmcnt(2)" : "+v"(R[4]), "+v"(R[5]) :: "memory");
                if (s == 3) asm volatile("s_waitcnt vmcnt(0)" : "+v"(R[6]), "+v"(R[7]) :: "memory");
            }
            i32x8 xf;
            xf[0] = R[2 * s].x; xf[1] = R[2 * s].y; xf[2] = R[2 * s].z; xf[3] = R[2 * s].w;
            xf[4] = R[2 * s + 1].x; xf[5] = R[2 * s + 1].y; xf[6] = R[2 * s + 1].z; xf[7] = R[2 * s + 1].w;
#pragma unroll
            for (int t = 0; t < 4; ++t)
                acc[t] = __builtin_amdgcn_mfma_scale_f32_16x16x128_f8f6f4(Wf[s][t], xf, acc[t], 1, 0, 0, 0, 0, 0);
        }
        __builtin_amdgcn_sched_barrier(0);
#pragma unroll
        for (int t = 0; t < 4; ++t) *(LDS_AS f32x4*)(pw + buf * (8 * PART_BYTES) + t * 64) = acc[t];
        wait_lgkm();
        __builtin_amdgcn_s_barrier();
        LDS_FENCE();
        if (wave < 4) {
            f32x4 v[8];
#pragma unroll
            for (int w2 = 0; w2 < 8; ++w2) v[w2] = *(const LDS_AS f32x4*)(pr + buf * (8 * PART_BYTES) + w2 * PART_BYTES);
            __builtin_amdgcn_sched_barrier(0);
            const f32x4 sum = (((v[0] + v[1]) + (v[2] + v[3])) + ((v[4] + v[5]) + (v[6] + v[7]))) * scale;
            st16_sys<0>((unsigned char*)(outp + (size_t)b * (16 * NSH)), __builtin_bit_cast(u32x4, sum));
        }
        buf ^= 1;
    }
}

extern "C" void kernel_launch(int rank, int nranks, ncclComm_t comm, hipStream_t stream,
                              void* const* d_in, const int* in_sizes, int n_in,
                              void* d_out, int out_size,
                              const int* in_dtypes, int out_dtype,
                              void* const* d_out_all,
                              void* const* d_ws_all, size_t ws_bytes) {
    (void)comm; (void)in_dtypes; (void)out_dtype; (void)d_out_all;
    if (nranks != NR || (unsigned)rank >= (unsigned)NR || n_in != 4 || ws_bytes < WS_NEED ||
        in_sizes[0] != (int)SHARD || in_sizes[1] != KD * NSH || out_size != NR * MSH * NSH) {
        fprintf(stderr, "unexpected problem shape\n");
        return;
    }
    static bool init = false;
    if (!init) {
        HIP_CHECK(hipFuncSetAttribute((const void*)ag_gemm, hipFuncAttributeMaxDynamicSharedMemorySize, LDS_BYTES));
        init = true;
    }
    Params p;
    p.x = (const unsigned char*)d_in[0];
    p.w = (const unsigned char*)d_in[1];
    p.sx = (const float*)d_in[2];
    p.sw = (const float*)d_in[3];
    p.out = (float*)d_out;
    for (int d = 0; d < NR; ++d) p.ws[d] = (unsigned char*)d_ws_all[(rank + d) % NR];
    p.rank = rank;
    p.pad = 0;
    hipLaunchKernelGGL(ag_gemm, dim3(GRID), dim3(BLOCK), LDS_BYTES, stream, p);
    HIP_CHECK(hipGetLastError());
}
```
